```python
import numpy as np

kernel_name = "gpu_ag_gemm_m4096_k4096_n16384_e4m3e5m2_dequant"


ABSMAX_RTOL = 0.05

COMPUTE_DTYPE = "float32"
EPILOGUE = "dequant"
IN_QUANT = None
OUT_QUANT = None
IN_FP8 = "e4m3e5m2"
IN_DTYPES = {"x": "e4m3", "w_mat": "e5m2",
             "scale_x": "float32", "scale_w": "float32"}
WORLD_SIZE = 8

def _snap_fp8(x, mant_bits, max_abs, min_normal):
    a = np.abs(x.astype(np.float64))
    _m, _e = np.frexp(a)
    q = np.where(a >= min_normal, np.ldexp(1.0, _e - 1 - mant_bits),
                 min_normal * 2.0 ** -mant_bits)
    snapped = np.minimum(np.round(a / q) * q, max_abs)
    return (np.sign(x) * snapped).astype(np.float32)

def setup_inputs():

    x = _snap_fp8(np.random.randn(4096, 4096).astype(np.float32), 3, 448.0, 2.0 ** -6)
    w_mat = _snap_fp8(np.random.randn(4096, 16384).astype(np.float32), 2, 57344.0, 2.0 ** -14)
    scale_x = (0.02 * (1.0 + np.random.rand(1))).astype(np.float32)
    scale_w = (0.02 * (1.0 + np.random.rand(1))).astype(np.float32)
    return {"x": x, "w_mat": w_mat, "scale_x": scale_x, "scale_w": scale_w}

def reference(x, w_mat, scale_x, scale_w):
    acc = (x @ w_mat).astype(np.float32)
    y = (acc * (scale_x * scale_w)).astype(np.float32)
    return y
```

```cpp
#include <hip/hip_runtime.h>
#include <rccl/rccl.h>
#include <cstdio>

#define HIP_CHECK(expr) do { hipError_t _e = (expr); if (_e != hipSuccess) { \
    fprintf(stderr, "HIP error %s at %s:%d\n", hipGetErrorString(_e), __FILE__, __LINE__); \
    return; } } while (0)

typedef unsigned u32x4 __attribute__((ext_vector_type(4)));
#define LDS __attribute__((address_space(3)))

constexpr int NR = 8;
constexpr int KD = 4096, NL = 2048, MS = 512;
__host__ __device__ constexpr unsigned dt_of(unsigned link, unsigned rb) { return 12 + ((3u * (2 * link + 1 - rb)) >> 2); }
constexpr unsigned SHARD = MS * KD;
constexpr size_t WS_CNT = 0;
constexpr size_t WS_TMO = 8192;
constexpr size_t WS_TK = 12288;
constexpr size_t WS_ECNT = 16384;
constexpr size_t WS_ENCF = 20480;
constexpr size_t WS_FLAG = 65536;
constexpr size_t WS_STG = 1u << 20;
constexpr size_t WS_PK = WS_STG + 2ull * NR * SHARD;
constexpr unsigned PK_E = 512 * 3072, PK_RAW = 2u << 20;
constexpr size_t WS_NEED = WS_PK + (8u << 20);
constexpr int DL = 2;
constexpr int NR0 = 48;
constexpr int NR1 = 0;
constexpr int FNS = 8;
constexpr unsigned L_E = 8 * 16384;
constexpr unsigned L_ST = L_E;
constexpr unsigned L_CTL = L_E + 3 * 8192;
constexpr unsigned L_FULL = L_CTL, L_FREE = L_CTL + 16, L_READY = L_CTL + 32, L_FAIL = L_CTL + 36, L_ENC = L_CTL + 44;
constexpr unsigned L_SCR = L_CTL + 64;
constexpr unsigned LDS_BYTES = L_SCR + 3 * 640;
constexpr unsigned long long DEADLINE = 50000000ull;

struct IntTab { unsigned v[72]; constexpr IntTab() : v() { for (unsigned i = 0; i < 72; ++i) v[i] = i; } };
__device__ const IntTab int_tab = IntTab();

struct Params {
  char* ws[NR];
  const char* x;
  const char* w;
  const float* sx;
  const float* sw;
  float* out;
  unsigned long long* dbg;
  int rank;
  int pad;
};

template <int N> __device__ __forceinline__ void waitvm() { asm volatile("s_waitcnt vmcnt(%0)" :: "i"(N) : "memory"); }
__device__ __forceinline__ unsigned rfl(unsigned v) { return __builtin_amdgcn_readfirstlane(v); }
__device__ __forceinline__ unsigned lds_ld(unsigned a) { return *(volatile LDS unsigned*)(size_t)a; }
__device__ __forceinline__ void lds_st(unsigned a, unsigned v) { *(volatile LDS unsigned*)(size_t)a = v; }
__device__ __forceinline__ unsigned umin(unsigned a, unsigned b) { return a < b ? a : b; }
__device__ __forceinline__ unsigned next_call(char* ws, int b, int wid, int lane) {
  unsigned* const cnt = (unsigned*)(ws + WS_CNT) + b * 8 + wid;
  const unsigned call = rfl(__hip_atomic_load(cnt, __ATOMIC_RELAXED, __HIP_MEMORY_SCOPE_AGENT) + 1);
  if (lane == 0) __hip_atomic_store(cnt, call, __ATOMIC_RELAXED, __HIP_MEMORY_SCOPE_AGENT);
  return call;
}

constexpr unsigned E_SHORT = 384 * 8, E_BYTES = 512, E_BITS = (E_BYTES - 8) * 8, E_HDR = E_BYTES - 4;
__device__ __forceinline__ unsigned vperm(unsigned a, unsigned b, unsigned s) { return __builtin_amdgcn_perm(a, b, s); }
__device__ __forceinline__ unsigned alignbit(unsigned hi, unsigned lo, unsigned s) { return __builtin_amdgcn_alignbit(hi, lo, s); }
__device__ __forceinline__ unsigned bfi(unsigned m, unsigned a, unsigned b) { return (a & m) | (b & ~m); }
__device__ __forceinline__ unsigned wave_incl_scan(unsigned v) {
  v += __builtin_amdgcn_update_dpp(0u, v, 0x111, 0xf, 0xf, false);
  v += __builtin_amdgcn_update_dpp(0u, v, 0x112, 0xf, 0xf, false);
  v += __builtin_amdgcn_update_dpp(0u, v, 0x114, 0xf, 0xf, false);
  v += __builtin_amdgcn_update_dpp(0u, v, 0x118, 0xf, 0xf, false);
  v += __builtin_amdgcn_update_dpp(0u, v, 0x142, 0xa, 0xf, false);
  v += __builtin_amdgcn_update_dpp(0u, v, 0x143, 0xc, 0xf, false);
  return v;
}
__device__ __forceinline__ void enc_init(unsigned lds, int lane) {
  *(volatile LDS unsigned long long*)(size_t)(lds + lane * 8) = 0ull;
  unsigned sel = 0x0c0c0c0cu;
#pragma unroll
  for (int k = 3; k >= 0; --k) if (lane >> k & 1) sel = (sel << 8) | (unsigned)k;
  if (lane < 16) *(volatile LDS unsigned*)(size_t)(lds + 512 + lane * 4) = sel;
}
__device__ __forceinline__ void enc_nibbles(const u32x4 (&x)[4], u32x4 (&N)[2]) {
#pragma unroll
  for (int i = 0; i < 4; ++i) {
    N[i >> 1][(i & 1) * 2 + 0] = bfi(0x87878787u, x[i][0], alignbit(x[i][1], x[i][1], 28));
    N[i >> 1][(i & 1) * 2 + 1] = bfi(0x87878787u, x[i][2], alignbit(x[i][3], x[i][3], 28));
  }
}
template <class F>
__device__ __forceinline__ unsigned enc_block(const u32x4 (&x)[4], unsigned lds, int lane, F&& put_codes) {
  u32x4 P;
  unsigned w[8], nw[8], bad = 0, tot = 0;
#pragma unroll
  for (int i = 0; i < 4; ++i) {
    unsigned bits[4], nb[4], pp = 0;
#pragma unroll
    for (int j = 0; j < 4; ++j) {
      const unsigned v = x[i][j];
      const unsigned ex = (v >> 3) & 0x0f0f0f0fu;
      const unsigned hi = (v >> 6) & 0x01010101u;
      const unsigned hi3 = hi * 3u;
      const unsigned s = ex - hi3;
      bad |= s;
      const unsigned c = vperm(0x02010003u, 0x03030303u, ex & 0x07070707u) | hi3;
      const unsigned m = c & (c >> 1) & 0x01010101u;
      const unsigned idx = __builtin_amdgcn_udot4(m, 0x20100804u, lds + 512, false);
      const unsigned cs = vperm(0u, s, *(LDS unsigned*)(size_t)idx);
      bits[j] = __builtin_amdgcn_udot4(cs, 0x00400801u, (cs >> 15) & 0xe00u, false);
      nb[j] = __builtin_amdgcn_udot4(m, 0x03030303u, 0u, false);
      pp |= __builtin_amdgcn_udot4(c, 0x40100401u, 0u, false) << (8 * j);
    }
    P[i] = pp;
    w[2 * i] = bits[0] | (bits[1] << nb[0]);          nw[2 * i] = nb[0] + nb[1];
    w[2 * i + 1] = bits[2] | (bits[3] << nb[2]);      nw[2 * i + 1] = nb[2] + nb[3];
    tot += nw[2 * i] + nw[2 * i + 1];
  }
  put_codes(P);
  const unsigned incl = wave_incl_scan(tot);
  const unsigned total = __builtin_amdgcn_readlane(incl, 63);
  const bool fit = total <= E_BITS && !__any((bad & 0x08080808u) != 0);
  if (fit) {
    unsigned o = incl - tot;
#pragma unroll
    for (int k = 0; k < 8; ++k) {
      const unsigned long long t = (unsigned long long)w[k] << (o & 31);
      LDS unsigned* a = (LDS unsigned*)(size_t)(lds + ((o >> 5) << 2));
      __hip_atomic_fetch_or(a, (unsigned)t, __ATOMIC_RELAXED, __HIP_MEMORY_SCOPE_WORKGROUP);
      __hip_atomic_fetch_or(a + 1, (unsigned)(t >> 32), __ATOMIC_RELAXED, __HIP_MEMORY_SCOPE_WORKGROUP);
      o += nw[k];
    }
  } else if (lane == 0) {
    *(volatile LDS unsigned*)(size_t)(lds + E_HDR) = 1u;
  }
  return fit ? (total > E_SHORT ? 1u : 0u) : 2u;
}
constexpr unsigned T_ESC = 0, T_SEL = 16384, T_END = T_SEL + 65536;
typedef unsigned u32x2 __attribute__((ext_vector_type(2)));
__device__ __forceinline__ void dec_tables(int t) {
  unsigned sel = 0, n = 0;
#pragma unroll
  for (int k = 0; k < 4; ++k) {
    const unsigned c = (t >> (2 * k)) & 3;
    sel |= (c < 3 ? c : 4 + n) << (8 * k);
    n += c == 3;
  }
  const u32x2 e = {sel, 3 * n};
#pragma unroll 8
  for (int r = 0; r < 32; ++r) *(volatile LDS u32x2*)(size_t)(T_SEL + t * 256 + ((r + t) & 31) * 8) = e;
#pragma unroll 4
  for (int r = 0; r < 16; ++r) {
    const unsigned x = r * 256 + t;
    const unsigned u = (x & 0x3fu) | ((x << 10) & 0x3f0000u);
    *(volatile LDS unsigned*)(size_t)(T_ESC + x * 4) = vperm(0x50484020u, 0x18100800u, (u | (u << 5)) & 0x07070707u);
  }
}
template <class F>
__device__ __forceinline__ void dec_block(u32x4 (&r)[4], const u32x4& P, const u32x4& N0, const u32x4& N1, unsigned le, int lane, F&& between) {
  u32x2 s[4][4];
  unsigned o[4], d[4][3], eb[4][4], tot = 0;
  const unsigned l8 = (lane & 31) * 8;
#pragma unroll
  for (int i = 0; i < 4; ++i) {
    o[i] = tot;
    tot += __builtin_popcount(P[i] & (P[i] >> 1) & 0x55555555u);
  }
  const unsigned base = wave_incl_scan(tot) - tot + 10;
#pragma unroll
  for (int i = 0; i < 4; ++i) {
    o[i] = umin((o[i] + base) * 3u, E_BITS + 30);
    const LDS unsigned* a = (const LDS unsigned*)(size_t)(le - 4 + ((o[i] >> 5) << 2));
    d[i][0] = a[0]; d[i][1] = a[1]; d[i][2] = a[2];
  }
  __builtin_amdgcn_sched_barrier(0);
#pragma unroll
  for (int j = 0; j < 4; ++j)
#pragma unroll
    for (int i = 0; i < 4; ++i) s[i][j] = *(const LDS u32x2*)(size_t)(vperm(P[i], l8, 0x0c0c0400u + (j << 8)) + T_SEL);
  __builtin_amdgcn_sched_barrier(0);
  between();
  unsigned w0[4], w1[4];
#pragma unroll
  for (int i = 0; i < 4; ++i) { w0[i] = alignbit(d[i][1], d[i][0], o[i]); w1[i] = alignbit(d[i][2], d[i][1], o[i]); }
#pragma unroll
  for (int j = 0; j < 4; ++j) {
#pragma unroll
    for (int i = 0; i < 4; ++i) {
      eb[i][j] = *(const LDS unsigned*)(size_t)((w0[i] & 0x3ffcu) + T_ESC);
      if (j < 3) {
        w0[i] = alignbit(w1[i], w0[i], s[i][j][1]);
        if (j < 1) w1[i] >>= s[i][j][1];
      }
    }
    __builtin_amdgcn_sched_barrier(0);
  }
  between();
#pragma unroll
  for (int j = 0; j < 4; ++j)
#pragma unroll
    for (int i = 0; i < 4; ++i) {
      const unsigned nn = (i & 2) ? N1[(i & 1) * 2 + (j >> 1)] : N0[(i & 1) * 2 + (j >> 1)];
      r[i][j] = bfi(0x87878787u, (j & 1) ? alignbit(nn, nn, 4) : nn, vperm(eb[i][j], 0x00383028u, s[i][j][0]));
    }
}

__device__ __forceinline__ void st16_sys(char* p, u32x4 v) {
  asm volatile("global_store_dwordx4 %0, %1, off sc0 sc1\n\ts_nop 1" :: "v"(p), "v"(v) : "memory");
}

__device__ __forceinline__ bool late(unsigned& spins, unsigned long long t0, unsigned long long lim) {
  __builtin_amdgcn_s_sleep(1);
  return (++spins & 1023u) == 0 && __builtin_amdgcn_s_memrealtime() - t0 > lim;
}

#define DBGBIT(x) 0
#define LANE_VARS int tz_ = threadIdx.x; asm volatile("" : "+v"(tz_)); const int lane = tz_ & 63, r8 = lane >> 3; \
  const unsigned sw0 = ((lane & 7) ^ (r8 >> 1)) << 4, sw1 = ((lane & 7) ^ (4 + (r8 >> 1))) << 4; (void)sw0; (void)sw1;

__global__ __launch_bounds__(512, 2) void ag_gemm(Params p) {
  extern __shared__ __attribute__((aligned(16))) char smem[];
  const int wid = rfl(threadIdx.x >> 6);
  const int b = blockIdx.x;
  const unsigned long long t0 = __builtin_amdgcn_s_memrealtime();
  if (4 && wid == 5 && (b >> 3) % 4 == 0) {
    unsigned long long pc;
    unsigned len;
    asm volatile("s_getpc_b64 %0\n\tAG_E0:\n\ts_mov_b32 %1, AG_END-AG_E0" : "=s"(pc), "=s"(len));
    const __amdgpu_buffer_rsrc_t rc = __builtin_amdgcn_make_buffer_rsrc((void*)pc, 0, len, 0x00020000);
#pragma unroll
    for (int i = 0; i < 8; ++i)
      asm volatile("buffer_load_dword v255, %0, %1, %2 offen" :: "v"((threadIdx.x & 63) * 128), "s"(rc), "s"(i * 8192) : "v255");
  }
  char* const myws = p.ws[0];
  const int xcd = b & 7, jj = b >> 3;
  const int cb = (xcd & 3) * 4 + (jj & 3);
  const int rb = ((xcd >> 2) ^ (jj >> 2)) & 1;
  const int src = (p.rank + (jj >> 2)) & 7;
  const bool local = jj < 4;
  const int eg0 = ((jj >> 2) - 1) * 2 + rb, eg1 = 13 + (jj >> 2);
  const int dt = rfl(dt_of((jj >> 2) - 1, rb)), TSTEPS = 32 + dt;
  const bool ext1 = (jj >> 2) <= 2 && rb == 0;

  if (local) {
    LANE_VARS
    const int f = wid & 1;
    const int sx = (f ? ((jj + (1 ? xcd : 0)) & 3) * 2 + 1 : jj * 2);
    const bool extra = sx == NR - 1;
    const int pi = extra ? (xcd == 7 ? 0 : xcd) : sx;
    constexpr unsigned L_META = T_END + 2 * FNS * 4096, L_TBL = L_META + 256;
    static_assert(L_TBL + 64 <= LDS_BYTES, "");
    if (wid < 6 || extra || DBGBIT(1)) {
      if (threadIdx.x < 80) lds_st(L_META + threadIdx.x * 4, 0);
      asm volatile("s_waitcnt lgkmcnt(0)" ::: "memory");
      __builtin_amdgcn_s_barrier();
    }
    if (wid >= 2 && wid < 6) {
      dec_tables(threadIdx.x - 128);
      asm volatile("s_waitcnt lgkmcnt(0)" ::: "memory");
      if (lane == 0) __hip_atomic_fetch_add((LDS unsigned*)(size_t)L_TBL, 1u, __ATOMIC_RELAXED, __HIP_MEMORY_SCOPE_WORKGROUP);
    }
    if (extra && (!1 || xcd == 7)) return;
    char* const pws = p.ws[1 + pi];
    const int peer = (p.rank + 1 + pi) & 7;
    const unsigned lring = T_END + f * FNS * 4096, lmeta = L_META + f * 128;
    unsigned call;
    if (wid >= 6 && !extra) {
      const unsigned long long xb = (unsigned long long)p.x, db = (unsigned long long)(pws + WS_STG + (size_t)p.rank * SHARD);
      asm volatile(
        "v_add_u32 v48, 0, %[vx0]\n\t"
        "v_add_u32 v49, 32768, %[vx1]\n\t"
        "v_add_u32 v50, 65536, %[vx0]\n\t"
        "v_add_u32 v51, 98304, %[vx1]\n\t"
        "v_add_u32 v52, 131072, %[vx0]\n\t"
        "v_add_u32 v53, 163840, %[vx1]\n\t"
        "v_add_u32 v54, 196608, %[vx0]\n\t"
        "v_add_u32 v55, 229376, %[vx1]\n\t"
        "v_mov_b32 v56, %[vd]\n\t"
        "v_add_u32 v57, 4096, %[vd]\n\t"
        "v_mov_b32 v59, 0\n\t"
        "global_load_dword v60, v59, %[cnt] sc1\n\t"
        "s_add_u32 s90, %[w], 0\n\t"
        "s_lshr_b32 s91, s90, 1\n\t"
        "s_and_b32 s92, s90, 1\n\t"
        "s_sub_u32 s93, s91, 2\n\t"
        "s_and_b32 s94, s93, 1\n\t"
        "s_mul_i32 s95, s94, 2\n\t"
        "s_lshr_b32 s93, s93, 1\n\t"
        "s_add_u32 s93, s93, s95\n\t"
        "s_xor_b32 s94, s94, 1\n\t"
        "s_cmp_lt_u32 s91, 2\n\t"
        "s_cselect_b32 s93, s91, s93\n\t"
        "s_cselect_b32 s94, 0, s94\n\t"
        "s_sub_u32 s95, s91, 64\n\t"
        "s_cmp_ge_u32 s91, 126\n\t"
        "s_cselect_b32 s93, s95, s93\n\t"
        "s_cselect_b32 s94, 1, s94\n\t"
        "s_lshl_b32 s95, s94, 6\n\t"
        "s_add_u32 s95, s95, s93\n\t"
        "s_lshl_b32 s95, s95, 1\n\t"
        "s_add_u32 s95, s95, s92\n\t"
        "s_lshr_b32 s91, s95, 1\n\t"
        "s_lshl_b32 s91, s91, 7\n\t"
        "s_lshl3_add_u32 s83, s92, s91\n\t"
        "s_lshl_b32 s82, s95, 13\n\t"
        "s_lshl_b32 s95, s94, 2\n\t"
        "s_add_u32 s95, s95, s92\n\t"
        "s_lshl_b32 s95, s95, 18\n\t"
        "s_lshl_b32 s93, s93, 7\n\t"
        "s_add_u32 s95, s95, s93\n\t"
        "s_add_u32 s80, %[xlo], s95\n\t"
        "s_addc_u32 s81, %[xhi], 0\n\t"
        "global_load_dwordx4 v[64:67], v48, s[80:81]\n\t"
        "global_load_dwordx4 v[68:71], v49, s[80:81]\n\t"
        "global_load_dwordx4 v[72:75], v50, s[80:81]\n\t"
        "global_load_dwordx4 v[76:79], v51, s[80:81]\n\t"
        "global_load_dwordx4 v[80:83], v52, s[80:81]\n\t"
        "global_load_dwordx4 v[84:87], v53, s[80:81]\n\t"
        "global_load_dwordx4 v[88:91], v54, s[80:81]\n\t"
        "global_load_dwordx4 v[92:95], v55, s[80:81]\n\t"
        "s_add_u32 s90, %[w], 8\n\t"
        "s_lshr_b32 s91, s90, 1\n\t"
        "s_and_b32 s92, s90, 1\n\t"
        "s_sub_u32 s93, s91, 2\n\t"
        "s_and_b32 s94, s93, 1\n\t"
        "s_mul_i32 s95, s94, 2\n\t"
        "s_lshr_b32 s93, s93, 1\n\t"
        "s_add_u32 s93, s93, s95\n\t"
        "s_xor_b32 s94, s94, 1\n\t"
        "s_cmp_lt_u32 s91, 2\n\t"
        "s_cselect_b32 s93, s91, s93\n\t"
        "s_cselect_b32 s94, 0, s94\n\t"
        "s_sub_u32 s95, s91, 64\n\t"
        "s_cmp_ge_u32 s91, 126\n\t"
        "s_cselect_b32 s93, s95, s93\n\t"
        "s_cselect_b32 s94, 1, s94\n\t"
        "s_lshl_b32 s95, s94, 6\n\t"
        "s_add_u32 s95, s95, s93\n\t"
        "s_lshl_b32 s95, s95, 1\n\t"
        "s_add_u32 s95, s95, s92\n\t"
        "s_lshr_b32 s91, s95, 1\n\t"
        "s_lshl_b32 s91, s91, 7\n\t"
        "s_lshl3_add_u32 s85, s92, s91\n\t"
        "s_lshl_b32 s84, s95, 13\n\t"
        "s_lshl_b32 s95, s94, 2\n\t"
        "s_add_u32 s95, s95, s92\n\t"
        "s_lshl_b32 s95, s95, 18\n\t"
        "s_lshl_b32 s93, s93, 7\n\t"
        "s_add_u32 s95, s95, s93\n\t"
        "s_add_u32 s80, %[xlo], s95\n\t"
        "s_addc_u32 s81, %[xhi], 0\n\t"
        "global_load_dwordx4 v[96:99], v48, s[80:81]\n\t"
        "global_load_dwordx4 v[100:103], v49, s[80:81]\n\t"
        "global_load_dwordx4 v[104:107], v50, s[80:81]\n\t"
        "global_load_dwordx4 v[108:111], v51, s[80:81]\n\t"
        "global_load_dwordx4 v[112:115], v52, s[80:81]\n\t"
        "global_load_dwordx4 v[116:119], v53, s[80:81]\n\t"
        "global_load_dwordx4 v[120:123], v54, s[80:81]\n\t"
        "global_load_dwordx4 v[124:127], v55, s[80:81]\n\t"
        "s_add_u32 s90, %[w], 16\n\t"
        "s_lshr_b32 s91, s90, 1\n\t"
        "s_and_b32 s92, s90, 1\n\t"
        "s_sub_u32 s93, s91, 2\n\t"
        "s_and_b32 s94, s93, 1\n\t"
        "s_mul_i32 s95, s94, 2\n\t"
        "s_lshr_b32 s93, s93, 1\n\t"
        "s_add_u32 s93, s93, s95\n\t"
        "s_xor_b32 s94, s94, 1\n\t"
        "s_cmp_lt_u32 s91, 2\n\t"
        "s_cselect_b32 s93, s91, s93\n\t"
        "s_cselect_b32 s94, 0, s94\n\t"
        "s_sub_u32 s95, s91, 64\n\t"
        "s_cmp_ge_u32 s91, 126\n\t"
        "s_cselect_b32 s93, s95, s93\n\t"
        "s_cselect_b32 s94, 1, s94\n\t"
        "s_lshl_b32 s95, s94, 6\n\t"
        "s_add_u32 s95, s95, s93\n\t"
        "s_lshl_b32 s95, s95, 1\n\t"
        "s_add_u32 s95, s95, s92\n\t"
        "s_lshr_b32 s91, s95, 1\n\t"
        "s_lshl_b32 s91, s91, 7\n\t"
        "s_lshl3_add_u32 s87, s92, s91\n\t"
        "s_lshl_b32 s86, s95, 13\n\t"
        "s_lshl_b32 s95, s94, 2\n\t"
        "s_add_u32 s95, s95, s92\n\t"
        "s_lshl_b32 s95, s95, 18\n\t"
        "s_lshl_b32 s93, s93, 7\n\t"
        "s_add_u32 s95, s95, s93\n\t"
        "s_add_u32 s80, %[xlo], s95\n\t"
        "s_addc_u32 s81, %[xhi], 0\n\t"
        "global_load_dwordx4 v[128:131], v48, s[80:81]\n\t"
        "global_load_dwordx4 v[132:135], v49, s[80:81]\n\t"
        "global_load_dwordx4 v[136:139], v50, s[80:81]\n\t"
        "global_load_dwordx4 v[140:143], v51, s[80:81]\n\t"
        "global_load_dwordx4 v[144:147], v52, s[80:81]\n\t"
        "global_load_dwordx4 v[148:151], v53, s[80:81]\n\t"
        "global_load_dwordx4 v[152:155], v54, s[80:81]\n\t"
        "global_load_dwordx4 v[156:159], v55, s[80:81]\n\t"
        "s_barrier\n\t"
        "s_waitcnt vmcnt(24)\n\t"
        "v_add_u32 v60, 1, v60\n\t"
        "s_nop 0\n\t"
        "v_readfirstlane_b32 %[call], v60\n\t"
        "s_and_b32 s90, %[call], 1\n\t"
        "s_lshl_b32 s90, s90, 24\n\t"
        "s_add_u32 s78, %[dlo], s90\n\t"
        "s_addc_u32 s79, %[dhi], 0\n\t"
        "s_waitcnt vmcnt(23)\n\t"
        "s_add_u32 s80, s78, s82\n\t"
        "s_addc_u32 s81, s79, 0\n\t"
        "global_store_dwordx4 v56, v[64:67], s[80:81] offset:0 sc0 sc1\n\t"
        "s_waitcnt vmcnt(23)\n\t"
        "global_store_dwordx4 v56, v[68:71], s[80:81] offset:1024 sc0 sc1\n\t"
        "s_waitcnt vmcnt(23)\n\t"
        "global_store_dwordx4 v56, v[72:75], s[80:81] offset:2048 sc0 sc1\n\t"
        "s_waitcnt vmcnt(23)\n\t"
        "global_store_dwordx4 v56, v[76:79], s[80:81] offset:3072 sc0 sc1\n\t"
        "s_waitcnt vmcnt(23)\n\t"
        "global_store_dwordx4 v57, v[80:83], s[80:81] offset:0 sc0 sc1\n\t"
        "s_waitcnt vmcnt(23)\n\t"
        "global_store_dwordx4 v57, v[84:87], s[80:81] offset:1024 sc0 sc1\n\t"
        "s_waitcnt vmcnt(23)\n\t"
        "global_store_dwordx4 v57, v[88:91], s[80:81] offset:2048 sc0 sc1\n\t"
        "s_waitcnt vmcnt(6)\n\t"
        "global_store_dwordx4 v57, v[92:95], s[80:81] offset:3072 sc0 sc1\n\t"
        "s_waitcnt vmcnt(6)\n\t"
        "s_add_u32 s80, s78, s84\n\t"
        "s_addc_u32 s81, s79, 0\n\t"
        "global_store_dwordx4 v56, v[96:99], s[80:81] offset:0 sc0 sc1\n\t"
        "s_waitcnt vmcnt(6)\n\t"
        "global_store_dwordx4 v56, v[100:103], s[80:81] offset:1024 sc0 sc1\n\t"
        "s_waitcnt vmcnt(6)\n\t"
        "global_store_dwordx4 v56, v[104:107], s[80:81] offset:2048 sc0 sc1\n\t"
        "s_waitcnt vmcnt(6)\n\t"
        "global_store_dwordx4 v56, v[108:111], s[80:81] offset:3072 sc0 sc1\n\t"
        "s_waitcnt vmcnt(6)\n\t"
        "global_store_dwordx4 v57, v[112:115], s[80:81] offset:0 sc0 sc1\n\t"
        "s_waitcnt vmcnt(6)\n\t"
        "global_store_dwordx4 v57, v[116:119], s[80:81] offset:1024 sc0 sc1\n\t"
        "s_waitcnt vmcnt(6)\n\t"
        "v_lshrrev_b32 v62, 4, %[vd]\n\t"
        "v_lshl_add_u32 v62, v62, 2, s83\n\t"
        "s_mov_b64 exec, 3\n\t"
        "global_store_dword v62, v60, %[fl] sc0 sc1\n\t"
        "s_mov_b64 exec, -1\n\t"
        "global_store_dwordx4 v57, v[120:123], s[80:81] offset:2048 sc0 sc1\n\t"
        "s_waitcnt vmcnt(7)\n\t"
        "global_store_dwordx4 v57, v[124:127], s[80:81] offset:3072 sc0 sc1\n\t"
        "s_waitcnt vmcnt(7)\n\t"
        "s_add_u32 s80, s78, s86\n\t"
        "s_addc_u32 s81, s79, 0\n\t"
        "global_store_dwordx4 v56, v[128:131], s[80:81] offset:0 sc0 sc1\n\t"
        "s_waitcnt vmcnt(7)\n\t"
        "global_store_dwordx4 v56, v[132:135], s[80:81] offset:1024 sc0 sc1\n\t"
        "s_waitcnt vmcnt(7)\n\t"
        "global_store_dwordx4 v56, v[136:139], s[80:81] offset:2048 sc0 sc1\n\t"
        "s_waitcnt vmcnt(7)\n\t"
        "global_store_dwordx4 v56, v[140:143], s[80:81] offset:3072 sc0 sc1\n\t"
        "s_waitcnt vmcnt(7)\n\t"
        "global_store_dwordx4 v57, v[144:147], s[80:81] offset:0 sc0 sc1\n\t"
        "s_waitcnt vmcnt(6)\n\t"
        "global_store_dwordx4 v57, v[148:151], s[80:81] offset:1024 sc0 sc1\n\t"
        "s_waitcnt vmcnt(6)\n\t"
        "v_lshrrev_b32 v62, 4, %[vd]\n\t"
        "v_lshl_add_u32 v62, v62, 2, s85\n\t"
        "s_mov_b64 exec, 3\n\t"
        "global_store_dword v62, v60, %[fl] sc0 sc1\n\t"
        "s_mov_b64 exec, -1\n\t"
        "global_store_dwordx4 v57, v[152:155], s[80:81] offset:2048 sc0 sc1\n\t"
        "s_waitcnt vmcnt(7)\n\t"
        "global_store_dwordx4 v57, v[156:159], s[80:81] offset:3072 sc0 sc1\n\t"
        "s_waitcnt vmcnt(0)\n\t"
        "v_lshrrev_b32 v62, 4, %[vd]\n\t"
        "v_lshl_add_u32 v62, v62, 2, s87\n\t"
        "s_mov_b64 exec, 3\n\t"
        "global_store_dword v62, v60, %[fl] sc0 sc1\n\t"
        "s_mov_b64 exec, -1\n\t"
        "s_mov_b64 exec, 1\n\t"
        "global_store_dword v59, v60, %[cnt] sc1\n\t"
        "s_mov_b64 exec, -1\n\t"
        : [call] "=&s"(call)
        : [w] "s"(xcd), [vx0] "v"(r8 * KD + sw0), [vx1] "v"(r8 * KD + sw1), [vd] "v"(lane * 16), [xlo] "s"((unsigned)xb), [xhi] "s"((unsigned)(xb >> 32)),
          [dlo] "s"((unsigned)db), [dhi] "s"((unsigned)(db >> 32)),
          [fl] "s"(pws + WS_FLAG + p.rank * 65536), [cnt] "s"((unsigned*)(myws + WS_CNT) + b * 8 + wid)
        : "memory", "scc", "s78", "s79", "s80", "s81", "s82", "s83", "s84", "s85", "s86", "s87", "s88", "s89", "s90", "s91", "s92", "s93", "s94", "s95", "v48", "v49", "v50", "v51", "v52", "v53", "v54", "v55", "v56", "v57", "v58", "v59", "v60", "v61", "v62", "v63", "v64", "v65", "v66", "v67", "v68", "v69", "v70", "v71", "v72", "v73", "v74", "v75", "v76", "v77", "v78", "v79", "v80", "v81", "v82", "v83", "v84", "v85", "v86", "v87", "v88", "v89", "v90", "v91", "v92", "v93", "v94", "v95", "v96", "v97", "v98", "v99", "v100", "v101", "v102", "v103", "v104", "v105", "v106", "v107", "v108", "v109", "v110", "v111", "v112", "v113", "v114", "v115", "v116", "v117", "v118", "v119", "v120", "v121", "v122", "v123", "v124", "v125", "v126", "v127", "v128", "v129", "v130", "v131", "v132", "v133", "v134", "v135", "v136", "v137", "v138", "v139", "v140", "v141", "v142", "v143", "v144", "v145", "v146", "v147", "v148", "v149", "v150", "v151", "v152", "v153", "v154", "v155", "v156", "v157", "v158", "v159");
    } else {
      call = next_call(myws, b, wid, lane);
    }
    char* const ppk = pws + WS_PK + (size_t)(call & 1) * (4u << 20);
    if (wid < 2) {
      if (1) {
        const unsigned junk = __hip_atomic_load((const unsigned*)(ppk + (lane & 7) * 262144 + 262080), __ATOMIC_RELAXED, __HIP_MEMORY_SCOPE_SYSTEM);
        asm volatile("" :: "v"(junk));
      }
      const unsigned long long pb = (unsigned long long)ppk, tp = t0 + DEADLINE;
      asm volatile(
        "v_mov_b32 v59, 0\n\t"
        "v_mov_b32 v58, 1\n\t"
        "s_mov_b64 exec, 1\n\t"
        "global_atomic_add v60, v59, v58, %[tk] sc0\n\t"
        "global_atomic_add v61, v59, v58, %[tk] sc0\n\t"
        "s_mov_b64 exec, -1\n\t"
        "s_mov_b32 s76, 0\n\t"
        "FW_%=:\n\t"
        "s_load_dwordx16 s[44:59], %[ef], 0 glc\n\t"
        "s_load_dwordx16 s[60:75], %[ef], 64 glc\n\t"
        "s_waitcnt lgkmcnt(0)\n\t"
        "s_min_u32 s90, s45, s47\n\t"
        "s_min_u32 s90, s90, s49\n\t"
        "s_min_u32 s90, s90, s51\n\t"
        "s_min_u32 s90, s90, s53\n\t"
        "s_min_u32 s90, s90, s55\n\t"
        "s_min_u32 s90, s90, s57\n\t"
        "s_min_u32 s90, s90, s59\n\t"
        "s_min_u32 s90, s90, s61\n\t"
        "s_min_u32 s90, s90, s63\n\t"
        "s_min_u32 s90, s90, s65\n\t"
        "s_min_u32 s90, s90, s67\n\t"
        "s_min_u32 s90, s90, s69\n\t"
        "s_min_u32 s90, s90, s71\n\t"
        "s_min_u32 s90, s90, s73\n\t"
        "s_min_u32 s90, s90, s75\n\t"
        "s_cmp_ge_u32 s90, %[call]\n\t"
        "s_cbranch_scc1 FG_%=\n\t"
        "s_add_u32 s76, s76, 1\n\t"
        "s_and_b32 s90, s76, 255\n\t"
        "s_cbranch_scc1 FW_%=\n\t"
        "global_load_dword v57, v59, %[tmo] sc1\n\t"
        "s_memrealtime s[90:91]\n\t"
        "s_waitcnt vmcnt(0) lgkmcnt(0)\n\t"
        "v_readfirstlane_b32 s92, v57\n\t"
        "s_cmp_eq_u32 s92, %[call]\n\t"
        "s_cbranch_scc1 FFAIL_%=\n\t"
        "s_sub_u32 s90, %[tlo], s90\n\t"
        "s_subb_u32 s91, %[thi], s91\n\t"
        "s_cbranch_scc0 FW_%=\n\t"
        "FFAIL_%=:\n\t"
        "v_mov_b32 v57, %[call]\n\t"
        "s_mov_b64 exec, 1\n\t"
        "global_store_dword v59, v57, %[tmo] sc1\n\t"
        "s_mov_b64 exec, -1\n\t"
        "v_mov_b32 v57, 0x80000000\n\t"
        "ds_write_b32 %[vm], v57 offset:32\n\t"
        "s_branch FDONE_%=\n\t"
        "FG_%=:\n\t"
        "v_writelane_b32 v50, s44, 0\n\t"
        "v_writelane_b32 v51, s46, 0\n\t"
        "v_writelane_b32 v50, s48, 1\n\t"
        "v_writelane_b32 v51, s50, 1\n\t"
        "v_writelane_b32 v50, s52, 2\n\t"
        "v_writelane_b32 v51, s54, 2\n\t"
        "v_writelane_b32 v50, s56, 3\n\t"
        "v_writelane_b32 v51, s58, 3\n\t"
        "v_writelane_b32 v50, s60, 4\n\t"
        "v_writelane_b32 v51, s62, 4\n\t"
        "v_writelane_b32 v50, s64, 5\n\t"
        "v_writelane_b32 v51, s66, 5\n\t"
        "v_writelane_b32 v50, s68, 6\n\t"
        "v_writelane_b32 v51, s70, 6\n\t"
        "v_writelane_b32 v50, s72, 7\n\t"
        "v_writelane_b32 v51, s74, 7\n\t"
        "s_sub_u32 s77, %[call], 1\n\t"
        "s_mul_i32 s77, s77, 464\n\t"
        "s_sub_u32 s77, s77, 48\n\t"
        "s_mul_i32 s90, %[nw], 2\n\t"
        "s_sub_u32 s77, s77, s90\n\t"
        "s_add_u32 s89, %[w], 48\n\t"
        "s_mov_b32 s88, 0\n\t"
        "s_mov_b32 s87, 0\n\t"
        "s_mov_b32 s76, 0\n\t"
        "s_lshr_b32 s91, s89, 2\n\t"
        "s_and_b32 s92, s89, 3\n\t"
        "s_sub_u32 s93, s91, 2\n\t"
        "s_and_b32 s94, s93, 1\n\t"
        "s_mul_i32 s95, s94, 2\n\t"
        "s_lshr_b32 s93, s93, 1\n\t"
        "s_add_u32 s93, s93, s95\n\t"
        "s_xor_b32 s94, s94, 1\n\t"
        "s_cmp_lt_u32 s91, 2\n\t"
        "s_cselect_b32 s93, s91, s93\n\t"
        "s_cselect_b32 s94, 0, s94\n\t"
        "s_sub_u32 s95, s91, 64\n\t"
        "s_cmp_ge_u32 s91, 126\n\t"
        "s_cselect_b32 s93, s95, s93\n\t"
        "s_cselect_b32 s94, 1, s94\n\t"
        "s_lshl_b32 s95, s94, 6\n\t"
        "s_add_u32 s79, s95, s93\n\t"
        "s_lshl2_add_u32 s79, s79, s92\n\t"
        "s_cmp_eq_u32 s94, 0\n\t"
        "s_cselect_b32 s94, %[dt0], %[dt1]\n\t"
        "s_sub_u32 s90, s93, s94\n\t"
        "s_and_b32 s91, s90, 1\n\t"
        "s_lshl_b32 s91, s91, 5\n\t"
        "s_lshr_b32 s90, s90, 1\n\t"
        "s_add_u32 s90, s90, s94\n\t"
        "s_add_u32 s90, s90, s91\n\t"
        "s_cmp_lt_u32 s93, s94\n\t"
        "s_cselect_b32 s90, s93, s90\n\t"
        "s_add_u32 s91, s93, s94\n\t"
        "s_sub_u32 s91, s91, 32\n\t"
        "s_cmp_ge_i32 s91, 32\n\t"
        "s_cselect_b32 s90, s91, s90\n\t"
        "s_add_u32 s90, s90, s95\n\t"
        "s_lshl2_add_u32 s78, s90, s92\n\t"
        "s_pack_ll_b32_b16 s79, s79, s78\n\t"
        "s_mov_b32 s84, -1\n\t"
        "s_mov_b32 s85, 0\n\t"
        "s_cmp_lt_u32 s78, 256\n\t"
        "s_cselect_b32 s90, 28, 48\n\t"
        "s_sub_u32 s90, s78, s90\n\t"
        "s_and_b32 s91, s90, 7\n\t"
        "v_readlane_b32 s92, v50, s91\n\t"
        "v_readlane_b32 s93, v51, s91\n\t"
        "s_lshr_b32 s90, s90, 3\n\t"
        "s_bitcmp1_b64 s[92:93], s90\n\t"
        "s_cselect_b32 s84, s84, 0xffffff\n\t"
        "s_cselect_b32 s90, 0, 0x80000000\n\t"
        "s_or_b32 s79, s79, s90\n\t"
        "s_mul_i32 s90, s78, 3072\n\t"
        "s_lshl_b32 s91, s78, 9\n\t"
        "s_add_u32 s91, s91, %[eoff]\n\t"
        "s_add_u32 s80, %[plo], s90\n\t"
        "s_addc_u32 s81, %[phi], 0\n\t"
        "s_add_u32 s82, %[plo], s91\n\t"
        "s_addc_u32 s83, %[phi], 0\n\t"
        "s_and_b32 s90, s88, 7\n\t"
        "s_lshl_b32 s91, s90, 12\n\t"
        "s_add_u32 m0, s91, %[ring]\n\t"
        "s_lshl_b32 s90, s90, 2\n\t"
        "v_add_u32 v56, s90, %[vm]\n\t"
        "v_mov_b32 v57, s79\n\t"
        "ds_write_b32 v56, v57\n\t"
        "s_mov_b64 exec, 1\n\t"
        "global_load_dword v53, v59, %[tmo]\n\t"
        "s_mov_b64 exec, -1\n\t"
        "global_load_lds_dwordx4 %[vd], s[80:81] sc0 sc1\n\t"
        "global_load_lds_dwordx4 %[vd], s[80:81] offset:1024 sc0 sc1\n\t"
        "global_load_lds_dwordx4 %[vd], s[80:81] offset:2048 sc0 sc1\n\t"
        "s_mov_b64 exec, s[84:85]\n\t"
        "global_load_lds_dwordx4 %[vd], s[82:83] offset:3072 sc0 sc1\n\t"
        "s_mov_b64 exec, -1\n\t"
        "s_add_u32 s88, s88, 1\n\t"
        "s_and_b32 s90, s88, 7\n\t"
        "s_lshl_b32 s90, s90, 2\n\t"
        "v_add_u32 v55, s90, %[vm]\n\t"
        "ds_read_b32 v54, v55 offset:64\n\t"
        "s_add_u32 s89, s89, %[nw]\n\t"
        "s_lshr_b32 s91, s89, 2\n\t"
        "s_and_b32 s92, s89, 3\n\t"
        "s_sub_u32 s93, s91, 2\n\t"
        "s_and_b32 s94, s93, 1\n\t"
        "s_mul_i32 s95, s94, 2\n\t"
        "s_lshr_b32 s93, s93, 1\n\t"
        "s_add_u32 s93, s93, s95\n\t"
        "s_xor_b32 s94, s94, 1\n\t"
        "s_cmp_lt_u32 s91, 2\n\t"
        "s_cselect_b32 s93, s91, s93\n\t"
        "s_cselect_b32 s94, 0, s94\n\t"
        "s_sub_u32 s95, s91, 64\n\t"
        "s_cmp_ge_u32 s91, 126\n\t"
        "s_cselect_b32 s93, s95, s93\n\t"
        "s_cselect_b32 s94, 1, s94\n\t"
        "s_lshl_b32 s95, s94, 6\n\t"
        "s_add_u32 s79, s95, s93\n\t"
        "s_lshl2_add_u32 s79, s79, s92\n\t"
        "s_cmp_eq_u32 s94, 0\n\t"
        "s_cselect_b32 s94, %[dt0], %[dt1]\n\t"
        "s_sub_u32 s90, s93, s94\n\t"
        "s_and_b32 s91, s90, 1\n\t"
        "s_lshl_b32 s91, s91, 5\n\t"
        "s_lshr_b32 s90, s90, 1\n\t"
        "s_add_u32 s90, s90, s94\n\t"
        "s_add_u32 s90, s90, s91\n\t"
        "s_cmp_lt_u32 s93, s94\n\t"
        "s_cselect_b32 s90, s93, s90\n\t"
        "s_add_u32 s91, s93, s94\n\t"
        "s_sub_u32 s91, s91, 32\n\t"
        "s_cmp_ge_i32 s91, 32\n\t"
        "s_cselect_b32 s90, s91, s90\n\t"
        "s_add_u32 s90, s90, s95\n\t"
        "s_lshl2_add_u32 s78, s90, s92\n\t"
        "s_pack_ll_b32_b16 s79, s79, s78\n\t"
        "s_mov_b32 s84, -1\n\t"
        "s_mov_b32 s85, 0\n\t"
        "s_cmp_lt_u32 s78, 256\n\t"
        "s_cselect_b32 s90, 28, 48\n\t"
        "s_sub_u32 s90, s78, s90\n\t"
        "s_and_b32 s91, s90, 7\n\t"
        "v_readlane_b32 s92, v50, s91\n\t"
        "v_readlane_b32 s93, v51, s91\n\t"
        "s_lshr_b32 s90, s90, 3\n\t"
        "s_bitcmp1_b64 s[92:93], s90\n\t"
        "s_cselect_b32 s84, s84, 0xffffff\n\t"
        "s_cselect_b32 s90, 0, 0x80000000\n\t"
        "s_or_b32 s79, s79, s90\n\t"
        "s_mul_i32 s90, s78, 3072\n\t"
        "s_lshl_b32 s91, s78, 9\n\t"
        "s_add_u32 s91, s91, %[eoff]\n\t"
        "s_add_u32 s80, %[plo], s90\n\t"
        "s_addc_u32 s81, %[phi], 0\n\t"
        "s_add_u32 s82, %[plo], s91\n\t"
        "s_addc_u32 s83, %[phi], 0\n\t"
        "s_and_b32 s90, s88, 7\n\t"
        "s_lshl_b32 s91, s90, 12\n\t"
        "s_add_u32 m0, s91, %[ring]\n\t"
        "s_lshl_b32 s90, s90, 2\n\t"
        "v_add_u32 v56, s90, %[vm]\n\t"
        "v_mov_b32 v57, s79\n\t"
        "ds_write_b32 v56, v57\n\t"
        "s_mov_b64 exec, 1\n\t"
        "global_load_dword v53, v59, %[tmo]\n\t"
        "s_mov_b64 exec, -1\n\t"
        "global_load_lds_dwordx4 %[vd], s[80:81] sc0 sc1\n\t"
        "global_load_lds_dwordx4 %[vd], s[80:81] offset:1024 sc0 sc1\n\t"
        "global_load_lds_dwordx4 %[vd], s[80:81] offset:2048 sc0 sc1\n\t"
        "s_waitcnt vmcnt(7)\n\t"
        "s_mov_b64 exec, s[84:85]\n\t"
        "global_load_lds_dwordx4 %[vd], s[82:83] offset:3072 sc0 sc1\n\t"
        "s_mov_b64 exec, -1\n\t"
        "s_add_u32 s88, s88, 1\n\t"
        "s_and_b32 s90, s88, 7\n\t"
        "s_lshl_b32 s90, s90, 2\n\t"
        "v_add_u32 v55, s90, %[vm]\n\t"
        "ds_read_b32 v54, v55 offset:64\n\t"
        "s_waitcnt vmcnt(10)\n\t"
        "FLOOP_%=:\n\t"
        "v_readfirstlane_b32 s89, v60\n\t"
        "s_sub_u32 s89, s89, s77\n\t"
        "s_cmp_lt_u32 s89, 512\n\t"
        "s_cbranch_scc0 FEND_%=\n\t"
        "s_waitcnt lgkmcnt(0)\n\t"
        "FSP0_%=:\n\t"
        "v_readfirstlane_b32 s87, v54\n\t"
        "s_add_u32 s90, s87, 8\n\t"
        "s_cmp_lt_u32 s88, s90\n\t"
        "s_cbranch_scc1 FGO0_%=\n\t"
        "ds_read_b32 v54, v55 offset:64\n\t"
        "s_add_u32 s76, s76, 1\n\t"
        "s_waitcnt lgkmcnt(0)\n\t"
        "s_cmp_lt_u32 s76, 0x1000000\n\t"
        "s_cbranch_scc1 FSP0_%=\n\t"
        "FGO0_%=:\n\t"
        "s_lshr_b32 s91, s89, 2\n\t"
        "s_and_b32 s92, s89, 3\n\t"
        "s_sub_u32 s93, s91, 2\n\t"
        "s_and_b32 s94, s93, 1\n\t"
        "s_mul_i32 s95, s94, 2\n\t"
        "s_lshr_b32 s93, s93, 1\n\t"
        "s_add_u32 s93, s93, s95\n\t"
        "s_xor_b32 s94, s94, 1\n\t"
        "s_cmp_lt_u32 s91, 2\n\t"
        "s_cselect_b32 s93, s91, s93\n\t"
        "s_cselect_b32 s94, 0, s94\n\t"
        "s_sub_u32 s95, s91, 64\n\t"
        "s_cmp_ge_u32 s91, 126\n\t"
        "s_cselect_b32 s93, s95, s93\n\t"
        "s_cselect_b32 s94, 1, s94\n\t"
        "s_lshl_b32 s95, s94, 6\n\t"
        "s_add_u32 s79, s95, s93\n\t"
        "s_lshl2_add_u32 s79, s79, s92\n\t"
        "s_cmp_eq_u32 s94, 0\n\t"
        "s_cselect_b32 s94, %[dt0], %[dt1]\n\t"
        "s_sub_u32 s90, s93, s94\n\t"
        "s_and_b32 s91, s90, 1\n\t"
        "s_lshl_b32 s91, s91, 5\n\t"
        "s_lshr_b32 s90, s90, 1\n\t"
        "s_add_u32 s90, s90, s94\n\t"
        "s_add_u32 s90, s90, s91\n\t"
        "s_cmp_lt_u32 s93, s94\n\t"
        "s_cselect_b32 s90, s93, s90\n\t"
        "s_add_u32 s91, s93, s94\n\t"
        "s_sub_u32 s91, s91, 32\n\t"
        "s_cmp_ge_i32 s91, 32\n\t"
        "s_cselect_b32 s90, s91, s90\n\t"
        "s_add_u32 s90, s90, s95\n\t"
        "s_lshl2_add_u32 s78, s90, s92\n\t"
        "s_pack_ll_b32_b16 s79, s79, s78\n\t"
        "s_mov_b32 s84, -1\n\t"
        "s_mov_b32 s85, 0\n\t"
        "s_cmp_lt_u32 s78, 256\n\t"
        "s_cselect_b32 s90, 28, 48\n\t"
        "s_sub_u32 s90, s78, s90\n\t"
        "s_and_b32 s91, s90, 7\n\t"
        "v_readlane_b32 s92, v50, s91\n\t"
        "v_readlane_b32 s93, v51, s91\n\t"
        "s_lshr_b32 s90, s90, 3\n\t"
        "s_bitcmp1_b64 s[92:93], s90\n\t"
        "s_cselect_b32 s84, s84, 0xffffff\n\t"
        "s_cselect_b32 s90, 0, 0x80000000\n\t"
        "s_or_b32 s79, s79, s90\n\t"
        "s_mul_i32 s90, s78, 3072\n\t"
        "s_lshl_b32 s91, s78, 9\n\t"
        "s_add_u32 s91, s91, %[eoff]\n\t"
        "s_add_u32 s80, %[plo], s90\n\t"
        "s_addc_u32 s81, %[phi], 0\n\t"
        "s_add_u32 s82, %[plo], s91\n\t"
        "s_addc_u32 s83, %[phi], 0\n\t"
        "s_and_b32 s90, s88, 7\n\t"
        "s_lshl_b32 s91, s90, 12\n\t"
        "s_add_u32 m0, s91, %[ring]\n\t"
        "s_lshl_b32 s90, s90, 2\n\t"
        "v_add_u32 v56, s90, %[vm]\n\t"
        "v_mov_b32 v57, s79\n\t"
        "ds_write_b32 v56, v57\n\t"
        "s_waitcnt vmcnt(7)\n\t"
        "s_mov_b64 exec, 1\n\t"
        "global_atomic_add v60, v59, v58, %[tk] sc0\n\t"
        "s_mov_b64 exec, -1\n\t"
        "global_load_lds_dwordx4 %[vd], s[80:81] sc0 sc1\n\t"
        "s_waitcnt vmcnt(8)\n\t"
        "global_load_lds_dwordx4 %[vd], s[80:81] offset:1024 sc0 sc1\n\t"
        "s_waitcnt vmcnt(8)\n\t"
        "s_sub_u32 s90, s88, 1\n\t"
        "v_mov_b32 v57, s90\n\t"
        "ds_write_b32 %[vm], v57 offset:32\n\t"
        "global_load_lds_dwordx4 %[vd], s[80:81] offset:2048 sc0 sc1\n\t"
        "s_waitcnt vmcnt(7)\n\t"
        "s_mov_b64 exec, s[84:85]\n\t"
        "global_load_lds_dwordx4 %[vd], s[82:83] offset:3072 sc0 sc1\n\t"
        "s_mov_b64 exec, -1\n\t"
        "s_add_u32 s88, s88, 1\n\t"
        "s_and_b32 s90, s88, 7\n\t"
        "s_lshl_b32 s90, s90, 2\n\t"
        "v_add_u32 v55, s90, %[vm]\n\t"
        "ds_read_b32 v54, v55 offset:64\n\t"
        "v_readfirstlane_b32 s89, v61\n\t"
        "s_sub_u32 s89, s89, s77\n\t"
        "s_cmp_lt_u32 s89, 512\n\t"
        "s_cbranch_scc0 FEND_%=\n\t"
        "s_waitcnt lgkmcnt(0)\n\t"
        "FSP1_%=:\n\t"
        "v_readfirstlane_b32 s87, v54\n\t"
        "s_add_u32 s90, s87, 8\n\t"
        "s_cmp_lt_u32 s88, s90\n\t"
        "s_cbranch_scc1 FGO1_%=\n\t"
        "ds_read_b32 v54, v55 offset:64\n\t"
        "s_add_u32 s76, s76, 1\n\t"
        "s_waitcnt lgkmcnt(0)\n\t"
        "s_cmp_lt_u32 s76, 0x1000000\n\t"
        "s_cbranch_scc1 FSP1_%=\n\t"
        "FGO1_%=:\n\t"
        "s_lshr_b32 s91, s89, 2\n\t"
        "s_and_b32 s92, s89, 3\n\t"
        "s_sub_u32 s93, s91, 2\n\t"
        "s_and_b32 s94, s93, 1\n\t"
        "s_mul_i32 s95, s94, 2\n\t"
        "s_lshr_b32 s93, s93, 1\n\t"
        "s_add_u32 s93, s93, s95\n\t"
        "s_xor_b32 s94, s94, 1\n\t"
        "s_cmp_lt_u32 s91, 2\n\t"
        "s_cselect_b32 s93, s91, s93\n\t"
        "s_cselect_b32 s94, 0, s94\n\t"
        "s_sub_u32 s95, s91, 64\n\t"
        "s_cmp_ge_u32 s91, 126\n\t"
        "s_cselect_b32 s93, s95, s93\n\t"
        "s_cselect_b32 s94, 1, s94\n\t"
        "s_lshl_b32 s95, s94, 6\n\t"
        "s_add_u32 s79, s95, s93\n\t"
        "s_lshl2_add_u32 s79, s79, s92\n\t"
        "s_cmp_eq_u32 s94, 0\n\t"
        "s_cselect_b32 s94, %[dt0], %[dt1]\n\t"
        "s_sub_u32 s90, s93, s94\n\t"
        "s_and_b32 s91, s90, 1\n\t"
        "s_lshl_b32 s91, s91, 5\n\t"
        "s_lshr_b32 s90, s90, 1\n\t"
        "s_add_u32 s90, s90, s94\n\t"
        "s_add_u32 s90, s90, s91\n\t"
        "s_cmp_lt_u32 s93, s94\n\t"
        "s_cselect_b32 s90, s93, s90\n\t"
        "s_add_u32 s91, s93, s94\n\t"
        "s_sub_u32 s91, s91, 32\n\t"
        "s_cmp_ge_i32 s91, 32\n\t"
        "s_cselect_b32 s90, s91, s90\n\t"
        "s_add_u32 s90, s90, s95\n\t"
        "s_lshl2_add_u32 s78, s90, s92\n\t"
        "s_pack_ll_b32_b16 s79, s79, s78\n\t"
        "s_mov_b32 s84, -1\n\t"
        "s_mov_b32 s85, 0\n\t"
        "s_cmp_lt_u32 s78, 256\n\t"
        "s_cselect_b32 s90, 28, 48\n\t"
        "s_sub_u32 s90, s78, s90\n\t"
        "s_and_b32 s91, s90, 7\n\t"
        "v_readlane_b32 s92, v50, s91\n\t"
        "v_readlane_b32 s93, v51, s91\n\t"
        "s_lshr_b32 s90, s90, 3\n\t"
        "s_bitcmp1_b64 s[92:93], s90\n\t"
        "s_cselect_b32 s84, s84, 0xffffff\n\t"
        "s_cselect_b32 s90, 0, 0x80000000\n\t"
        "s_or_b32 s79, s79, s90\n\t"
        "s_mul_i32 s90, s78, 3072\n\t"
        "s_lshl_b32 s91, s78, 9\n\t"
        "s_add_u32 s91, s91, %[eoff]\n\t"
        "s_add_u32 s80, %[plo], s90\n\t"
        "s_addc_u32 s81, %[phi], 0\n\t"
        "s_add_u32 s82, %[plo], s91\n\t"
        "s_addc_u32 s83, %[phi], 0\n\t"
        "s_and_b32 s90, s88, 7\n\t"
        "s_lshl_b32 s91, s90, 12\n\t"
        "s_add_u32 m0, s91, %[ring]\n\t"
        "s_lshl_b32 s90, s90, 2\n\t"
        "v_add_u32 v56, s90, %[vm]\n\t"
        "v_mov_b32 v57, s79\n\t"
        "ds_write_b32 v56, v57\n\t"
        "s_waitcnt vmcnt(7)\n\t"
        "s_mov_b64 exec, 1\n\t"
        "global_atomic_add v61, v59, v58, %[tk] sc0\n\t"
        "s_mov_b64 exec, -1\n\t"
        "global_load_lds_dwordx4 %[vd], s[80:81] sc0 sc1\n\t"
        "s_waitcnt vmcnt(8)\n\t"
        "global_load_lds_dwordx4 %[vd], s[80:81] offset:1024 sc0 sc1\n\t"
        "s_waitcnt vmcnt(8)\n\t"
        "s_sub_u32 s90, s88, 1\n\t"
        "v_mov_b32 v57, s90\n\t"
        "ds_write_b32 %[vm], v57 offset:32\n\t"
        "global_load_lds_dwordx4 %[vd], s[80:81] offset:2048 sc0 sc1\n\t"
        "s_waitcnt vmcnt(7)\n\t"
        "s_mov_b64 exec, s[84:85]\n\t"
        "global_load_lds_dwordx4 %[vd], s[82:83] offset:3072 sc0 sc1\n\t"
        "s_mov_b64 exec, -1\n\t"
        "s_add_u32 s88, s88, 1\n\t"
        "s_and_b32 s90, s88, 7\n\t"
        "s_lshl_b32 s90, s90, 2\n\t"
        "v_add_u32 v55, s90, %[vm]\n\t"
        "ds_read_b32 v54, v55 offset:64\n\t"
        "s_branch FLOOP_%=\n\t"
        "FEND_%=:\n\t"
        "s_waitcnt vmcnt(5)\n\t"
        "s_sub_u32 s90, s88, 1\n\t"
        "v_mov_b32 v57, s90\n\t"
        "ds_write_b32 %[vm], v57 offset:32\n\t"
        "s_waitcnt vmcnt(0)\n\t"
        "s_or_b32 s90, s88, 0x80000000\n\t"
        "v_mov_b32 v57, s90\n\t"
        "ds_write_b32 %[vm], v57 offset:32\n\t"
        "FDONE_%=:\n\t"
        "AG_END:\n\t"
        "s_waitcnt vmcnt(0) lgkmcnt(0)\n\t"
        :
        : [w] "s"(rfl(extra ? 8 : xcd)), [nw] "s"(1 ? 9 : 8), [call] "s"(call), [vd] "v"(lane * 16), [vm] "v"(lmeta),
          [ring] "s"(lring), [plo] "s"((unsigned)pb), [phi] "s"((unsigned)(pb >> 32)), [eoff] "s"(PK_E - 3072), [roff] "s"(PK_RAW), [tk] "s"(myws + WS_TK + pi * 128),
          [ef] "s"(myws + WS_ENCF + ((call & 1) * 8 + peer) * 128), [tmo] "s"(myws + WS_TMO), [tlo] "s"((unsigned)tp), [thi] "s"((unsigned)(tp >> 32)),
          [dt0] "s"(rfl(dt_of(pi, 0))), [dt1] "s"(rfl(dt_of(pi, 1)))
        : "memory", "scc", "s44", "s45", "s46", "s47", "s48", "s49", "s50", "s51", "s52", "s53", "s54", "s55", "s56", "s57", "s58", "s59", "s60", "s61", "s62", "s63", "s64", "s65", "s66", "s67", "s68", "s69", "s70", "s71", "s72", "s73", "s74", "s75", "s76", "s77", "s78", "s79", "s80", "s81", "s82", "s83", "s84", "s85", "s86", "s87", "s88", "s89", "s90", "s91", "s92", "s93", "s94", "s95", "v50", "v51", "v52", "v53", "v54", "v55", "v56", "v57", "v58", "v59", "v60", "v61", "v62", "v63");
      return;
    }
    char* const stg = myws + WS_STG + (size_t)(call & 1) * NR * SHARD + (size_t)peer * SHARD;
    unsigned* const flg = (unsigned*)(myws + WS_FLAG + peer * 65536);
    unsigned pend = ~0u, spins = 0;
    auto flag_out = [&](bool wait) {
      unsigned tmp, vtmp;
      if (wait) asm volatile("s_waitcnt vmcnt(0)" ::: "memory");
      asm volatile("s_cmp_eq_u32 %0, -1\n\ts_cbranch_scc1 NF_%=\n\t"
                   "s_getreg_b32 %1, hwreg(HW_REG_IB_STS)\n\ts_and_b32 %1, %1, 0xc0000f\n\ts_cbranch_scc1 NF_%=\n\t"
                   "s_waitcnt vmcnt(0)\n\tv_mov_b32 %2, %0\n\ts_mov_b64 exec, 1\n\tglobal_store_dword %2, %3, %4 sc0 sc1\n\ts_mov_b64 exec, -1\n\t"
                   "s_mov_b32 %0, -1\n\tNF_%=:"
                   : "+s"(pend), "=&s"(tmp), "=&v"(vtmp) : "v"(call), "s"(flg) : "scc");
    };
    while (rfl(lds_ld(L_TBL)) < 4) if (late(spins, t0, DEADLINE)) break;
#define TRF() do {} while (0)
    while (true) {
      unsigned n = 0;
      if (lane == 0) n = __hip_atomic_fetch_add((LDS unsigned*)(size_t)(lmeta + 4 * FNS + 4), 1u, __ATOMIC_RELAXED, __HIP_MEMORY_SCOPE_WORKGROUP);
      n = rfl(n);
      unsigned full = rfl(lds_ld(lmeta + 4 * FNS));
      while ((full & 0x7fffffffu) <= n && !(full >> 31)) {
        flag_out(false); TRF();
        if (late(spins, t0, 2 * DEADLINE)) break;
        full = rfl(lds_ld(lmeta + 4 * FNS));
      }
      if ((full & 0x7fffffffu) <= n) break;
      asm volatile("" ::: "memory");
      const unsigned ls = lring + (n & (FNS - 1)) * 4096;
      const u32x4 c0 = *(LDS u32x4*)(size_t)(ls + lane * 16);
      const unsigned qv = *(LDS unsigned*)(size_t)(lmeta + (n & (FNS - 1)) * 4), hv = *(LDS unsigned*)(size_t)(ls + 3072 + E_HDR);
      const u32x4 c1 = *(LDS u32x4*)(size_t)(ls + 1024 + lane * 16), c2 = *(LDS u32x4*)(size_t)(ls + 2048 + lane * 16);
      __builtin_amdgcn_sched_barrier(0);
      u32x4 r[4];
      const unsigned meta = rfl(qv);
      if (NR1 && (meta & 0x40000000u)) { r[0] = c0; r[1] = c1; r[2] = c2; r[3] = *(LDS u32x4*)(size_t)(ls + 3072 + lane * 16); }
      else dec_block(r, c0, c1, c2, ls + 3072, lane, [&]() { flag_out(false); });
      asm volatile("" : "+v"(r[0]), "+v"(r[1]), "+v"(r[2]), "+v"(r[3]) :: "memory");
      lds_st(lmeta + 64 + (n & (FNS - 1)) * 4, n + 1);
      const unsigned gq = meta & 511u, q = (meta >> 16) & 511u;
      if (pend != ~0u) flag_out(true);
      TRF();
      if (!(meta >> 30) && rfl(hv)) {
        const char* rp = ppk + PK_RAW + q * 4096 + lane * 16;
        asm volatile("global_load_dwordx4 %0, %4, off sc0 sc1\n\tglobal_load_dwordx4 %1, %4, off offset:1024 sc0 sc1\n\t"
                     "global_load_dwordx4 %2, %4, off offset:2048 sc0 sc1\n\tglobal_load_dwordx4 %3, %4, off offset:3072 sc0 sc1\n\ts_waitcnt vmcnt(0)"
                     : "=&v"(r[0]), "=&v"(r[1]), "=&v"(r[2]), "=&v"(r[3]) : "v"(rp) : "memory");
      }
      const char* dp = stg + gq * 4096;
      asm volatile("global_store_dwordx4 %0, %1, %5 sc0 sc1\n\tglobal_store_dwordx4 %0, %2, %5 offset:1024 sc0 sc1\n\t"
                   "global_store_dwordx4 %0, %3, %5 offset:2048 sc0 sc1\n\tglobal_store_dwordx4 %0, %4, %5 offset:3072 sc0 sc1\n\ts_nop 1"
                   :: "v"(lane * 16), "v"(r[0]), "v"(r[1]), "v"(r[2]), "v"(r[3]), "s"(dp) : "memory");
      pend = 1 ? (gq >> 2) * 128 + (gq & 3) * 4 : gq << 7;
    }
    flag_out(true);
    return;
  }

  constexpr unsigned R0 = 4 * ((NR0 / 4 + DL) / 2), R1 = NR0 - R0;
  static_assert((NR0 / 4 - DL) % 2 == 0 && R0 / 4 <= dt_of(0, 1) && dt_of(6, 0) <= 30, "");
  const unsigned en = (b - 32) + 224 * wid, eq = wid >= 4 ? 512 : en < 256 - R0 ? en + R0 : en + R0 + R1;
  u32x4 v[4];
  unsigned prev;
  if (eq < 512) {
    LANE_VARS
    const char* xs = p.x + (size_t)((eq >> 7) * 128 + (eq & 3) * 32) * KD + ((eq >> 2) & 31) * 128;
    asm volatile("global_load_dwordx4 %0, %5, %7\n\tglobal_load_dwordx4 %1, %6, %7\n\tglobal_load_dwordx4 %2, %5, %8\n\tglobal_load_dwordx4 %3, %6, %8\n\t"
                 "global_load_dword %4, %9, %10 sc1"
                 : "=&v"(v[0]), "=&v"(v[1]), "=&v"(v[2]), "=&v"(v[3]), "=&v"(prev)
                 : "v"(r8 * KD + sw0), "v"((r8 + 8) * KD + sw1), "s"(xs), "s"(xs + 16 * KD), "v"(0u), "s"((unsigned*)(myws + WS_CNT) + b * 8 + wid) : "memory");
  }
  if (threadIdx.x < 16) lds_st(L_CTL + threadIdx.x * 4, 0);
  asm volatile("s_waitcnt lgkmcnt(0)" ::: "memory");
  __builtin_amdgcn_s_barrier();

  if (wid == 7) {
    LANE_VARS
    const unsigned call = next_call(myws, b, wid, lane);
    const unsigned long long tp = t0 + DEADLINE, fb = (unsigned long long)(myws + WS_FLAG + src * 65536 + (rb << (1 ? 13 : 8 + 7)));
    waitvm<0>();
    asm volatile(
      "s_mov_b32 s80, 0\n\t"
      "s_mov_b32 s86, 0\n\t"
      "QL_%=:\n\t"
      "s_and_b32 s81, s80, -4\n\t"
      "s_lshl_b32 s90, s81, 5\n\t"
      "s_add_u32 s82, %[flo], s90\n\t"
      "s_addc_u32 s83, %[fhi], 0\n\t"
      "s_load_dwordx4 s[60:63], s[82:83], 0 glc\n\t"
      "s_load_dwordx4 s[64:67], s[82:83], 128 glc\n\t"
      "s_load_dwordx4 s[68:71], s[82:83], 256 glc\n\t"
      "s_load_dwordx4 s[72:75], s[82:83], 384 glc\n\t"
      "s_mov_b32 s90, 16\n\t"
      "s_waitcnt lgkmcnt(0)\n\t"
      "s_cmp_lt_u32 s75, %[call]\n\t"
      "s_cselect_b32 s90, 15, s90\n\t"
      "s_cmp_lt_u32 s74, %[call]\n\t"
      "s_cselect_b32 s90, 14, s90\n\t"
      "s_cmp_lt_u32 s73, %[call]\n\t"
      "s_cselect_b32 s90, 13, s90\n\t"
      "s_cmp_lt_u32 s72, %[call]\n\t"
      "s_cselect_b32 s90, 12, s90\n\t"
      "s_cmp_lt_u32 s71, %[call]\n\t"
      "s_cselect_b32 s90, 11, s90\n\t"
      "s_cmp_lt_u32 s70, %[call]\n\t"
      "s_cselect_b32 s90, 10, s90\n\t"
      "s_cmp_lt_u32 s69, %[call]\n\t"
      "s_cselect_b32 s90, 9, s90\n\t"
      "s_cmp_lt_u32 s68, %[call]\n\t"
      "s_cselect_b32 s90, 8, s90\n\t"
      "s_cmp_lt_u32 s67, %[call]\n\t"
      "s_cselect_b32 s90, 7, s90\n\t"
      "s_cmp_lt_u32 s66, %[call]\n\t"
      "s_cselect_b32 s90, 6, s90\n\t"
      "s_cmp_lt_u32 s65, %[call]\n\t"
      "s_cselect_b32 s90, 5, s90\n\t"
      "s_cmp_lt_u32 s64, %[call]\n\t"
      "s_cselect_b32 s90, 4, s90\n\t"
      "s_cmp_lt_u32 s63, %[call]\n\t"
      "s_cselect_b32 s90, 3, s90\n\t"
      "s_cmp_lt_u32 s62, %[call]\n\t"
      "s_cselect_b32 s90, 2, s90\n\t"
      "s_cmp_lt_u32 s61, %[call]\n\t"
      "s_cselect_b32 s90, 1, s90\n\t"
      "s_cmp_lt_u32 s60, %[call]\n\t"
      "s_cselect_b32 s90, 0, s90\n\t"
      "s_add_u32 s90, s90, s81\n\t"
      "s_min_u32 s90, s90, 256\n\t"
      "s_cmp_gt_u32 s90, s80\n\t"
      "s_cbranch_scc0 QN_%=\n\t"
      "s_mov_b32 s80, s90\n\t"
      "v_mov_b32 v58, s80\n\t"
      "ds_write_b32 %[vctl], v58 offset:32\n\t"
      "s_cmp_ge_u32 s80, 256\n\t"
      "s_cbranch_scc1 QE_%=\n\t"
      "QN_%=:\n\t"
      "s_sleep 2\n\t"
      "s_add_u32 s86, s86, 1\n\t"
      "s_and_b32 s90, s86, 255\n\t"
      "s_cbranch_scc1 QL_%=\n\t"
      "v_mov_b32 v58, 0\n\t"
      "global_load_dword v59, v58, %[tmo] sc1\n\t"
      "s_memrealtime s[90:91]\n\t"
      "s_waitcnt vmcnt(0) lgkmcnt(0)\n\t"
      "v_readfirstlane_b32 s92, v59\n\t"
      "s_cmp_eq_u32 s92, %[call]\n\t"
      "s_cbranch_scc1 QF_%=\n\t"
      "s_sub_u32 s90, %[tlo], s90\n\t"
      "s_subb_u32 s91, %[thi], s91\n\t"
      "s_cbranch_scc0 QL_%=\n\t"
      "QF_%=:\n\t"
      "v_mov_b32 v59, %[call]\n\t"
      "s_mov_b64 exec, 1\n\t"
      "global_store_dword v58, v59, %[tmo] sc1\n\t"
      "s_mov_b64 exec, -1\n\t"
      "v_mov_b32 v58, 1\n\t"
      "ds_write_b32 %[vctl], v58 offset:36\n\t"
      "v_mov_b32 v58, 0x100\n\t"
      "ds_write_b32 %[vctl], v58 offset:32\n\t"
      "QE_%=:\n\t"
      "s_waitcnt vmcnt(0) lgkmcnt(0)\n\t"
      :
      : [lane] "v"(lane), [fl] "s"(fb), [flo] "s"((unsigned)fb), [fhi] "s"((unsigned)(fb >> 32)), [call] "s"(call), [tmo] "s"(myws + WS_TMO),
        [vctl] "v"(L_CTL), [tlo] "s"((unsigned)tp), [thi] "s"((unsigned)(tp >> 32))
      : "memory", "scc", "vcc", "s60", "s61", "s62", "s63", "s64", "s65", "s66", "s67", "s68", "s69", "s70", "s71", "s72", "s73", "s74", "s75", "s76", "s77", "s78", "s79", "s80", "s81", "s82", "s83", "s84", "s85", "s86", "s87", "s88", "s89", "s90", "s91", "s92", "s93", "v55", "v56", "v57", "v58", "v59", "v60", "v61", "v62", "v63");
    return;
  }
  const unsigned long long tlim = t0 + 3 * DEADLINE;
  const __amdgpu_buffer_rsrc_t rt = __builtin_amdgcn_make_buffer_rsrc((void*)&int_tab, 0, (int)sizeof(IntTab), 0x00020000);
  if (wid == 4) {
    LANE_VARS
    const unsigned call = next_call(myws, b, wid, lane);
    const __amdgpu_buffer_rsrc_t rx = __builtin_amdgcn_make_buffer_rsrc(
        (void*)(myws + WS_STG + (size_t)(call & 1) * NR * SHARD + (size_t)(DBGBIT(8) ? ((p.rank + 1) & 7) : src) * SHARD), 0, (int)SHARD, 0x00020000);
    waitvm<0>();
    asm volatile(
      "v_mov_b32 v54, 0\n\t"
      "s_mov_b32 s80, 0\n\t"
      "s_mov_b32 s81, 0\n\t"
      "s_mov_b32 s83, %[so]\n\t"
      "s_mov_b32 s86, 0\n\t"
      "s_mov_b32 s85, 8\n\t"
      "s_mov_b32 s88, 1\n\t"
      "s_mov_b32 s82, 0x4000\n\t"
      "s_lshl2_add_u32 s76, %[dt], 4\n\t"
      "s_lshl_b32 s77, %[dt], 3\n\t"
      "s_sub_u32 s77, 252, s77\n\t"
      "XL_%=:\n\t"
      "s_cmp_lt_u32 s88, s85\n\t"
      "s_cbranch_scc0 XR0_%=\n\t"
      "s_cmp_lt_u32 s80, s81\n\t"
      "s_cbranch_scc0 XR0_%=\n\t"
      "XB0_%=:\n\t"
      "s_add_u32 m0, s82, 0\n\t"
      "s_waitcnt vmcnt(36)\n\t"
      "buffer_load_dwordx4 %[vx], %[rs], s83 offen offset:0 sc0 sc1 lds\n\t"
      "buffer_load_dwordx4 %[vx], %[rs], s83 offen offset:1024 sc0 sc1 lds\n\t"
      "buffer_load_dword v55, v54, %[rt], 0 offen\n\t"
      "buffer_load_dwordx4 %[vx], %[rs], s83 offen offset:2048 sc0 sc1 lds\n\t"
      "buffer_load_dwordx4 %[vx], %[rs], s83 offen offset:3072 sc0 sc1 lds\n\t"
      "buffer_load_dword v55, v54, %[rt], 0 offen\n\t"
      "s_add_u32 s80, s80, 1\n\t"
      "s_add_u32 s83, s83, 4096\n\t"
      "s_cmp_lt_u32 s80, s81\n\t"
      "s_cbranch_scc0 XR1_%=\n\t"
      "XB1_%=:\n\t"
      "s_add_u32 m0, s82, 4096\n\t"
      "s_waitcnt vmcnt(36)\n\t"
      "buffer_load_dwordx4 %[vx], %[rs], s83 offen offset:0 sc0 sc1 lds\n\t"
      "buffer_load_dwordx4 %[vx], %[rs], s83 offen offset:1024 sc0 sc1 lds\n\t"
      "buffer_load_dword v55, v54, %[rt], 0 offen\n\t"
      "buffer_load_dwordx4 %[vx], %[rs], s83 offen offset:2048 sc0 sc1 lds\n\t"
      "buffer_load_dwordx4 %[vx], %[rs], s83 offen offset:3072 sc0 sc1 lds\n\t"
      "buffer_load_dword v55, v54, %[rt], 0 offen\n\t"
      "s_add_u32 s80, s80, 1\n\t"
      "s_add_u32 s83, s83, 4096\n\t"
      "s_cmp_lt_u32 s80, s81\n\t"
      "s_cbranch_scc0 XR2_%=\n\t"
      "XB2_%=:\n\t"
      "s_add_u32 m0, s82, 8192\n\t"
      "s_waitcnt vmcnt(36)\n\t"
      "buffer_load_dwordx4 %[vx], %[rs], s83 offen offset:0 sc0 sc1 lds\n\t"
      "buffer_load_dwordx4 %[vx], %[rs], s83 offen offset:1024 sc0 sc1 lds\n\t"
      "buffer_load_dword v55, v54, %[rt], 0 offen\n\t"
      "buffer_load_dwordx4 %[vx], %[rs], s83 offen offset:2048 sc0 sc1 lds\n\t"
      "buffer_load_dwordx4 %[vx], %[rs], s83 offen offset:3072 sc0 sc1 lds\n\t"
      "buffer_load_dword v55, v54, %[rt], 0 offen\n\t"
      "s_add_u32 s80, s80, 1\n\t"
      "s_add_u32 s83, s83, 4096\n\t"
      "s_cmp_lt_u32 s80, s81\n\t"
      "s_cbranch_scc0 XR3_%=\n\t"
      "XB3_%=:\n\t"
      "s_add_u32 m0, s82, 12288\n\t"
      "s_waitcnt vmcnt(36)\n\t"
      "buffer_load_dwordx4 %[vx], %[rs], s83 offen offset:0 sc0 sc1 lds\n\t"
      "buffer_load_dwordx4 %[vx], %[rs], s83 offen offset:1024 sc0 sc1 lds\n\t"
      "buffer_load_dword v55, v54, %[rt], 0 offen\n\t"
      "buffer_load_dwordx4 %[vx], %[rs], s83 offen offset:2048 sc0 sc1 lds\n\t"
      "buffer_load_dwordx4 %[vx], %[rs], s83 offen offset:3072 sc0 sc1 lds\n\t"
      "buffer_load_dword v55, v54, %[rt], 0 offen\n\t"
      "s_add_u32 s80, s80, 1\n\t"
      "s_add_u32 s83, s83, 4096\n\t"
      "s_add_u32 m0, %[sctl], 0\n\t"
      "s_mov_b64 exec, 1\n\t"
      "buffer_load_dword v54, %[rt], s80 offen lds\n\t"
      "s_mov_b64 exec, -1\n\t"
      "s_sub_u32 s90, s80, s76\n\t"
      "s_cmp_lt_u32 s90, s77\n\t"
      "s_cselect_b32 s90, s90, 4\n\t"
      "s_bfe_u32 s90, s90, 0x10002\n\t"
      "s_add_u32 s88, s88, s90\n\t"
      "s_add_u32 s88, s88, 1\n\t"
      "s_and_b32 s82, s88, 7\n\t"
      "s_lshl_b32 s82, s82, 14\n\t"
      "s_cmp_lt_u32 s80, 256\n\t"
      "s_cbranch_scc1 XL_%=\n\t"
      "s_branch XE_%=\n\t"
      "XR0_%=:\n\t"
      "ds_read_b128 v[56:59], %[vctl] offset:16\n\t"
      "ds_read_b32 v61, %[vctl] offset:32\n\t"
      "s_waitcnt lgkmcnt(0)\n\t"
      "v_readfirstlane_b32 s81, v61\n\t"
      "v_min_u32 v56, v56, v57\n\t"
      "v_min3_u32 v56, v56, v58, v59\n\t"
      "s_nop 0\n\t"
      "v_readfirstlane_b32 s85, v56\n\t"
      "s_add_u32 s85, s85, 8\n\t"
      "s_cmp_lt_u32 s88, s85\n\t"
      "s_cselect_b32 s90, s81, 0\n\t"
      "s_cmp_lt_u32 s80, s90\n\t"
      "s_cbranch_scc1 XB0_%=\n\t"
      "s_sleep 3\n\t"
      "s_add_u32 s86, s86, 1\n\t"
      "s_and_b32 s90, s86, 1023\n\t"
      "s_cbranch_scc1 XR0_%=\n\t"
      "s_memrealtime s[90:91]\n\t"
      "s_waitcnt lgkmcnt(0)\n\t"
      "s_sub_u32 s90, %[tlo], s90\n\t"
      "s_subb_u32 s91, %[thi], s91\n\t"
      "s_cbranch_scc0 XR0_%=\n\t"
      "v_mov_b32 v60, 1\n\t"
      "ds_write_b32 %[vctl], v60 offset:36\n\t"
      "s_mov_b32 s81, 256\n\t"
      "s_branch XB0_%=\n\t"
      "XR1_%=:\n\t"
      "ds_read_b32 v61, %[vctl] offset:32\n\t"
      "s_waitcnt lgkmcnt(0)\n\t"
      "v_readfirstlane_b32 s81, v61\n\t"
      "s_cmp_lt_u32 s80, s81\n\t"
      "s_cbranch_scc1 XB1_%=\n\t"
      "s_sleep 3\n\t"
      "s_add_u32 s86, s86, 1\n\t"
      "s_and_b32 s90, s86, 1023\n\t"
      "s_cbranch_scc1 XR1_%=\n\t"
      "s_memrealtime s[90:91]\n\t"
      "s_waitcnt lgkmcnt(0)\n\t"
      "s_sub_u32 s90, %[tlo], s90\n\t"
      "s_subb_u32 s91, %[thi], s91\n\t"
      "s_cbranch_scc0 XR1_%=\n\t"
      "v_mov_b32 v60, 1\n\t"
      "ds_write_b32 %[vctl], v60 offset:36\n\t"
      "s_mov_b32 s81, 256\n\t"
      "s_branch XB1_%=\n\t"
      "XR2_%=:\n\t"
      "ds_read_b32 v61, %[vctl] offset:32\n\t"
      "s_waitcnt lgkmcnt(0)\n\t"
      "v_readfirstlane_b32 s81, v61\n\t"
      "s_cmp_lt_u32 s80, s81\n\t"
      "s_cbranch_scc1 XB2_%=\n\t"
      "s_sleep 3\n\t"
      "s_add_u32 s86, s86, 1\n\t"
      "s_and_b32 s90, s86, 1023\n\t"
      "s_cbranch_scc1 XR2_%=\n\t"
      "s_memrealtime s[90:91]\n\t"
      "s_waitcnt lgkmcnt(0)\n\t"
      "s_sub_u32 s90, %[tlo], s90\n\t"
      "s_subb_u32 s91, %[thi], s91\n\t"
      "s_cbranch_scc0 XR2_%=\n\t"
      "v_mov_b32 v60, 1\n\t"
      "ds_write_b32 %[vctl], v60 offset:36\n\t"
      "s_mov_b32 s81, 256\n\t"
      "s_branch XB2_%=\n\t"
      "XR3_%=:\n\t"
      "ds_read_b32 v61, %[vctl] offset:32\n\t"
      "s_waitcnt lgkmcnt(0)\n\t"
      "v_readfirstlane_b32 s81, v61\n\t"
      "s_cmp_lt_u32 s80, s81\n\t"
      "s_cbranch_scc1 XB3_%=\n\t"
      "s_sleep 3\n\t"
      "s_add_u32 s86, s86, 1\n\t"
      "s_and_b32 s90, s86, 1023\n\t"
      "s_cbranch_scc1 XR3_%=\n\t"
      "s_memrealtime s[90:91]\n\t"
      "s_waitcnt lgkmcnt(0)\n\t"
      "s_sub_u32 s90, %[tlo], s90\n\t"
      "s_subb_u32 s91, %[thi], s91\n\t"
      "s_cbranch_scc0 XR3_%=\n\t"
      "v_mov_b32 v60, 1\n\t"
      "ds_write_b32 %[vctl], v60 offset:36\n\t"
      "s_mov_b32 s81, 256\n\t"
      "s_branch XB3_%=\n\t"
      "XE_%=:\n\t"
      "s_waitcnt vmcnt(0) lgkmcnt(0)\n\t"
      :
      : [vx] "v"(lane * 16), [rs] "s"(rx), [so] "s"(DBGBIT(8) ? 0 : rb << 20), [dt] "s"(dt), [vctl] "v"(L_CTL), [sctl] "s"(L_CTL), [rt] "s"(rt), [tlo] "s"((unsigned)tlim), [thi] "s"((unsigned)(tlim >> 32))
      : "memory", "scc", "s76", "s77", "s78", "s79", "s80", "s81", "s82", "s83", "s84", "s85", "s86", "s87", "s88", "s89", "s90", "s91", "s92", "s93", "v54", "v55", "v56", "v57", "v58", "v59", "v60", "v61");
    return;
  }
  if (wid == 5) {
    LANE_VARS
    const __amdgpu_buffer_rsrc_t rw = __builtin_amdgcn_make_buffer_rsrc((void*)p.w, 0, KD * NL, 0x00020000);
    const __amdgpu_buffer_rsrc_t re = __builtin_amdgcn_make_buffer_rsrc((void*)p.x, 0, (int)SHARD, 0x00020000);
    unsigned vw[4], ve[4];
#pragma unroll
    for (int i = 0; i < 4; ++i) {
      vw[i] = (8 * i + r8) * NL + ((i & 2) ? sw1 : sw0) - i * 1024;
      ve[i] = (8 * i + r8) * KD + ((i & 1) ? sw1 : sw0) - i * 1024;
    }
    asm volatile(
      "v_mov_b32 v54, 0\n\t"
      "s_mov_b32 s80, 0\n\t"
      "s_mov_b32 s81, 0\n\t"
      "s_mov_b32 s82, 0\n\t"
      "s_mov_b32 s83, %[so]\n\t"
      "s_mov_b32 s86, 0\n\t"
      "s_mov_b32 s84, 131072\n\t"
      "s_mov_b32 s85, %[se0]\n\t"
      "s_mov_b32 s87, %[se1]\n\t"
      "s_mov_b32 s88, 0\n\t"
      "s_mov_b32 s89, -1\n\t"
      "s_mov_b32 s77, -1\n\t"
      "s_mov_b32 s78, -1\n\t"
      "s_lshl_b32 s76, %[dt], 2\n\t"
      "s_add_u32 s93, s76, 128\n\t"
      "s_cmp_eq_u32 %[ext], 0\n\t"
      "s_cbranch_scc0 WLB_%=\n\t"
      "WLA_%=:\n\t"
      "s_cmp_gt_i32 s81, s89\n\t"
      "s_cbranch_scc0 WRA_%=\n\t"
      "WBA_%=:\n\t"
      "s_add_u32 m0, s82, 0\n\t"
      "s_waitcnt vmcnt(24)\n\t"
      "buffer_load_dwordx4 %[vw0], %[rw], s83 offen offset:0 lds\n\t"
      "buffer_load_dwordx4 %[vw1], %[rw], s83 offen offset:1024 lds\n\t"
      "buffer_load_dword v55, v54, %[rt], 0 offen\n\t"
      "buffer_load_dwordx4 %[vw2], %[rw], s83 offen offset:2048 lds\n\t"
      "buffer_load_dwordx4 %[vw3], %[rw], s83 offen offset:3072 lds\n\t"
      "buffer_load_dword v55, v54, %[rt], 0 offen\n\t"
      "s_add_u32 s83, s83, 65536\n\t"
      "s_add_u32 m0, s82, 4096\n\t"
      "s_waitcnt vmcnt(24)\n\t"
      "buffer_load_dwordx4 %[vw0], %[rw], s83 offen offset:0 lds\n\t"
      "buffer_load_dwordx4 %[vw1], %[rw], s83 offen offset:1024 lds\n\t"
      "buffer_load_dword v55, v54, %[rt], 0 offen\n\t"
      "buffer_load_dwordx4 %[vw2], %[rw], s83 offen offset:2048 lds\n\t"
      "buffer_load_dwordx4 %[vw3], %[rw], s83 offen offset:3072 lds\n\t"
      "buffer_load_dword v55, v54, %[rt], 0 offen\n\t"
      "s_add_u32 s83, s83, 65536\n\t"
      "s_add_u32 m0, s82, 8192\n\t"
      "s_waitcnt vmcnt(24)\n\t"
      "buffer_load_dwordx4 %[vw0], %[rw], s83 offen offset:0 lds\n\t"
      "buffer_load_dwordx4 %[vw1], %[rw], s83 offen offset:1024 lds\n\t"
      "buffer_load_dword v55, v54, %[rt], 0 offen\n\t"
      "buffer_load_dwordx4 %[vw2], %[rw], s83 offen offset:2048 lds\n\t"
      "buffer_load_dwordx4 %[vw3], %[rw], s83 offen offset:3072 lds\n\t"
      "buffer_load_dword v55, v54, %[rt], 0 offen\n\t"
      "s_add_u32 s83, s83, 65536\n\t"
      "s_add_u32 m0, s82, 12288\n\t"
      "s_waitcnt vmcnt(24)\n\t"
      "buffer_load_dwordx4 %[vw0], %[rw], s83 offen offset:0 lds\n\t"
      "buffer_load_dwordx4 %[vw1], %[rw], s83 offen offset:1024 lds\n\t"
      "buffer_load_dword v55, v54, %[rt], 0 offen\n\t"
      "buffer_load_dwordx4 %[vw2], %[rw], s83 offen offset:2048 lds\n\t"
      "buffer_load_dwordx4 %[vw3], %[rw], s83 offen offset:3072 lds\n\t"
      "buffer_load_dword v55, v54, %[rt], 0 offen\n\t"
      "s_add_u32 s83, s83, 65536\n\t"
      "s_add_u32 m0, s84, 0\n\t"
      "s_waitcnt vmcnt(24)\n\t"
      "buffer_load_dwordx4 %[ve0], %[re], s85 offen offset:0 lds\n\t"
      "buffer_load_dwordx4 %[ve1], %[re], s85 offen offset:1024 lds\n\t"
      "buffer_load_dword v55, v54, %[rt], 0 offen\n\t"
      "buffer_load_dwordx4 %[ve2], %[re], s85 offen offset:2048 lds\n\t"
      "buffer_load_dwordx4 %[ve3], %[re], s85 offen offset:3072 lds\n\t"
      "buffer_load_dword v55, v54, %[rt], 0 offen\n\t"
      "s_add_u32 s85, s85, 128\n\t"
      "s_add_u32 s87, s87, 128\n\t"
      "s_add_u32 s84, s84, 0x2000\n\t"
      "s_cmp_eq_u32 s84, 155648\n\t"
      "s_cselect_b32 s84, 131072, s84\n\t"
      "s_mov_b32 s79, s78\n\t"
      "s_mov_b32 s78, s77\n\t"
      "s_mov_b32 s77, s88\n\t"
      "s_cmp_ge_u32 s80, s76\n\t"
      "s_addc_u32 s88, s88, 2\n\t"
      "s_sub_i32 s89, s88, 8\n\t"
      "s_max_i32 s89, s89, s79\n\t"
      "s_add_u32 s80, s80, 4\n\t"
      "s_add_u32 m0, %[sctl], 4\n\t"
      "s_mov_b64 exec, 1\n\t"
      "buffer_load_dword v54, %[rt], s80 offen lds\n\t"
      "s_mov_b64 exec, -1\n\t"
      "s_and_b32 s82, s88, 7\n\t"
      "s_lshl_b32 s82, s82, 14\n\t"
      "s_cmp_lt_u32 s80, 128\n\t"
      "s_cbranch_scc1 WLA_%=\n\t"
      "s_branch WT_%=\n\t"
      "WRA_%=:\n\t"
      "ds_read_b128 v[56:59], %[vctl] offset:16\n\t"
      "s_waitcnt lgkmcnt(0)\n\t"
      "v_min_u32 v56, v56, v57\n\t"
      "v_min3_u32 v56, v56, v58, v59\n\t"
      "s_nop 0\n\t"
      "v_readfirstlane_b32 s81, v56\n\t"
      "s_cmp_gt_i32 s81, s89\n\t"
      "s_cbranch_scc1 WBA_%=\n\t"
      "s_sleep 3\n\t"
      "s_add_u32 s86, s86, 1\n\t"
      "s_and_b32 s90, s86, 1023\n\t"
      "s_cbranch_scc1 WRA_%=\n\t"
      "s_memrealtime s[90:91]\n\t"
      "s_waitcnt lgkmcnt(0)\n\t"
      "s_sub_u32 s90, %[tlo], s90\n\t"
      "s_subb_u32 s91, %[thi], s91\n\t"
      "s_cbranch_scc0 WRA_%=\n\t"
      "v_mov_b32 v60, 1\n\t"
      "ds_write_b32 %[vctl], v60 offset:36\n\t"
      "s_mov_b32 s81, 256\n\t"
      "s_branch WBA_%=\n\t"
      "WLB_%=:\n\t"
      "s_cmp_gt_i32 s81, s89\n\t"
      "s_cbranch_scc0 WRB_%=\n\t"
      "WBB_%=:\n\t"
      "s_add_u32 m0, s82, 0\n\t"
      "s_waitcnt vmcnt(24)\n\t"
      "buffer_load_dwordx4 %[vw0], %[rw], s83 offen offset:0 lds\n\t"
      "buffer_load_dwordx4 %[vw1], %[rw], s83 offen offset:1024 lds\n\t"
      "buffer_load_dword v55, v54, %[rt], 0 offen\n\t"
      "buffer_load_dwordx4 %[vw2], %[rw], s83 offen offset:2048 lds\n\t"
      "buffer_load_dwordx4 %[vw3], %[rw], s83 offen offset:3072 lds\n\t"
      "buffer_load_dword v55, v54, %[rt], 0 offen\n\t"
      "s_add_u32 s83, s83, 65536\n\t"
      "s_add_u32 m0, s82, 4096\n\t"
      "s_waitcnt vmcnt(24)\n\t"
      "buffer_load_dwordx4 %[vw0], %[rw], s83 offen offset:0 lds\n\t"
      "buffer_load_dwordx4 %[vw1], %[rw], s83 offen offset:1024 lds\n\t"
      "buffer_load_dword v55, v54, %[rt], 0 offen\n\t"
      "buffer_load_dwordx4 %[vw2], %[rw], s83 offen offset:2048 lds\n\t"
      "buffer_load_dwordx4 %[vw3], %[rw], s83 offen offset:3072 lds\n\t"
      "buffer_load_dword v55, v54, %[rt], 0 offen\n\t"
      "s_add_u32 s83, s83, 65536\n\t"
      "s_add_u32 m0, s82, 8192\n\t"
      "s_waitcnt vmcnt(24)\n\t"
      "buffer_load_dwordx4 %[vw0], %[rw], s83 offen offset:0 lds\n\t"
      "buffer_load_dwordx4 %[vw1], %[rw], s83 offen offset:1024 lds\n\t"
      "buffer_load_dword v55, v54, %[rt], 0 offen\n\t"
      "buffer_load_dwordx4 %[vw2], %[rw], s83 offen offset:2048 lds\n\t"
      "buffer_load_dwordx4 %[vw3], %[rw], s83 offen offset:3072 lds\n\t"
      "buffer_load_dword v55, v54, %[rt], 0 offen\n\t"
      "s_add_u32 s83, s83, 65536\n\t"
      "s_add_u32 m0, s82, 12288\n\t"
      "s_waitcnt vmcnt(24)\n\t"
      "buffer_load_dwordx4 %[vw0], %[rw], s83 offen offset:0 lds\n\t"
      "buffer_load_dwordx4 %[vw1], %[rw], s83 offen offset:1024 lds\n\t"
      "buffer_load_dword v55, v54, %[rt], 0 offen\n\t"
      "buffer_load_dwordx4 %[vw2], %[rw], s83 offen offset:2048 lds\n\t"
      "buffer_load_dwordx4 %[vw3], %[rw], s83 offen offset:3072 lds\n\t"
      "buffer_load_dword v55, v54, %[rt], 0 offen\n\t"
      "s_add_u32 s83, s83, 65536\n\t"
      "s_add_u32 m0, s84, 0\n\t"
      "s_waitcnt vmcnt(24)\n\t"
      "buffer_load_dwordx4 %[ve0], %[re], s85 offen offset:0 lds\n\t"
      "buffer_load_dwordx4 %[ve1], %[re], s85 offen offset:1024 lds\n\t"
      "buffer_load_dword v55, v54, %[rt], 0 offen\n\t"
      "buffer_load_dwordx4 %[ve2], %[re], s85 offen offset:2048 lds\n\t"
      "buffer_load_dwordx4 %[ve3], %[re], s85 offen offset:3072 lds\n\t"
      "buffer_load_dword v55, v54, %[rt], 0 offen\n\t"
      "s_add_u32 m0, s84, 4096\n\t"
      "s_waitcnt vmcnt(24)\n\t"
      "buffer_load_dwordx4 %[ve0], %[re], s87 offen offset:0 lds\n\t"
      "buffer_load_dwordx4 %[ve1], %[re], s87 offen offset:1024 lds\n\t"
      "buffer_load_dword v55, v54, %[rt], 0 offen\n\t"
      "buffer_load_dwordx4 %[ve2], %[re], s87 offen offset:2048 lds\n\t"
      "buffer_load_dwordx4 %[ve3], %[re], s87 offen offset:3072 lds\n\t"
      "buffer_load_dword v55, v54, %[rt], 0 offen\n\t"
      "s_add_u32 s85, s85, 128\n\t"
      "s_add_u32 s87, s87, 128\n\t"
      "s_add_u32 s84, s84, 0x2000\n\t"
      "s_cmp_eq_u32 s84, 155648\n\t"
      "s_cselect_b32 s84, 131072, s84\n\t"
      "s_mov_b32 s79, s78\n\t"
      "s_mov_b32 s78, s77\n\t"
      "s_mov_b32 s77, s88\n\t"
      "s_cmp_ge_u32 s80, s76\n\t"
      "s_addc_u32 s88, s88, 2\n\t"
      "s_sub_i32 s89, s88, 8\n\t"
      "s_max_i32 s89, s89, s79\n\t"
      "s_add_u32 s80, s80, 4\n\t"
      "s_add_u32 m0, %[sctl], 4\n\t"
      "s_mov_b64 exec, 1\n\t"
      "buffer_load_dword v54, %[rt], s80 offen lds\n\t"
      "s_mov_b64 exec, -1\n\t"
      "s_and_b32 s82, s88, 7\n\t"
      "s_lshl_b32 s82, s82, 14\n\t"
      "s_cmp_lt_u32 s80, 128\n\t"
      "s_cbranch_scc1 WLB_%=\n\t"
      "s_branch WT_%=\n\t"
      "WRB_%=:\n\t"
      "ds_read_b128 v[56:59], %[vctl] offset:16\n\t"
      "s_waitcnt lgkmcnt(0)\n\t"
      "v_min_u32 v56, v56, v57\n\t"
      "v_min3_u32 v56, v56, v58, v59\n\t"
      "s_nop 0\n\t"
      "v_readfirstlane_b32 s81, v56\n\t"
      "s_cmp_gt_i32 s81, s89\n\t"
      "s_cbranch_scc1 WBB_%=\n\t"
      "s_sleep 3\n\t"
      "s_add_u32 s86, s86, 1\n\t"
      "s_and_b32 s90, s86, 1023\n\t"
      "s_cbranch_scc1 WRB_%=\n\t"
      "s_memrealtime s[90:91]\n\t"
      "s_waitcnt lgkmcnt(0)\n\t"
      "s_sub_u32 s90, %[tlo], s90\n\t"
      "s_subb_u32 s91, %[thi], s91\n\t"
      "s_cbranch_scc0 WRB_%=\n\t"
      "v_mov_b32 v60, 1\n\t"
      "ds_write_b32 %[vctl], v60 offset:36\n\t"
      "s_mov_b32 s81, 256\n\t"
      "s_branch WBB_%=\n\t"
      "WT_%=:\n\t"
      "s_mov_b32 s83, %[so]\n\t"
      "s_sub_i32 s89, s88, 8\n\t"
      "WLC_%=:\n\t"
      "s_cmp_gt_i32 s81, s89\n\t"
      "s_cbranch_scc0 WRC_%=\n\t"
      "WBC_%=:\n\t"
      "s_add_u32 m0, s82, 0\n\t"
      "s_waitcnt vmcnt(30)\n\t"
      "buffer_load_dwordx4 %[vw0], %[rw], s83 offen offset:0 lds\n\t"
      "buffer_load_dwordx4 %[vw1], %[rw], s83 offen offset:1024 lds\n\t"
      "buffer_load_dword v55, v54, %[rt], 0 offen\n\t"
      "buffer_load_dwordx4 %[vw2], %[rw], s83 offen offset:2048 lds\n\t"
      "buffer_load_dwordx4 %[vw3], %[rw], s83 offen offset:3072 lds\n\t"
      "buffer_load_dword v55, v54, %[rt], 0 offen\n\t"
      "s_add_u32 s83, s83, 65536\n\t"
      "s_add_u32 m0, s82, 4096\n\t"
      "s_waitcnt vmcnt(30)\n\t"
      "buffer_load_dwordx4 %[vw0], %[rw], s83 offen offset:0 lds\n\t"
      "buffer_load_dwordx4 %[vw1], %[rw], s83 offen offset:1024 lds\n\t"
      "buffer_load_dword v55, v54, %[rt], 0 offen\n\t"
      "buffer_load_dwordx4 %[vw2], %[rw], s83 offen offset:2048 lds\n\t"
      "buffer_load_dwordx4 %[vw3], %[rw], s83 offen offset:3072 lds\n\t"
      "buffer_load_dword v55, v54, %[rt], 0 offen\n\t"
      "s_add_u32 s83, s83, 65536\n\t"
      "s_add_u32 m0, s82, 8192\n\t"
      "s_waitcnt vmcnt(30)\n\t"
      "buffer_load_dwordx4 %[vw0], %[rw], s83 offen offset:0 lds\n\t"
      "buffer_load_dwordx4 %[vw1], %[rw], s83 offen offset:1024 lds\n\t"
      "buffer_load_dword v55, v54, %[rt], 0 offen\n\t"
      "buffer_load_dwordx4 %[vw2], %[rw], s83 offen offset:2048 lds\n\t"
      "buffer_load_dwordx4 %[vw3], %[rw], s83 offen offset:3072 lds\n\t"
      "buffer_load_dword v55, v54, %[rt], 0 offen\n\t"
      "s_add_u32 s83, s83, 65536\n\t"
      "s_add_u32 m0, s82, 12288\n\t"
      "s_waitcnt vmcnt(30)\n\t"
      "buffer_load_dwordx4 %[vw0], %[rw], s83 offen offset:0 lds\n\t"
      "buffer_load_dwordx4 %[vw1], %[rw], s83 offen offset:1024 lds\n\t"
      "buffer_load_dword v55, v54, %[rt], 0 offen\n\t"
      "buffer_load_dwordx4 %[vw2], %[rw], s83 offen offset:2048 lds\n\t"
      "buffer_load_dwordx4 %[vw3], %[rw], s83 offen offset:3072 lds\n\t"
      "buffer_load_dword v55, v54, %[rt], 0 offen\n\t"
      "s_add_u32 s83, s83, 65536\n\t"
      "s_add_u32 s88, s88, 2\n\t"
      "s_sub_i32 s89, s88, 8\n\t"
      "s_add_u32 s80, s80, 4\n\t"
      "s_add_u32 m0, %[sctl], 4\n\t"
      "s_mov_b64 exec, 1\n\t"
      "buffer_load_dword v54, %[rt], s80 offen lds\n\t"
      "s_mov_b64 exec, -1\n\t"
      "s_and_b32 s82, s88, 7\n\t"
      "s_lshl_b32 s82, s82, 14\n\t"
      "s_cmp_lt_u32 s80, s93\n\t"
      "s_cbranch_scc1 WLC_%=\n\t"
      "s_branch WE_%=\n\t"
      "WRC_%=:\n\t"
      "ds_read_b128 v[56:59], %[vctl] offset:16\n\t"
      "s_waitcnt lgkmcnt(0)\n\t"
      "v_min_u32 v56, v56, v57\n\t"
      "v_min3_u32 v56, v56, v58, v59\n\t"
      "s_nop 0\n\t"
      "v_readfirstlane_b32 s81, v56\n\t"
      "s_cmp_gt_i32 s81, s89\n\t"
      "s_cbranch_scc1 WBC_%=\n\t"
      "s_sleep 3\n\t"
      "s_add_u32 s86, s86, 1\n\t"
      "s_and_b32 s90, s86, 1023\n\t"
      "s_cbranch_scc1 WRC_%=\n\t"
      "s_memrealtime s[90:91]\n\t"
      "s_waitcnt lgkmcnt(0)\n\t"
      "s_sub_u32 s90, %[tlo], s90\n\t"
      "s_subb_u32 s91, %[thi], s91\n\t"
      "s_cbranch_scc0 WRC_%=\n\t"
      "v_mov_b32 v60, 1\n\t"
      "ds_write_b32 %[vctl], v60 offset:36\n\t"
      "s_mov_b32 s81, 256\n\t"
      "s_branch WBC_%=\n\t"
      "WE_%=:\n\t"
      "s_waitcnt vmcnt(0) lgkmcnt(0)\n\t"
      :
      : [vw0] "v"(vw[0]), [vw1] "v"(vw[1]), [vw2] "v"(vw[2]), [vw3] "v"(vw[3]), [ve0] "v"(ve[0]), [ve1] "v"(ve[1]), [ve2] "v"(ve[2]), [ve3] "v"(ve[3]),
        [rw] "s"(rw), [re] "s"(re), [so] "s"(DBGBIT(8) ? 0 : cb * 128), [se0] "s"(DBGBIT(8) ? 0 : eg0 * 32 * KD), [se1] "s"(eg1 * 32 * KD), [ext] "s"(rfl(ext1 ? 1u : 0u)),
        [dt] "s"(dt), [vctl] "v"(L_CTL), [sctl] "s"(L_CTL), [rt] "s"(rt), [tlo] "s"((unsigned)tlim), [thi] "s"((unsigned)(tlim >> 32))
      : "memory", "scc", "s76", "s77", "s78", "s79", "s80", "s81", "s82", "s83", "s84", "s85", "s86", "s87", "s88", "s89", "s90", "s91", "s92", "s93", "v54", "v55", "v56", "v57", "v58", "v59", "v60", "v61");
    return;
  }
  if (wid == 6 && 1) {
    LANE_VARS
    const __amdgpu_buffer_rsrc_t rw = __builtin_amdgcn_make_buffer_rsrc((void*)p.w, 0, KD * NL, 0x00020000);
    constexpr int SH = 1 == 1 ? 14 : 7, NI = (KD / SH + 64) / 64;
    const unsigned r = 1 == 1 ? ((jj >> 2) - 1) * 2 + (xcd >> 2) : (jj >> 2) - 1;
    unsigned junk = 0, v[NI], spins = 0;
    while ((rfl(lds_ld(L_ENC)) & 255u) < 2) if (late(spins, t0, DEADLINE)) break;
    __builtin_amdgcn_s_sleep(48);
#pragma unroll
    for (int i = 0; i < NI; ++i) {
      v[i] = __builtin_amdgcn_raw_buffer_load_b32(rw, (r + SH * (i * 64 + lane)) * NL + cb * 128, 0, 0);
      __builtin_amdgcn_sched_barrier(0);
      __builtin_amdgcn_s_sleep(32);
      __builtin_amdgcn_sched_barrier(0);
    }
#pragma unroll
    for (int i = 0; i < NI; ++i) junk ^= v[i];
    asm volatile("" :: "v"(junk));
  }
  if (wid == 6 && 28) {
    LANE_VARS
    unsigned spins = 0;
    while (rfl(lds_ld(L_FULL + 4)) < 28) { __builtin_amdgcn_s_sleep(8); if (late(spins, t0, DEADLINE)) break; }
    unsigned long long pc;
    unsigned len;
    asm volatile("s_getpc_b64 s[90:91]\n\tAG_W0:\n\ts_add_u32 s90, s90, AG_C0-AG_W0\n\ts_addc_u32 s91, s91, (AG_C0-AG_W0)>>32\n\t"
                 "s_mov_b64 %0, s[90:91]\n\ts_mov_b32 %1, AG_C1-AG_C0" : "=s"(pc), "=s"(len) :: "scc", "s90", "s91");
    unsigned junk = 0;
    if ((jj & 3) == 0)
      for (unsigned o = lane * 128; o < len; o += 8192) junk ^= *(const volatile unsigned*)(pc + o);
    asm volatile("" :: "v"(junk));
  }
  if (wid >= 4) return;
  LANE_VARS
  {
    if (eq < 512) {
      const unsigned q = eq;
      unsigned* const cnt = (unsigned*)(myws + WS_CNT) + b * 8 + wid;
      const unsigned le = L_SCR + wid * 640;
      enc_init(le, lane);
      unsigned long long pw = (unsigned long long)p.ws[1];
#pragma unroll
      for (int i = 2; i < NR; ++i) { unsigned long long z = (unsigned long long)p.ws[i]; asm volatile("" : "+s"(z)); if (lane == i - 1) pw = z; }
      asm volatile("s_waitcnt vmcnt(0)" : "+v"(prev), "+v"(v[0]), "+v"(v[1]), "+v"(v[2]), "+v"(v[3]), "+v"(pw) :: "memory");
      const unsigned call = rfl(prev + 1);
      char* const pk = myws + WS_PK + (size_t)(call & 1) * (4u << 20);
      char* const pn = pk + q * 3072 + lane * 16;
      { u32x4 N[2]; enc_nibbles(v, N); st16_sys(pn + 1024, N[0]); st16_sys(pn + 2048, N[1]); }
      const unsigned kq = (q >> 2) & 31;
      bool raw = NR1 && (q & 128) && kq + (NR1 / 4 + DL) / 2 + 1 >= dt_of(0, 1) && kq < dt_of(6, 0);
      if (raw) {
        char* d = pk + PK_RAW + q * 4096 + lane * 16;
#pragma unroll
        for (int i = 0; i < 4; ++i) st16_sys(d + i * 1024, v[i]);
      }
      const unsigned mode = enc_block(v, le, lane, [&](const u32x4& P) { st16_sys(pn, P); });
      asm volatile("s_waitcnt lgkmcnt(0)" ::: "memory");
      const u32x4 ev = *(volatile LDS u32x4*)(size_t)(le + (lane & 31) * 16);
      if (lane < 32) st16_sys(pk + PK_E + q * 512 + lane * 16, ev);
      if (mode == 2 && !raw) {
        char* d = pk + PK_RAW + q * 4096 + lane * 16;
#pragma unroll
        for (int i = 0; i < 4; ++i) st16_sys(d + i * 1024, v[i]);
      }
      waitvm<0>();
      const unsigned nenc = (512 - NR0 - (b - 32) + 223) / 224;
      static_assert(NR0 >= 48, "58 blocks per shard at most");
      const unsigned mine = 1u + ((mode ? 1u : 0u) << (8 + wid));
      unsigned a = 0;
      if (lane == 0) a = __hip_atomic_fetch_add((LDS unsigned*)(size_t)L_ENC, mine, __ATOMIC_RELAXED, __HIP_MEMORY_SCOPE_WORKGROUP);
      a = rfl(a) + mine;
      if ((a & 255u) == nenc) {
        const unsigned gi = jj - 4;
        const unsigned long long add = (1ull << 58) | ((unsigned long long)(a >> 8 & 1) << gi) | ((unsigned long long)(a >> 9 & 1) << (gi + 28)) |
                                       ((unsigned long long)(a >> 10 & 1) << ((gi + 56) & 63));
        unsigned long long* ec = (unsigned long long*)(myws + WS_ECNT + ((call & 1) * 8 + xcd) * 128);
        unsigned long long o = 0;
        if (lane == 0) o = __hip_atomic_fetch_add(ec, add, __ATOMIC_RELAXED, __HIP_MEMORY_SCOPE_AGENT) + add;
        const unsigned lo = rfl((unsigned)o), hi = rfl((unsigned)(o >> 32));
        if ((hi >> 26) == 28) {
          if (lane < NR - 1) st16_sys((char*)pw + WS_ENCF + ((call & 1) * 8 + p.rank) * 128 + xcd * 16, (u32x4){lo, call, hi, call});
          if (lane == 0) __hip_atomic_store(ec, 0ull, __ATOMIC_RELAXED, __HIP_MEMORY_SCOPE_AGENT);
        }
      }
      if (lane == 0) __hip_atomic_store(cnt, call, __ATOMIC_RELAXED, __HIP_MEMORY_SCOPE_AGENT);
    }
  }

  const int wm = wid >> 1, wn = wid & 1;
  const int c16 = lane & 15, g = lane >> 4;
  const unsigned ixa = (wm * 64 + c16) * 128 + ((g ^ (c16 >> 1)) << 4);
  const unsigned iwa = (16 * g + (c16 >> 1)) * 128 + (((wn * 4) ^ ((c16 >> 2) | ((g & 1) << 2))) << 4) + 8 * (c16 & 1);
  const float scale = p.sx[0] * p.sw[0];
  const unsigned sscale = rfl(__builtin_bit_cast(unsigned, scale));
  static_assert(L_CTL % 64 == 0, "");
  const __amdgpu_buffer_rsrc_t ro = __builtin_amdgcn_make_buffer_rsrc((void*)p.out, 0, NR * MS * NL * 4, 0x00020000);
  const unsigned vout = r8 * NL * 4 + (lane & 7) * 16;
  const unsigned vsw0 = L_ST + wid * 2048 + c16 * 128 + ((g ^ (c16 & 7)) << 4);
  const unsigned vsr = L_ST + wid * 2048 + r8 * 128 + (((lane & 7) ^ r8) << 4);
  const unsigned sout = rfl(((src * MS + rb * 256 + wm * 64) * NL + cb * 128 + wn * 64) * 4);
  const unsigned soute = rfl(((p.rank * MS + eg0 * 32 + wm * 16) * NL + cb * 128 + wn * 64) * 4);
  const unsigned soute2 = rfl(((p.rank * MS + eg1 * 32 + wm * 16) * NL + cb * 128 + wn * 64) * 4);
  const unsigned sext = rfl(ext1 ? 1u : 0u);
  asm volatile(
      "AG_C0:\n\t"
      "s_setprio 2\n\t"
      "s_mov_b32 s91, 0\n\t"
      "s_mov_b32 s80, 0\n\t"
      "s_mov_b32 s81, 0\n\t"
      "s_mov_b32 s82, 0\n\t"
      "s_mov_b32 s85, 0\n\t"
      "s_mov_b32 s88, 0\n\t"
      "s_add_u32 s94, %[dt], 32\n\t"
      "s_sub_u32 s95, 93, %[dt]\n\t"
      "s_mov_b32 s86, 0x4000\n\t"
      "s_add_u32 s87, %[dt], 1\n\t"
      "s_and_b32 s87, s87, 3\n\t"
      "s_lshl_b32 s87, s87, 15\n\t"
      "v_add_u32 v7, 0x4000, %[ixa]\n\t"
      "v_xor_b32 v8, 64, v7\n\t"
      "v_add_u32 v9, s87, %[ixa]\n\t"
      "v_xor_b32 v10, 64, v9\n\t"
      "v_add_u32 v11, %[sdxe], %[ixa]\n\t"
      "v_xor_b32 v12, 64, v11\n\t"
      "v_xor_b32 v13, 0, %[iwa]\n\t"
      "v_xor_b32 v14, 16, %[iwa]\n\t"
      "v_xor_b32 v15, 32, %[iwa]\n\t"
      "v_xor_b32 v16, 48, %[iwa]\n\t"
      "v_xor_b32 v17, 64, %[vsw0]\n\t"
      "v_mov_b32 v18, %[sfree]\n\t"
      "v_and_b32 v19, 0xffffffc0, v18\n\t"
      "v_mov_b32 v96, 0\n\t"
      "v_mov_b32 v97, 0\n\t"
      "v_mov_b32 v98, 0\n\t"
      "v_mov_b32 v99, 0\n\t"
      "v_mov_b32 v100, 0\n\t"
      "v_mov_b32 v101, 0\n\t"
      "v_mov_b32 v102, 0\n\t"
      "v_mov_b32 v103, 0\n\t"
      "v_mov_b32 v104, 0\n\t"
      "v_mov_b32 v105, 0\n\t"
      "v_mov_b32 v106, 0\n\t"
      "v_mov_b32 v107, 0\n\t"
      "v_mov_b32 v108, 0\n\t"
      "v_mov_b32 v109, 0\n\t"
      "v_mov_b32 v110, 0\n\t"
      "v_mov_b32 v111, 0\n\t"
      "v_mov_b32 v112, 0\n\t"
      "v_mov_b32 v113, 0\n\t"
      "v_mov_b32 v114, 0\n\t"
      "v_mov_b32 v115, 0\n\t"
      "v_mov_b32 v116, 0\n\t"
      "v_mov_b32 v117, 0\n\t"
      "v_mov_b32 v118, 0\n\t"
      "v_mov_b32 v119, 0\n\t"
      "v_mov_b32 v120, 0\n\t"
      "v_mov_b32 v121, 0\n\t"
      "v_mov_b32 v122, 0\n\t"
      "v_mov_b32 v123, 0\n\t"
      "v_mov_b32 v124, 0\n\t"
      "v_mov_b32 v125, 0\n\t"
      "v_mov_b32 v126, 0\n\t"
      "v_mov_b32 v127, 0\n\t"
      "v_mov_b32 v128, 0\n\t"
      "v_mov_b32 v129, 0\n\t"
      "v_mov_b32 v130, 0\n\t"
      "v_mov_b32 v131, 0\n\t"
      "v_mov_b32 v132, 0\n\t"
      "v_mov_b32 v133, 0\n\t"
      "v_mov_b32 v134, 0\n\t"
      "v_mov_b32 v135, 0\n\t"
      "v_mov_b32 v136, 0\n\t"
      "v_mov_b32 v137, 0\n\t"
      "v_mov_b32 v138, 0\n\t"
      "v_mov_b32 v139, 0\n\t"
      "v_mov_b32 v140, 0\n\t"
      "v_mov_b32 v141, 0\n\t"
      "v_mov_b32 v142, 0\n\t"
      "v_mov_b32 v143, 0\n\t"
      "v_mov_b32 v144, 0\n\t"
      "v_mov_b32 v145, 0\n\t"
      "v_mov_b32 v146, 0\n\t"
      "v_mov_b32 v147, 0\n\t"
      "v_mov_b32 v148, 0\n\t"
      "v_mov_b32 v149, 0\n\t"
      "v_mov_b32 v150, 0\n\t"
      "v_mov_b32 v151, 0\n\t"
      "v_mov_b32 v152, 0\n\t"
      "v_mov_b32 v153, 0\n\t"
      "v_mov_b32 v154, 0\n\t"
      "v_mov_b32 v155, 0\n\t"
      "v_mov_b32 v156, 0\n\t"
      "v_mov_b32 v157, 0\n\t"
      "v_mov_b32 v158, 0\n\t"
      "v_mov_b32 v159, 0\n\t"
      "v_mov_b32 v160, 0\n\t"
      "v_mov_b32 v161, 0\n\t"
      "v_mov_b32 v162, 0\n\t"
      "v_mov_b32 v163, 0\n\t"
      "v_mov_b32 v164, 0\n\t"
      "v_mov_b32 v165, 0\n\t"
      "v_mov_b32 v166, 0\n\t"
      "v_mov_b32 v167, 0\n\t"
      "v_mov_b32 v168, 0\n\t"
      "v_mov_b32 v169, 0\n\t"
      "v_mov_b32 v170, 0\n\t"
      "v_mov_b32 v171, 0\n\t"
      "v_mov_b32 v172, 0\n\t"
      "v_mov_b32 v173, 0\n\t"
      "v_mov_b32 v174, 0\n\t"
      "v_mov_b32 v175, 0\n\t"
      "v_mov_b32 v176, 0\n\t"
      "v_mov_b32 v177, 0\n\t"
      "v_mov_b32 v178, 0\n\t"
      "v_mov_b32 v179, 0\n\t"
      "v_mov_b32 v180, 0\n\t"
      "v_mov_b32 v181, 0\n\t"
      "v_mov_b32 v182, 0\n\t"
      "v_mov_b32 v183, 0\n\t"
      "v_mov_b32 v184, 0\n\t"
      "v_mov_b32 v185, 0\n\t"
      "v_mov_b32 v186, 0\n\t"
      "v_mov_b32 v187, 0\n\t"
      "v_mov_b32 v188, 0\n\t"
      "v_mov_b32 v189, 0\n\t"
      "v_mov_b32 v190, 0\n\t"
      "v_mov_b32 v191, 0\n\t"
      "v_mov_b32 v192, 0\n\t"
      "v_mov_b32 v193, 0\n\t"
      "v_mov_b32 v194, 0\n\t"
      "v_mov_b32 v195, 0\n\t"
      "v_mov_b32 v196, 0\n\t"
      "v_mov_b32 v197, 0\n\t"
      "v_mov_b32 v198, 0\n\t"
      "v_mov_b32 v199, 0\n\t"
      "v_mov_b32 v200, 0\n\t"
      "v_mov_b32 v201, 0\n\t"
      "v_mov_b32 v202, 0\n\t"
      "v_mov_b32 v203, 0\n\t"
      "v_mov_b32 v204, 0\n\t"
      "v_mov_b32 v205, 0\n\t"
      "v_mov_b32 v206, 0\n\t"
      "v_mov_b32 v207, 0\n\t"
      "v_mov_b32 v208, 0\n\t"
      "v_mov_b32 v209, 0\n\t"
      "v_mov_b32 v210, 0\n\t"
      "v_mov_b32 v211, 0\n\t"
      "v_mov_b32 v212, 0\n\t"
      "v_mov_b32 v213, 0\n\t"
      "v_mov_b32 v214, 0\n\t"
      "v_mov_b32 v215, 0\n\t"
      "v_mov_b32 v216, 0\n\t"
      "v_mov_b32 v217, 0\n\t"
      "v_mov_b32 v218, 0\n\t"
      "v_mov_b32 v219, 0\n\t"
      "v_mov_b32 v220, 0\n\t"
      "v_mov_b32 v221, 0\n\t"
      "v_mov_b32 v222, 0\n\t"
      "v_mov_b32 v223, 0\n\t"
      "v_mov_b32 v224, 0\n\t"
      "v_mov_b32 v225, 0\n\t"
      "v_mov_b32 v226, 0\n\t"
      "v_mov_b32 v227, 0\n\t"
      "v_mov_b32 v228, 0\n\t"
      "v_mov_b32 v229, 0\n\t"
      "v_mov_b32 v230, 0\n\t"
      "v_mov_b32 v231, 0\n\t"
      "v_mov_b32 v232, 0\n\t"
      "v_mov_b32 v233, 0\n\t"
      "v_mov_b32 v234, 0\n\t"
      "v_mov_b32 v235, 0\n\t"
      "v_mov_b32 v236, 0\n\t"
      "v_mov_b32 v237, 0\n\t"
      "v_mov_b32 v238, 0\n\t"
      "v_mov_b32 v239, 0\n\t"
      "v_mov_b32 v240, 0\n\t"
      "v_mov_b32 v241, 0\n\t"
      "v_mov_b32 v242, 0\n\t"
      "v_mov_b32 v243, 0\n\t"
      "v_mov_b32 v244, 0\n\t"
      "v_mov_b32 v245, 0\n\t"
      "v_mov_b32 v246, 0\n\t"
      "v_mov_b32 v247, 0\n\t"
      "v_mov_b32 v248, 0\n\t"
      "v_mov_b32 v249, 0\n\t"
      "v_mov_b32 v250, 0\n\t"
      "v_mov_b32 v251, 0\n\t"
      "v_mov_b32 v252, 0\n\t"
      "v_mov_b32 v253, 0\n\t"
      "v_mov_b32 v254, 0\n\t"
      "v_mov_b32 v255, 0\n\t"
      "SPINP_%=:\n\t"
      "ds_read_b64 v[22:23], v19\n\t"
      "s_waitcnt lgkmcnt(0)\n\t"
      "v_readfirstlane_b32 s83, v22\n\t"
      "v_readfirstlane_b32 s84, v23\n\t"
      "s_cmp_gt_u32 s84, s80\n\t"
      "s_cselect_b32 s89, s83, 0\n\t"
      "s_cmp_gt_u32 s89, s81\n\t"
      "s_cbranch_scc1 GOP_%=\n\t"
      "s_sleep 1\n\t"
      "s_add_u32 s91, s91, 1\n\t"
      "s_and_b32 s96, s91, 1023\n\t"
      "s_cbranch_scc1 SPINP_%=\n\t"
      "s_memrealtime s[96:97]\n\t"
      "s_waitcnt lgkmcnt(0)\n\t"
      "s_sub_u32 s96, %[tlo], s96\n\t"
      "s_subb_u32 s97, %[thi], s97\n\t"
      "s_cbranch_scc0 SPINP_%=\n\t"
      "v_mov_b32 v22, 1\n\t"
      "ds_write_b32 v19, v22 offset:36\n\t"
      "GOP_%=:\n\t"
      "s_mov_b32 s81, 1\n\t"
      "s_cmp_eq_u32 %[sext], 0\n\t"
      "s_cbranch_scc0 PROGX_%=\n\t"
      "PROGN_%=:\n\t"
      "ds_read_b128 v[56:59], v11 offset:0\n\t"
      "ds_read_b128 v[60:63], v12 offset:0\n\t"
      "ds_read_b128 v[64:67], v7 offset:0\n\t"
      "ds_read_b128 v[68:71], v8 offset:0\n\t"
      "ds_read_b64_tr_b8 v[24:25], v13 offset:0\n\t"
      "ds_read_b64_tr_b8 v[26:27], v13 offset:1024\n\t"
      "ds_read_b64_tr_b8 v[28:29], v13 offset:8192\n\t"
      "ds_read_b64_tr_b8 v[30:31], v13 offset:9216\n\t"
      "ds_read_b64_tr_b8 v[32:33], v14 offset:0\n\t"
      "ds_read_b64_tr_b8 v[34:35], v14 offset:1024\n\t"
      "ds_read_b64_tr_b8 v[36:37], v14 offset:8192\n\t"
      "ds_read_b64_tr_b8 v[38:39], v14 offset:9216\n\t"
      "ds_read_b64_tr_b8 v[40:41], v15 offset:0\n\t"
      "ds_read_b64_tr_b8 v[42:43], v15 offset:1024\n\t"
      "ds_read_b64_tr_b8 v[44:45], v15 offset:8192\n\t"
      "ds_read_b64_tr_b8 v[46:47], v15 offset:9216\n\t"
      "ds_read_b64_tr_b8 v[48:49], v16 offset:0\n\t"
      "ds_read_b64_tr_b8 v[50:51], v16 offset:1024\n\t"
      "ds_read_b64_tr_b8 v[52:53], v16 offset:8192\n\t"
      "ds_read_b64_tr_b8 v[54:55], v16 offset:9216\n\t"
      "s_add_u32 s82, s82, 1\n\t"
      "v_mov_b32 v21, s82\n\t"
      "ds_write_b32 v18, v21\n\t"
      "P1N_%=:\n\t"
      "s_waitcnt lgkmcnt(13)\n\t"
      "v_mfma_f32_16x16x128_f8f6f4 v[224:227], v[24:31], v[56:63], v[224:227] cbsz:1\n\t"
      "ds_read_b128 v[72:75], v7 offset:2048\n\t"
      "ds_read_b128 v[76:79], v8 offset:2048\n\t"
      "ds_read_b128 v[80:83], v7 offset:4096\n\t"
      "ds_read_b128 v[84:87], v8 offset:4096\n\t"
      "ds_read_b128 v[88:91], v7 offset:6144\n\t"
      "v_mfma_f32_16x16x128_f8f6f4 v[96:99], v[24:31], v[64:71], v[96:99] cbsz:1\n\t"
      "ds_read_b128 v[92:95], v8 offset:6144\n\t"
      "s_add_u32 s82, s82, 1\n\t"
      "v_mov_b32 v21, s82\n\t"
      "ds_write_b32 v18, v21\n\t"
      "s_add_u32 s89, s86, 32768\n\t"
      "s_and_b32 s89, s89, 0x1ffff\n\t"
      "s_waitcnt lgkmcnt(15)\n\t"
      "v_mfma_f32_16x16x128_f8f6f4 v[228:231], v[32:39], v[56:63], v[228:231] cbsz:1\n\t"
      "s_xor_b32 s90, s89, s86\n\t"
      "s_mov_b32 s86, s89\n\t"
      "v_xor_b32 v7, s90, v7\n\t"
      "v_xor_b32 v8, s90, v8\n\t"
      "s_add_u32 s89, s85, 32768\n\t"
      "s_and_b32 s89, s89, 0x1ffff\n\t"
      "s_xor_b32 s90, s89, s85\n\t"
      "v_mfma_f32_16x16x128_f8f6f4 v[100:103], v[32:39], v[64:71], v[100:103] cbsz:1\n\t"
      "s_mov_b32 s85, s89\n\t"
      "ds_read_b64 v[22:23], v19\n\t"
      "v_xor_b32 v13, s90, v13\n\t"
      "v_xor_b32 v14, s90, v14\n\t"
      "v_xor_b32 v15, s90, v15\n\t"
      "s_waitcnt lgkmcnt(13)\n\t"
      "v_mfma_f32_16x16x128_f8f6f4 v[232:235], v[40:47], v[56:63], v[232:235] cbsz:1\n\t"
      "v_xor_b32 v16, s90, v16\n\t"
      "s_add_u32 s89, s88, 0x2000\n\t"
      "s_cmp_eq_u32 s89, 0x6000\n\t"
      "s_cselect_b32 s89, 0, s89\n\t"
      "s_xor_b32 s90, s89, s88\n\t"
      "s_mov_b32 s88, s89\n\t"
      "v_xor_b32 v11, s90, v11\n\t"
      "v_mfma_f32_16x16x128_f8f6f4 v[104:107], v[40:47], v[64:71], v[104:107] cbsz:1\n\t"
      "v_xor_b32 v12, s90, v12\n\t"
      "s_add_u32 s80, s80, 1\n\t"
      "s_waitcnt lgkmcnt(9)\n\t"
      "v_mfma_f32_16x16x128_f8f6f4 v[236:239], v[48:55], v[56:63], v[236:239] cbsz:1\n\t"
      "s_waitcnt lgkmcnt(0)\n\t"
      "v_mfma_f32_16x16x128_f8f6f4 v[108:111], v[48:55], v[64:71], v[108:111] cbsz:1\n\t"
      "v_readfirstlane_b32 s84, v23\n\t"
      "v_readfirstlane_b32 s83, v22\n\t"
      "s_cmp_gt_u32 s84, s80\n\t"
      "s_cbranch_scc0 Z1N_%=\n\t"
      "s_cmp_gt_u32 s83, s81\n\t"
      "s_cbranch_scc0 S1N_%=\n\t"
      "v_mfma_f32_16x16x128_f8f6f4 v[112:115], v[24:31], v[72:79], v[112:115] cbsz:1\n\t"
      "ds_read_b128 v[56:59], v11 offset:0\n\t"
      "ds_read_b128 v[60:63], v12 offset:0\n\t"
      "v_mfma_f32_16x16x128_f8f6f4 v[128:131], v[24:31], v[80:87], v[128:131] cbsz:1\n\t"
      "ds_read_b128 v[64:67], v7 offset:0\n\t"
      "ds_read_b128 v[68:71], v8 offset:0\n\t"
      "v_mfma_f32_16x16x128_f8f6f4 v[144:147], v[24:31], v[88:95], v[144:147] cbsz:1\n\t"
      "ds_read_b64_tr_b8 v[24:25], v13 offset:0\n\t"
      "ds_read_b64_tr_b8 v[26:27], v13 offset:1024\n\t"
      "v_mfma_f32_16x16x128_f8f6f4 v[116:119], v[32:39], v[72:79], v[116:119] cbsz:1\n\t"
      "ds_read_b64_tr_b8 v[28:29], v13 offset:8192\n\t"
      "ds_read_b64_tr_b8 v[30:31], v13 offset:9216\n\t"
      "v_mfma_f32_16x16x128_f8f6f4 v[132:135], v[32:39], v[80:87], v[132:135] cbsz:1\n\t"
      "v_mfma_f32_16x16x128_f8f6f4 v[148:151], v[32:39], v[88:95], v[148:151] cbsz:1\n\t"
      "ds_read_b64_tr_b8 v[32:33], v14 offset:0\n\t"
      "ds_read_b64_tr_b8 v[34:35], v14 offset:1024\n\t"
      "v_mfma_f32_16x16x128_f8f6f4 v[120:123], v[40:47], v[72:79], v[120:123] cbsz:1\n\t"
      "ds_read_b64_tr_b8 v[36:37], v14 offset:8192\n\t"
      "ds_read_b64_tr_b8 v[38:39], v14 offset:9216\n\t"
      "v_mfma_f32_16x16x128_f8f6f4 v[136:139], v[40:47], v[80:87], v[136:139] cbsz:1\n\t"
      "v_mfma_f32_16x16x128_f8f6f4 v[152:155], v[40:47], v[88:95], v[152:155] cbsz:1\n\t"
      "ds_read_b64_tr_b8 v[40:41], v15 offset:0\n\t"
      "ds_read_b64_tr_b8 v[42:43], v15 offset:1024\n\t"
      "v_mfma_f32_16x16x128_f8f6f4 v[124:127], v[48:55], v[72:79], v[124:127] cbsz:1\n\t"
      "ds_read_b64_tr_b8 v[44:45], v15 offset:8192\n\t"
      "ds_read_b64_tr_b8 v[46:47], v15 offset:9216\n\t"
      "s_add_u32 s82, s82, 1\n\t"
      "v_mov_b32 v21, s82\n\t"
      "v_mfma_f32_16x16x128_f8f6f4 v[140:143], v[48:55], v[80:87], v[140:143] cbsz:1\n\t"
      "v_mfma_f32_16x16x128_f8f6f4 v[156:159], v[48:55], v[88:95], v[156:159] cbsz:1\n\t"
      "ds_read_b64_tr_b8 v[48:49], v16 offset:0\n\t"
      "ds_read_b64_tr_b8 v[50:51], v16 offset:1024\n\t"
      "ds_read_b64_tr_b8 v[52:53], v16 offset:8192\n\t"
      "ds_read_b64_tr_b8 v[54:55], v16 offset:9216\n\t"
      "ds_write_b32 v18, v21\n\t"
      "JOIN1N_%=:\n\t"
      "s_add_u32 s81, s81, 1\n\t"
      "s_cmp_lt_u32 s80, %[dt]\n\t"
      "s_cbranch_scc1 P1N_%=\n\t"
      "P2N_%=:\n\t"
      "s_waitcnt lgkmcnt(13)\n\t"
      "v_mfma_f32_16x16x128_f8f6f4 v[224:227], v[24:31], v[56:63], v[224:227] cbsz:1\n\t"
      "ds_read_b128 v[72:75], v7 offset:2048\n\t"
      "ds_read_b128 v[76:79], v8 offset:2048\n\t"
      "ds_read_b128 v[80:83], v7 offset:4096\n\t"
      "v_mfma_f32_16x16x128_f8f6f4 v[96:99], v[24:31], v[64:71], v[96:99] cbsz:1\n\t"
      "ds_read_b128 v[84:87], v8 offset:4096\n\t"
      "ds_read_b128 v[88:91], v7 offset:6144\n\t"
      "ds_read_b128 v[92:95], v8 offset:6144\n\t"
      "s_waitcnt lgkmcnt(15)\n\t"
      "v_mfma_f32_16x16x128_f8f6f4 v[228:231], v[32:39], v[56:63], v[228:231] cbsz:1\n\t"
      "s_add_u32 s82, s82, 1\n\t"
      "v_mov_b32 v21, s82\n\t"
      "ds_write_b32 v18, v21\n\t"
      "v_mfma_f32_16x16x128_f8f6f4 v[100:103], v[32:39], v[64:71], v[100:103] cbsz:1\n\t"
      "s_add_u32 s89, s86, 49152\n\t"
      "s_and_b32 s89, s89, 0x1ffff\n\t"
      "s_xor_b32 s90, s89, s86\n\t"
      "s_mov_b32 s86, s89\n\t"
      "v_xor_b32 v7, s90, v7\n\t"
      "s_waitcnt lgkmcnt(12)\n\t"
      "v_mfma_f32_16x16x128_f8f6f4 v[232:235], v[40:47], v[56:63], v[232:235] cbsz:1\n\t"
      "v_xor_b32 v8, s90, v8\n\t"
      "ds_read_b64 v[22:23], v19\n\t"
      "v_mfma_f32_16x16x128_f8f6f4 v[104:107], v[40:47], v[64:71], v[104:107] cbsz:1\n\t"
      "s_waitcnt lgkmcnt(9)\n\t"
      "v_mfma_f32_16x16x128_f8f6f4 v[236:239], v[48:55], v[56:63], v[236:239] cbsz:1\n\t"
      "v_mfma_f32_16x16x128_f8f6f4 v[108:111], v[48:55], v[64:71], v[108:111] cbsz:1\n\t"
      "s_waitcnt lgkmcnt(6)\n\t"
      "v_mfma_f32_16x16x128_f8f6f4 v[112:115], v[24:31], v[72:79], v[112:115] cbsz:1\n\t"
      "s_add_u32 s89, s85, 49152\n\t"
      "s_and_b32 s89, s89, 0x1ffff\n\t"
      "s_xor_b32 s90, s89, s85\n\t"
      "s_mov_b32 s85, s89\n\t"
      "s_waitcnt lgkmcnt(4)\n\t"
      "v_mfma_f32_16x16x128_f8f6f4 v[128:131], v[24:31], v[80:87], v[128:131] cbsz:1\n\t"
      "v_xor_b32 v13, s90, v13\n\t"
      "v_xor_b32 v14, s90, v14\n\t"
      "s_waitcnt lgkmcnt(2)\n\t"
      "v_mfma_f32_16x16x128_f8f6f4 v[144:147], v[24:31], v[88:95], v[144:147] cbsz:1\n\t"
      "v_xor_b32 v15, s90, v15\n\t"
      "v_xor_b32 v16, s90, v16\n\t"
      "v_mfma_f32_16x16x128_f8f6f4 v[116:119], v[32:39], v[72:79], v[116:119] cbsz:1\n\t"
      "s_add_u32 s89, s88, 0x2000\n\t"
      "s_cmp_eq_u32 s89, 0x6000\n\t"
      "s_cselect_b32 s89, 0, s89\n\t"
      "s_xor_b32 s90, s89, s88\n\t"
      "v_mfma_f32_16x16x128_f8f6f4 v[132:135], v[32:39], v[80:87], v[132:135] cbsz:1\n\t"
      "s_mov_b32 s88, s89\n\t"
      "v_xor_b32 v11, s90, v11\n\t"
      "v_xor_b32 v12, s90, v12\n\t"
      "v_mfma_f32_16x16x128_f8f6f4 v[148:151], v[32:39], v[88:95], v[148:151] cbsz:1\n\t"
      "s_add_u32 s80, s80, 1\n\t"
      "s_waitcnt lgkmcnt(0)\n\t"
      "v_readfirstlane_b32 s83, v22\n\t"
      "s_cmp_gt_u32 s83, s81\n\t"
      "s_cbranch_scc0 SPINL2N_%=\n\t"
      "BACKL2N_%=:\n\t"
      "s_add_u32 s81, s81, 1\n\t"
      "ds_read_b128 v[56:59], v9 offset:0\n\t"
      "v_mfma_f32_16x16x128_f8f6f4 v[120:123], v[40:47], v[72:79], v[120:123] cbsz:1\n\t"
      "ds_read_b128 v[60:63], v10 offset:0\n\t"
      "ds_read_b128 v[64:67], v9 offset:2048\n\t"
      "v_mfma_f32_16x16x128_f8f6f4 v[136:139], v[40:47], v[80:87], v[136:139] cbsz:1\n\t"
      "ds_read_b128 v[68:71], v10 offset:2048\n\t"
      "v_mfma_f32_16x16x128_f8f6f4 v[152:155], v[40:47], v[88:95], v[152:155] cbsz:1\n\t"
      "v_mfma_f32_16x16x128_f8f6f4 v[124:127], v[48:55], v[72:79], v[124:127] cbsz:1\n\t"
      "v_mfma_f32_16x16x128_f8f6f4 v[140:143], v[48:55], v[80:87], v[140:143] cbsz:1\n\t"
      "v_mfma_f32_16x16x128_f8f6f4 v[156:159], v[48:55], v[88:95], v[156:159] cbsz:1\n\t"
      "s_waitcnt lgkmcnt(2)\n\t"
      "v_mfma_f32_16x16x128_f8f6f4 v[160:163], v[24:31], v[56:63], v[160:163] cbsz:1\n\t"
      "ds_read_b128 v[72:75], v9 offset:4096\n\t"
      "ds_read_b128 v[76:79], v10 offset:4096\n\t"
      "s_waitcnt lgkmcnt(2)\n\t"
      "v_mfma_f32_16x16x128_f8f6f4 v[176:179], v[24:31], v[64:71], v[176:179] cbsz:1\n\t"
      "ds_read_b128 v[80:83], v9 offset:6144\n\t"
      "ds_read_b128 v[84:87], v10 offset:6144\n\t"
      "v_mfma_f32_16x16x128_f8f6f4 v[164:167], v[32:39], v[56:63], v[164:167] cbsz:1\n\t"
      "s_add_u32 s82, s82, 1\n\t"
      "v_mov_b32 v21, s82\n\t"
      "ds_write_b32 v18, v21\n\t"
      "v_mfma_f32_16x16x128_f8f6f4 v[180:183], v[32:39], v[64:71], v[180:183] cbsz:1\n\t"
      "s_add_u32 s89, s87, 49152\n\t"
      "ds_read_b64 v[22:23], v19\n\t"
      "s_and_b32 s89, s89, 0x1ffff\n\t"
      "v_mfma_f32_16x16x128_f8f6f4 v[168:171], v[40:47], v[56:63], v[168:171] cbsz:1\n\t"
      "s_xor_b32 s90, s89, s87\n\t"
      "s_mov_b32 s87, s89\n\t"
      "v_xor_b32 v9, s90, v9\n\t"
      "v_mfma_f32_16x16x128_f8f6f4 v[184:187], v[40:47], v[64:71], v[184:187] cbsz:1\n\t"
      "v_xor_b32 v10, s90, v10\n\t"
      "v_mfma_f32_16x16x128_f8f6f4 v[172:175], v[48:55], v[56:63], v[172:175] cbsz:1\n\t"
      "s_waitcnt lgkmcnt(0)\n\t"
      "v_mfma_f32_16x16x128_f8f6f4 v[188:191], v[48:55], v[64:71], v[188:191] cbsz:1\n\t"
      "v_readfirstlane_b32 s84, v23\n\t"
      "v_readfirstlane_b32 s83, v22\n\t"
      "s_cmp_gt_u32 s84, s80\n\t"
      "s_cbranch_scc0 Z2N_%=\n\t"
      "s_cmp_gt_u32 s83, s81\n\t"
      "s_cbranch_scc0 S2N_%=\n\t"
      "v_mfma_f32_16x16x128_f8f6f4 v[192:195], v[24:31], v[72:79], v[192:195] cbsz:1\n\t"
      "ds_read_b128 v[56:59], v11 offset:0\n\t"
      "ds_read_b128 v[60:63], v12 offset:0\n\t"
      "ds_read_b128 v[64:67], v7 offset:0\n\t"
      "v_mfma_f32_16x16x128_f8f6f4 v[208:211], v[24:31], v[80:87], v[208:211] cbsz:1\n\t"
      "ds_read_b128 v[68:71], v8 offset:0\n\t"
      "ds_read_b64_tr_b8 v[24:25], v13 offset:0\n\t"
      "ds_read_b64_tr_b8 v[26:27], v13 offset:1024\n\t"
      "v_mfma_f32_16x16x128_f8f6f4 v[196:199], v[32:39], v[72:79], v[196:199] cbsz:1\n\t"
      "ds_read_b64_tr_b8 v[28:29], v13 offset:8192\n\t"
      "ds_read_b64_tr_b8 v[30:31], v13 offset:9216\n\t"
      "v_mfma_f32_16x16x128_f8f6f4 v[212:215], v[32:39], v[80:87], v[212:215] cbsz:1\n\t"
      "ds_read_b64_tr_b8 v[32:33], v14 offset:0\n\t"
      "ds_read_b64_tr_b8 v[34:35], v14 offset:1024\n\t"
      "ds_read_b64_tr_b8 v[36:37], v14 offset:8192\n\t"
      "v_mfma_f32_16x16x128_f8f6f4 v[200:203], v[40:47], v[72:79], v[200:203] cbsz:1\n\t"
      "ds_read_b64_tr_b8 v[38:39], v14 offset:9216\n\t"
      "v_mfma_f32_16x16x128_f8f6f4 v[216:219], v[40:47], v[80:87], v[216:219] cbsz:1\n\t"
      "ds_read_b64_tr_b8 v[40:41], v15 offset:0\n\t"
      "ds_read_b64_tr_b8 v[42:43], v15 offset:1024\n\t"
      "ds_read_b64_tr_b8 v[44:45], v15 offset:8192\n\t"
      "v_mfma_f32_16x16x128_f8f6f4 v[204:207], v[48:55], v[72:79], v[204:207] cbsz:1\n\t"
      "ds_read_b64_tr_b8 v[46:47], v15 offset:9216\n\t"
      "s_add_u32 s82, s82, 1\n\t"
      "v_mov_b32 v21, s82\n\t"
      "v_mfma_f32_16x16x128_f8f6f4 v[220:223], v[48:55], v[80:87], v[220:223] cbsz:1\n\t"
      "ds_read_b64_tr_b8 v[48:49], v16 offset:0\n\t"
      "ds_read_b64_tr_b8 v[50:51], v16 offset:1024\n\t"
      "ds_read_b64_tr_b8 v[52:53], v16 offset:8192\n\t"
      "ds_read_b64_tr_b8 v[54:55], v16 offset:9216\n\t"
      "ds_write_b32 v18, v21\n\t"
      "JOIN2N_%=:\n\t"
      "s_add_u32 s81, s81, 1\n\t"
      "s_cmp_lt_u32 s80, 31\n\t"
      "s_cbranch_scc1 P2N_%=\n\t"
      "s_waitcnt lgkmcnt(13)\n\t"
      "v_mfma_f32_16x16x128_f8f6f4 v[224:227], v[24:31], v[56:63], v[224:227] cbsz:1\n\t"
      "ds_read_b128 v[72:75], v7 offset:2048\n\t"
      "ds_read_b128 v[76:79], v8 offset:2048\n\t"
      "v_mfma_f32_16x16x128_f8f6f4 v[96:99], v[24:31], v[64:71], v[96:99] cbsz:1\n\t"
      "ds_read_b128 v[80:83], v7 offset:4096\n\t"
      "ds_read_b128 v[84:87], v8 offset:4096\n\t"
      "s_waitcnt lgkmcnt(13)\n\t"
      "v_mfma_f32_16x16x128_f8f6f4 v[228:231], v[32:39], v[56:63], v[228:231] cbsz:1\n\t"
      "ds_read_b128 v[88:91], v7 offset:6144\n\t"
      "ds_read_b128 v[92:95], v8 offset:6144\n\t"
      "v_mfma_f32_16x16x128_f8f6f4 v[100:103], v[32:39], v[64:71], v[100:103] cbsz:1\n\t"
      "s_add_u32 s82, s82, 1\n\t"
      "v_mov_b32 v21, s82\n\t"
      "ds_write_b32 v18, v21\n\t"
      "s_waitcnt lgkmcnt(12)\n\t"
      "v_mfma_f32_16x16x128_f8f6f4 v[232:235], v[40:47], v[56:63], v[232:235] cbsz:1\n\t"
      "ds_read_b64 v[22:23], v19\n\t"
      "v_mfma_f32_16x16x128_f8f6f4 v[104:107], v[40:47], v[64:71], v[104:107] cbsz:1\n\t"
      "s_waitcnt lgkmcnt(9)\n\t"
      "v_mfma_f32_16x16x128_f8f6f4 v[236:239], v[48:55], v[56:63], v[236:239] cbsz:1\n\t"
      "v_mfma_f32_16x16x128_f8f6f4 v[108:111], v[48:55], v[64:71], v[108:111] cbsz:1\n\t"
      "s_waitcnt lgkmcnt(6)\n\t"
      "v_mfma_f32_16x16x128_f8f6f4 v[112:115], v[24:31], v[72:79], v[112:115] cbsz:1\n\t"
      "s_add_u32 s89, s85, 49152\n\t"
      "s_and_b32 s89, s89, 0x1ffff\n\t"
      "s_xor_b32 s90, s89, s85\n\t"
      "s_mov_b32 s85, s89\n\t"
      "s_waitcnt lgkmcnt(4)\n\t"
      "v_mfma_f32_16x16x128_f8f6f4 v[128:131], v[24:31], v[80:87], v[128:131] cbsz:1\n\t"
      "v_xor_b32 v13, s90, v13\n\t"
      "v_xor_b32 v14, s90, v14\n\t"
      "s_waitcnt lgkmcnt(2)\n\t"
      "v_mfma_f32_16x16x128_f8f6f4 v[144:147], v[24:31], v[88:95], v[144:147] cbsz:1\n\t"
      "v_xor_b32 v15, s90, v15\n\t"
      "v_xor_b32 v16, s90, v16\n\t"
      "v_mfma_f32_16x16x128_f8f6f4 v[116:119], v[32:39], v[72:79], v[116:119] cbsz:1\n\t"
      "s_add_u32 s89, s88, 0x2000\n\t"
      "s_cmp_eq_u32 s89, 0x6000\n\t"
      "s_cselect_b32 s89, 0, s89\n\t"
      "s_xor_b32 s90, s89, s88\n\t"
      "v_mfma_f32_16x16x128_f8f6f4 v[132:135], v[32:39], v[80:87], v[132:135] cbsz:1\n\t"
      "s_mov_b32 s88, s89\n\t"
      "v_xor_b32 v11, s90, v11\n\t"
      "v_xor_b32 v12, s90, v12\n\t"
      "v_mfma_f32_16x16x128_f8f6f4 v[148:151], v[32:39], v[88:95], v[148:151] cbsz:1\n\t"
      "s_add_u32 s80, s80, 1\n\t"
      "s_waitcnt lgkmcnt(0)\n\t"
      "v_readfirstlane_b32 s83, v22\n\t"
      "s_cmp_gt_u32 s83, s81\n\t"
      "s_cbranch_scc0 SPINL2LN_%=\n\t"
      "BACKL2LN_%=:\n\t"
      "s_add_u32 s81, s81, 1\n\t"
      "ds_read_b128 v[56:59], v9 offset:0\n\t"
      "v_mfma_f32_16x16x128_f8f6f4 v[120:123], v[40:47], v[72:79], v[120:123] cbsz:1\n\t"
      "ds_read_b128 v[60:63], v10 offset:0\n\t"
      "ds_read_b128 v[64:67], v9 offset:2048\n\t"
      "v_mfma_f32_16x16x128_f8f6f4 v[136:139], v[40:47], v[80:87], v[136:139] cbsz:1\n\t"
      "ds_read_b128 v[68:71], v10 offset:2048\n\t"
      "v_mfma_f32_16x16x128_f8f6f4 v[152:155], v[40:47], v[88:95], v[152:155] cbsz:1\n\t"
      "v_mfma_f32_16x16x128_f8f6f4 v[124:127], v[48:55], v[72:79], v[124:127] cbsz:1\n\t"
      "v_mfma_f32_16x16x128_f8f6f4 v[140:143], v[48:55], v[80:87], v[140:143] cbsz:1\n\t"
      "v_mfma_f32_16x16x128_f8f6f4 v[156:159], v[48:55], v[88:95], v[156:159] cbsz:1\n\t"
      "s_waitcnt lgkmcnt(2)\n\t"
      "v_mfma_f32_16x16x128_f8f6f4 v[160:163], v[24:31], v[56:63], v[160:163] cbsz:1\n\t"
      "ds_read_b128 v[72:75], v9 offset:4096\n\t"
      "ds_read_b128 v[76:79], v10 offset:4096\n\t"
      "s_waitcnt lgkmcnt(2)\n\t"
      "v_mfma_f32_16x16x128_f8f6f4 v[176:179], v[24:31], v[64:71], v[176:179] cbsz:1\n\t"
      "ds_read_b128 v[80:83], v9 offset:6144\n\t"
      "ds_read_b128 v[84:87], v10 offset:6144\n\t"
      "v_mfma_f32_16x16x128_f8f6f4 v[164:167], v[32:39], v[56:63], v[164:167] cbsz:1\n\t"
      "s_add_u32 s82, s82, 1\n\t"
      "v_mov_b32 v21, s82\n\t"
      "ds_write_b32 v18, v21\n\t"
      "v_mfma_f32_16x16x128_f8f6f4 v[180:183], v[32:39], v[64:71], v[180:183] cbsz:1\n\t"
      "s_add_u32 s89, s87, 32768\n\t"
      "ds_read_b64 v[22:23], v19\n\t"
      "s_and_b32 s89, s89, 0x1ffff\n\t"
      "v_mfma_f32_16x16x128_f8f6f4 v[168:171], v[40:47], v[56:63], v[168:171] cbsz:1\n\t"
      "s_xor_b32 s90, s89, s87\n\t"
      "s_mov_b32 s87, s89\n\t"
      "v_xor_b32 v9, s90, v9\n\t"
      "v_mfma_f32_16x16x128_f8f6f4 v[184:187], v[40:47], v[64:71], v[184:187] cbsz:1\n\t"
      "v_xor_b32 v10, s90, v10\n\t"
      "v_mfma_f32_16x16x128_f8f6f4 v[172:175], v[48:55], v[56:63], v[172:175] cbsz:1\n\t"
      "s_waitcnt lgkmcnt(0)\n\t"
      "v_mfma_f32_16x16x128_f8f6f4 v[188:191], v[48:55], v[64:71], v[188:191] cbsz:1\n\t"
      "v_readfirstlane_b32 s84, v23\n\t"
      "v_readfirstlane_b32 s83, v22\n\t"
      "s_cmp_gt_u32 s84, s80\n\t"
      "s_cbranch_scc0 Z2LN_%=\n\t"
      "s_cmp_gt_u32 s83, s81\n\t"
      "s_cbranch_scc0 S2LN_%=\n\t"
      "v_mfma_f32_16x16x128_f8f6f4 v[192:195], v[24:31], v[72:79], v[192:195] cbsz:1\n\t"
      "ds_read_b128 v[56:59], v9 offset:0\n\t"
      "ds_read_b128 v[60:63], v10 offset:0\n\t"
      "ds_read_b128 v[64:67], v9 offset:2048\n\t"
      "v_mfma_f32_16x16x128_f8f6f4 v[208:211], v[24:31], v[80:87], v[208:211] cbsz:1\n\t"
      "ds_read_b128 v[68:71], v10 offset:2048\n\t"
      "ds_read_b64_tr_b8 v[24:25], v13 offset:0\n\t"
      "ds_read_b64_tr_b8 v[26:27], v13 offset:1024\n\t"
      "v_mfma_f32_16x16x128_f8f6f4 v[196:199], v[32:39], v[72:79], v[196:199] cbsz:1\n\t"
      "ds_read_b64_tr_b8 v[28:29], v13 offset:8192\n\t"
      "ds_read_b64_tr_b8 v[30:31], v13 offset:9216\n\t"
      "v_mfma_f32_16x16x128_f8f6f4 v[212:215], v[32:39], v[80:87], v[212:215] cbsz:1\n\t"
      "ds_read_b64_tr_b8 v[32:33], v14 offset:0\n\t"
      "ds_read_b64_tr_b8 v[34:35], v14 offset:1024\n\t"
      "ds_read_b64_tr_b8 v[36:37], v14 offset:8192\n\t"
      "v_mfma_f32_16x16x128_f8f6f4 v[200:203], v[40:47], v[72:79], v[200:203] cbsz:1\n\t"
      "ds_read_b64_tr_b8 v[38:39], v14 offset:9216\n\t"
      "v_mfma_f32_16x16x128_f8f6f4 v[216:219], v[40:47], v[80:87], v[216:219] cbsz:1\n\t"
      "ds_read_b64_tr_b8 v[40:41], v15 offset:0\n\t"
      "ds_read_b64_tr_b8 v[42:43], v15 offset:1024\n\t"
      "ds_read_b64_tr_b8 v[44:45], v15 offset:8192\n\t"
      "v_mfma_f32_16x16x128_f8f6f4 v[204:207], v[48:55], v[72:79], v[204:207] cbsz:1\n\t"
      "ds_read_b64_tr_b8 v[46:47], v15 offset:9216\n\t"
      "s_add_u32 s82, s82, 1\n\t"
      "v_mov_b32 v21, s82\n\t"
      "v_mfma_f32_16x16x128_f8f6f4 v[220:223], v[48:55], v[80:87], v[220:223] cbsz:1\n\t"
      "ds_read_b64_tr_b8 v[48:49], v16 offset:0\n\t"
      "ds_read_b64_tr_b8 v[50:51], v16 offset:1024\n\t"
      "ds_read_b64_tr_b8 v[52:53], v16 offset:8192\n\t"
      "ds_read_b64_tr_b8 v[54:55], v16 offset:9216\n\t"
      "ds_write_b32 v18, v21\n\t"
      "JOIN2LN_%=:\n\t"
      "s_add_u32 s81, s81, 1\n\t"
      "EWN_%=:\n\t"
      "ds_read_b128 v[72:75], v19 offset:16\n\t"
      "ds_read_b32 v22, v19 offset:36\n\t"
      "s_waitcnt lgkmcnt(0)\n\t"
      "v_readfirstlane_b32 s89, v22\n\t"
      "s_cmp_lg_u32 s89, 0\n\t"
      "s_cselect_b32 s89, 0x7fc00000, %[sscale]\n\t"
      "v_mov_b32 v20, s89\n\t"
      "v_min_u32 v72, v72, v73\n\t"
      "v_min3_u32 v72, v72, v74, v75\n\t"
      "v_lshl_or_b32 v72, v22, 8, v72\n\t"
      "s_nop 0\n\t"
      "v_readfirstlane_b32 s89, v72\n\t"
      "s_cmp_gt_u32 s89, s95\n\t"
      "s_cbranch_scc0 EWN_%=\n\t"
      "v_pk_mul_f32 v[96:97], v[96:97], v[20:21] op_sel_hi:[1,0]\n\t"
      "v_pk_mul_f32 v[98:99], v[98:99], v[20:21] op_sel_hi:[1,0]\n\t"
      "v_pk_mul_f32 v[100:101], v[100:101], v[20:21] op_sel_hi:[1,0]\n\t"
      "v_pk_mul_f32 v[102:103], v[102:103], v[20:21] op_sel_hi:[1,0]\n\t"
      "ds_write_b128 %[vsw0], v[96:99]\n\t"
      "ds_write_b128 v17, v[100:103]\n\t"
      "ds_read_b128 v[96:99], %[vsr]\n\t"
      "ds_read_b128 v[100:103], %[vsr] offset:1024\n\t"
      "v_pk_mul_f32 v[104:105], v[104:105], v[20:21] op_sel_hi:[1,0]\n\t"
      "v_pk_mul_f32 v[106:107], v[106:107], v[20:21] op_sel_hi:[1,0]\n\t"
      "v_pk_mul_f32 v[108:109], v[108:109], v[20:21] op_sel_hi:[1,0]\n\t"
      "v_pk_mul_f32 v[110:111], v[110:111], v[20:21] op_sel_hi:[1,0]\n\t"
      "ds_write_b128 %[vsw0], v[104:107]\n\t"
      "ds_write_b128 v17, v[108:111]\n\t"
      "ds_read_b128 v[104:107], %[vsr]\n\t"
      "ds_read_b128 v[108:111], %[vsr] offset:1024\n\t"
      "v_pk_mul_f32 v[112:113], v[112:113], v[20:21] op_sel_hi:[1,0]\n\t"
      "v_pk_mul_f32 v[114:115], v[114:115], v[20:21] op_sel_hi:[1,0]\n\t"
      "v_pk_mul_f32 v[116:117], v[116:117], v[20:21] op_sel_hi:[1,0]\n\t"
      "v_pk_mul_f32 v[118:119], v[118:119], v[20:21] op_sel_hi:[1,0]\n\t"
      "ds_write_b128 %[vsw0], v[112:115]\n\t"
      "ds_write_b128 v17, v[116:119]\n\t"
      "ds_read_b128 v[112:115], %[vsr]\n\t"
      "ds_read_b128 v[116:119], %[vsr] offset:1024\n\t"
      "s_waitcnt lgkmcnt(8)\n\t"
      "s_add_u32 s93, %[sout], 0\n\t"
      "s_add_u32 s92, s93, 0x10000\n\t"
      "buffer_store_dwordx4 v[96:99], %[vout], %[ro], s93 offen offset:0 sc1\n\t"
      "buffer_store_dwordx4 v[100:103], %[vout], %[ro], s92 offen offset:0 sc1\n\t"
      "v_pk_mul_f32 v[120:121], v[120:121], v[20:21] op_sel_hi:[1,0]\n\t"
      "v_pk_mul_f32 v[122:123], v[122:123], v[20:21] op_sel_hi:[1,0]\n\t"
      "v_pk_mul_f32 v[124:125], v[124:125], v[20:21] op_sel_hi:[1,0]\n\t"
      "v_pk_mul_f32 v[126:127], v[126:127], v[20:21] op_sel_hi:[1,0]\n\t"
      "ds_write_b128 %[vsw0], v[120:123]\n\t"
      "ds_write_b128 v17, v[124:127]\n\t"
      "ds_read_b128 v[120:123], %[vsr]\n\t"
      "ds_read_b128 v[124:127], %[vsr] offset:1024\n\t"
      "s_waitcnt lgkmcnt(8)\n\t"
      "buffer_store_dwordx4 v[104:107], %[vout], %[ro], s93 offen offset:128 sc1\n\t"
      "buffer_store_dwordx4 v[108:111], %[vout], %[ro], s92 offen offset:128 sc1\n\t"
      "v_pk_mul_f32 v[128:129], v[128:129], v[20:21] op_sel_hi:[1,0]\n\t"
      "v_pk_mul_f32 v[130:131], v[130:131], v[20:21] op_sel_hi:[1,0]\n\t"
      "v_pk_mul_f32 v[132:133], v[132:133], v[20:21] op_sel_hi:[1,0]\n\t"
      "v_pk_mul_f32 v[134:135], v[134:135], v[20:21] op_sel_hi:[1,0]\n\t"
      "ds_write_b128 %[vsw0], v[128:131]\n\t"
      "ds_write_b128 v17, v[132:135]\n\t"
      "ds_read_b128 v[128:131], %[vsr]\n\t"
      "ds_read_b128 v[132:135], %[vsr] offset:1024\n\t"
      "s_waitcnt lgkmcnt(8)\n\t"
      "s_add_u32 s93, %[sout], 131072\n\t"
      "s_add_u32 s92, s93, 0x10000\n\t"
      "buffer_store_dwordx4 v[112:115], %[vout], %[ro], s93 offen offset:0 sc1\n\t"
      "buffer_store_dwordx4 v[116:119], %[vout], %[ro], s92 offen offset:0 sc1\n\t"
      "v_pk_mul_f32 v[136:137], v[136:137], v[20:21] op_sel_hi:[1,0]\n\t"
      "v_pk_mul_f32 v[138:139], v[138:139], v[20:21] op_sel_hi:[1,0]\n\t"
      "v_pk_mul_f32 v[140:141], v[140:141], v[20:21] op_sel_hi:[1,0]\n\t"
      "v_pk_mul_f32 v[142:143], v[142:143], v[20:21] op_sel_hi:[1,0]\n\t"
      "ds_write_b128 %[vsw0], v[136:139]\n\t"
      "ds_write_b128 v17, v[140:143]\n\t"
      "ds_read_b128 v[136:139], %[vsr]\n\t"
      "ds_read_b128 v[140:143], %[vsr] offset:1024\n\t"
      "s_waitcnt lgkmcnt(8)\n\t"
      "buffer_store_dwordx4 v[120:123], %[vout], %[ro], s93 offen offset:128 sc1\n\t"
      "buffer_store_dwordx4 v[124:127], %[vout], %[ro], s92 offen offset:128 sc1\n\t"
      "v_pk_mul_f32 v[144:145], v[144:145], v[20:21] op_sel_hi:[1,0]\n\t"
      "v_pk_mul_f32 v[146:147], v[146:147], v[20:21] op_sel_hi:[1,0]\n\t"
      "v_pk_mul_f32 v[148:149], v[148:149], v[20:21] op_sel_hi:[1,0]\n\t"
      "v_pk_mul_f32 v[150:151], v[150:151], v[20:21] op_sel_hi:[1,0]\n\t"
      "ds_write_b128 %[vsw0], v[144:147]\n\t"
      "ds_write_b128 v17, v[148:151]\n\t"
      "ds_read_b128 v[144:147], %[vsr]\n\t"
      "ds_read_b128 v[148:151], %[vsr] offset:1024\n\t"
      "s_waitcnt lgkmcnt(8)\n\t"
      "s_add_u32 s93, %[sout], 262144\n\t"
      "s_add_u32 s92, s93, 0x10000\n\t"
      "buffer_store_dwordx4 v[128:131], %[vout], %[ro], s93 offen offset:0 sc1\n\t"
      "buffer_store_dwordx4 v[132:135], %[vout], %[ro], s92 offen offset:0 sc1\n\t"
      "v_pk_mul_f32 v[152:153], v[152:153], v[20:21] op_sel_hi:[1,0]\n\t"
      "v_pk_mul_f32 v[154:155], v[154:155], v[20:21] op_sel_hi:[1,0]\n\t"
      "v_pk_mul_f32 v[156:157], v[156:157], v[20:21] op_sel_hi:[1,0]\n\t"
      "v_pk_mul_f32 v[158:159], v[158:159], v[20:21] op_sel_hi:[1,0]\n\t"
      "ds_write_b128 %[vsw0], v[152:155]\n\t"
      "ds_write_b128 v17, v[156:159]\n\t"
      "ds_read_b128 v[152:155], %[vsr]\n\t"
      "ds_read_b128 v[156:159], %[vsr] offset:1024\n\t"
      "s_waitcnt lgkmcnt(8)\n\t"
      "buffer_store_dwordx4 v[136:139], %[vout], %[ro], s93 offen offset:128 sc1\n\t"
      "buffer_store_dwordx4 v[140:143], %[vout], %[ro], s92 offen offset:128 sc1\n\t"
      "v_pk_mul_f32 v[224:225], v[224:225], v[20:21] op_sel_hi:[1,0]\n\t"
      "v_pk_mul_f32 v[226:227], v[226:227], v[20:21] op_sel_hi:[1,0]\n\t"
      "v_pk_mul_f32 v[228:229], v[228:229], v[20:21] op_sel_hi:[1,0]\n\t"
      "v_pk_mul_f32 v[230:231], v[230:231], v[20:21] op_sel_hi:[1,0]\n\t"
      "ds_write_b128 %[vsw0], v[224:227]\n\t"
      "ds_write_b128 v17, v[228:231]\n\t"
      "ds_read_b128 v[224:227], %[vsr]\n\t"
      "ds_read_b128 v[228:231], %[vsr] offset:1024\n\t"
      "s_waitcnt lgkmcnt(8)\n\t"
      "s_add_u32 s93, %[sout], 393216\n\t"
      "s_add_u32 s92, s93, 0x10000\n\t"
      "buffer_store_dwordx4 v[144:147], %[vout], %[ro], s93 offen offset:0 sc1\n\t"
      "buffer_store_dwordx4 v[148:151], %[vout], %[ro], s92 offen offset:0 sc1\n\t"
      "v_pk_mul_f32 v[232:233], v[232:233], v[20:21] op_sel_hi:[1,0]\n\t"
      "v_pk_mul_f32 v[234:235], v[234:235], v[20:21] op_sel_hi:[1,0]\n\t"
      "v_pk_mul_f32 v[236:237], v[236:237], v[20:21] op_sel_hi:[1,0]\n\t"
      "v_pk_mul_f32 v[238:239], v[238:239], v[20:21] op_sel_hi:[1,0]\n\t"
      "ds_write_b128 %[vsw0], v[232:235]\n\t"
      "ds_write_b128 v17, v[236:239]\n\t"
      "ds_read_b128 v[232:235], %[vsr]\n\t"
      "ds_read_b128 v[236:239], %[vsr] offset:1024\n\t"
      "s_waitcnt lgkmcnt(8)\n\t"
      "buffer_store_dwordx4 v[152:155], %[vout], %[ro], s93 offen offset:128 sc1\n\t"
      "buffer_store_dwordx4 v[156:159], %[vout], %[ro], s92 offen offset:128 sc1\n\t"
      "s_waitcnt lgkmcnt(4)\n\t"
      "s_mov_b32 s93, %[soute]\n\t"
      "s_add_u32 s92, s93, 0x10000\n\t"
      "buffer_store_dwordx4 v[224:227], %[vout], %[ro], s93 offen offset:0 sc1\n\t"
      "buffer_store_dwordx4 v[228:231], %[vout], %[ro], s92 offen offset:0 sc1\n\t"
      "s_waitcnt lgkmcnt(0)\n\t"
      "buffer_store_dwordx4 v[232:235], %[vout], %[ro], s93 offen offset:128 sc1\n\t"
      "buffer_store_dwordx4 v[236:239], %[vout], %[ro], s92 offen offset:128 sc1\n\t"
      "s_branch P3_%=\n\t"
      "PROGX_%=:\n\t"
      "ds_read_b128 v[56:59], v11 offset:0\n\t"
      "ds_read_b128 v[60:63], v12 offset:0\n\t"
      "ds_read_b128 v[64:67], v11 offset:4096\n\t"
      "ds_read_b128 v[68:71], v12 offset:4096\n\t"
      "ds_read_b64_tr_b8 v[24:25], v13 offset:0\n\t"
      "ds_read_b64_tr_b8 v[26:27], v13 offset:1024\n\t"
      "ds_read_b64_tr_b8 v[28:29], v13 offset:8192\n\t"
      "ds_read_b64_tr_b8 v[30:31], v13 offset:9216\n\t"
      "ds_read_b64_tr_b8 v[32:33], v14 offset:0\n\t"
      "ds_read_b64_tr_b8 v[34:35], v14 offset:1024\n\t"
      "ds_read_b64_tr_b8 v[36:37], v14 offset:8192\n\t"
      "ds_read_b64_tr_b8 v[38:39], v14 offset:9216\n\t"
      "ds_read_b64_tr_b8 v[40:41], v15 offset:0\n\t"
      "ds_read_b64_tr_b8 v[42:43], v15 offset:1024\n\t"
      "ds_read_b64_tr_b8 v[44:45], v15 offset:8192\n\t"
      "ds_read_b64_tr_b8 v[46:47], v15 offset:9216\n\t"
      "ds_read_b64_tr_b8 v[48:49], v16 offset:0\n\t"
      "ds_read_b64_tr_b8 v[50:51], v16 offset:1024\n\t"
      "ds_read_b64_tr_b8 v[52:53], v16 offset:8192\n\t"
      "ds_read_b64_tr_b8 v[54:55], v16 offset:9216\n\t"
      "s_add_u32 s82, s82, 1\n\t"
      "v_mov_b32 v21, s82\n\t"
      "ds_write_b32 v18, v21\n\t"
      "P1X_%=:\n\t"
      "s_waitcnt lgkmcnt(13)\n\t"
      "v_mfma_f32_16x16x128_f8f6f4 v[224:227], v[24:31], v[56:63], v[224:227] cbsz:1\n\t"
      "ds_read_b128 v[72:75], v7 offset:0\n\t"
      "ds_read_b128 v[76:79], v8 offset:0\n\t"
      "ds_read_b128 v[80:83], v7 offset:2048\n\t"
      "ds_read_b128 v[84:87], v8 offset:2048\n\t"
      "ds_read_b128 v[88:91], v7 offset:4096\n\t"
      "v_mfma_f32_16x16x128_f8f6f4 v[240:243], v[24:31], v[64:71], v[240:243] cbsz:1\n\t"
      "ds_read_b128 v[92:95], v8 offset:4096\n\t"
      "ds_read_b128 v[160:163], v7 offset:6144\n\t"
      "ds_read_b128 v[164:167], v8 offset:6144\n\t"
      "s_add_u32 s82, s82, 1\n\t"
      "v_mov_b32 v21, s82\n\t"
      "s_waitcnt lgkmcnt(15)\n\t"
      "v_mfma_f32_16x16x128_f8f6f4 v[228:231], v[32:39], v[56:63], v[228:231] cbsz:1\n\t"
      "ds_write_b32 v18, v21\n\t"
      "s_add_u32 s89, s86, 32768\n\t"
      "s_and_b32 s89, s89, 0x1ffff\n\t"
      "s_xor_b32 s90, s89, s86\n\t"
      "s_mov_b32 s86, s89\n\t"
      "v_xor_b32 v7, s90, v7\n\t"
      "v_xor_b32 v8, s90, v8\n\t"
      "v_mfma_f32_16x16x128_f8f6f4 v[244:247], v[32:39], v[64:71], v[244:247] cbsz:1\n\t"
      "s_add_u32 s89, s85, 32768\n\t"
      "s_and_b32 s89, s89, 0x1ffff\n\t"
      "s_xor_b32 s90, s89, s85\n\t"
      "ds_read_b64 v[22:23], v19\n\t"
      "s_mov_b32 s85, s89\n\t"
      "v_xor_b32 v13, s90, v13\n\t"
      "v_xor_b32 v14, s90, v14\n\t"
      "s_waitcnt lgkmcnt(15)\n\t"
      "v_mfma_f32_16x16x128_f8f6f4 v[232:235], v[40:47], v[56:63], v[232:235] cbsz:1\n\t"
      "v_xor_b32 v15, s90, v15\n\t"
      "v_xor_b32 v16, s90, v16\n\t"
      "s_add_u32 s89, s88, 0x2000\n\t"
      "s_cmp_eq_u32 s89, 0x6000\n\t"
      "s_cselect_b32 s89, 0, s89\n\t"
      "s_xor_b32 s90, s89, s88\n\t"
      "s_mov_b32 s88, s89\n\t"
      "v_mfma_f32_16x16x128_f8f6f4 v[248:251], v[40:47], v[64:71], v[248:251] cbsz:1\n\t"
      "v_xor_b32 v11, s90, v11\n\t"
      "v_xor_b32 v12, s90, v12\n\t"
      "s_add_u32 s80, s80, 1\n\t"
      "s_waitcnt lgkmcnt(11)\n\t"
      "v_mfma_f32_16x16x128_f8f6f4 v[236:239], v[48:55], v[56:63], v[236:239] cbsz:1\n\t"
      "s_waitcnt lgkmcnt(0)\n\t"
      "v_mfma_f32_16x16x128_f8f6f4 v[252:255], v[48:55], v[64:71], v[252:255] cbsz:1\n\t"
      "v_readfirstlane_b32 s84, v23\n\t"
      "v_readfirstlane_b32 s83, v22\n\t"
      "s_cmp_gt_u32 s84, s80\n\t"
      "s_cbranch_scc0 Z1X_%=\n\t"
      "s_cmp_gt_u32 s83, s81\n\t"
      "s_cbranch_scc0 S1X_%=\n\t"
      "v_mfma_f32_16x16x128_f8f6f4 v[96:99], v[24:31], v[72:79], v[96:99] cbsz:1\n\t"
      "ds_read_b128 v[56:59], v11 offset:0\n\t"
      "ds_read_b128 v[60:63], v12 offset:0\n\t"
      "v_mfma_f32_16x16x128_f8f6f4 v[112:115], v[24:31], v[80:87], v[112:115] cbsz:1\n\t"
      "ds_read_b128 v[64:67], v11 offset:4096\n\t"
      "ds_read_b128 v[68:71], v12 offset:4096\n\t"
      "v_mfma_f32_16x16x128_f8f6f4 v[128:131], v[24:31], v[88:95], v[128:131] cbsz:1\n\t"
      "v_mfma_f32_16x16x128_f8f6f4 v[144:147], v[24:31], v[160:167], v[144:147] cbsz:1\n\t"
      "ds_read_b64_tr_b8 v[24:25], v13 offset:0\n\t"
      "ds_read_b64_tr_b8 v[26:27], v13 offset:1024\n\t"
      "v_mfma_f32_16x16x128_f8f6f4 v[100:103], v[32:39], v[72:79], v[100:103] cbsz:1\n\t"
      "ds_read_b64_tr_b8 v[28:29], v13 offset:8192\n\t"
      "ds_read_b64_tr_b8 v[30:31], v13 offset:9216\n\t"
      "v_mfma_f32_16x16x128_f8f6f4 v[116:119], v[32:39], v[80:87], v[116:119] cbsz:1\n\t"
      "v_mfma_f32_16x16x128_f8f6f4 v[132:135], v[32:39], v[88:95], v[132:135] cbsz:1\n\t"
      "v_mfma_f32_16x16x128_f8f6f4 v[148:151], v[32:39], v[160:167], v[148:151] cbsz:1\n\t"
      "ds_read_b64_tr_b8 v[32:33], v14 offset:0\n\t"
      "ds_read_b64_tr_b8 v[34:35], v14 offset:1024\n\t"
      "v_mfma_f32_16x16x128_f8f6f4 v[104:107], v[40:47], v[72:79], v[104:107] cbsz:1\n\t"
      "ds_read_b64_tr_b8 v[36:37], v14 offset:8192\n\t"
      "ds_read_b64_tr_b8 v[38:39], v14 offset:9216\n\t"
      "v_mfma_f32_16x16x128_f8f6f4 v[120:123], v[40:47], v[80:87], v[120:123] cbsz:1\n\t"
      "v_mfma_f32_16x16x128_f8f6f4 v[136:139], v[40:47], v[88:95], v[136:139] cbsz:1\n\t"
      "v_mfma_f32_16x16x128_f8f6f4 v[152:155], v[40:47], v[160:167], v[152:155] cbsz:1\n\t"
      "ds_read_b64_tr_b8 v[40:41], v15 offset:0\n\t"
      "ds_read_b64_tr_b8 v[42:43], v15 offset:1024\n\t"
      "v_mfma_f32_16x16x128_f8f6f4 v[108:111], v[48:55], v[72:79], v[108:111] cbsz:1\n\t"
      "ds_read_b64_tr_b8 v[44:45], v15 offset:8192\n\t"
      "ds_read_b64_tr_b8 v[46:47], v15 offset:9216\n\t"
      "s_add_u32 s82, s82, 1\n\t"
      "v_mov_b32 v21, s82\n\t"
      "v_mfma_f32_16x16x128_f8f6f4 v[124:127], v[48:55], v[80:87], v[124:127] cbsz:1\n\t"
      "v_mfma_f32_16x16x128_f8f6f4 v[140:143], v[48:55], v[88:95], v[140:143] cbsz:1\n\t"
      "v_mfma_f32_16x16x128_f8f6f4 v[156:159], v[48:55], v[160:167], v[156:159] cbsz:1\n\t"
      "ds_read_b64_tr_b8 v[48:49], v16 offset:0\n\t"
      "ds_read_b64_tr_b8 v[50:51], v16 offset:1024\n\t"
      "ds_read_b64_tr_b8 v[52:53], v16 offset:8192\n\t"
      "ds_read_b64_tr_b8 v[54:55], v16 offset:9216\n\t"
      "ds_write_b32 v18, v21\n\t"
      "JOIN1X_%=:\n\t"
      "s_add_u32 s81, s81, 1\n\t"
      "s_cmp_lt_u32 s80, %[dt]\n\t"
      "s_cbranch_scc1 P1X_%=\n\t"
      "v_mov_b32 v160, 0\n\t"
      "v_mov_b32 v161, 0\n\t"
      "v_mov_b32 v162, 0\n\t"
      "v_mov_b32 v163, 0\n\t"
      "v_mov_b32 v164, 0\n\t"
      "v_mov_b32 v165, 0\n\t"
      "v_mov_b32 v166, 0\n\t"
      "v_mov_b32 v167, 0\n\t"
      "P2X_%=:\n\t"
      "s_waitcnt lgkmcnt(13)\n\t"
      "v_mfma_f32_16x16x128_f8f6f4 v[224:227], v[24:31], v[56:63], v[224:227] cbsz:1\n\t"
      "ds_read_b128 v[72:75], v7 offset:0\n\t"
      "ds_read_b128 v[76:79], v8 offset:0\n\t"
      "v_mfma_f32_16x16x128_f8f6f4 v[240:243], v[24:31], v[64:71], v[240:243] cbsz:1\n\t"
      "ds_read_b128 v[80:83], v7 offset:2048\n\t"
      "ds_read_b128 v[84:87], v8 offset:2048\n\t"
      "s_waitcnt lgkmcnt(13)\n\t"
      "v_mfma_f32_16x16x128_f8f6f4 v[228:231], v[32:39], v[56:63], v[228:231] cbsz:1\n\t"
      "ds_read_b128 v[88:91], v7 offset:4096\n\t"
      "ds_read_b128 v[92:95], v8 offset:4096\n\t"
      "v_mfma_f32_16x16x128_f8f6f4 v[244:247], v[32:39], v[64:71], v[244:247] cbsz:1\n\t"
      "ds_read_b64 v[22:23], v19\n\t"
      "s_waitcnt lgkmcnt(12)\n\t"
      "v_mfma_f32_16x16x128_f8f6f4 v[232:235], v[40:47], v[56:63], v[232:235] cbsz:1\n\t"
      "v_mfma_f32_16x16x128_f8f6f4 v[248:251], v[40:47], v[64:71], v[248:251] cbsz:1\n\t"
      "s_waitcnt lgkmcnt(8)\n\t"
      "v_mfma_f32_16x16x128_f8f6f4 v[236:239], v[48:55], v[56:63], v[236:239] cbsz:1\n\t"
      "v_mfma_f32_16x16x128_f8f6f4 v[252:255], v[48:55], v[64:71], v[252:255] cbsz:1\n\t"
      "s_waitcnt lgkmcnt(5)\n\t"
      "v_mfma_f32_16x16x128_f8f6f4 v[96:99], v[24:31], v[72:79], v[96:99] cbsz:1\n\t"
      "ds_read_b128 v[56:59], v7 offset:6144\n\t"
      "ds_read_b128 v[60:63], v8 offset:6144\n\t"
      "s_add_u32 s82, s82, 1\n\t"
      "s_waitcnt lgkmcnt(5)\n\t"
      "v_mfma_f32_16x16x128_f8f6f4 v[112:115], v[24:31], v[80:87], v[112:115] cbsz:1\n\t"
      "v_mov_b32 v21, s82\n\t"
      "ds_write_b32 v18, v21\n\t"
      "s_add_u32 s89, s86, 49152\n\t"
      "s_waitcnt lgkmcnt(4)\n\t"
      "v_mfma_f32_16x16x128_f8f6f4 v[128:131], v[24:31], v[88:95], v[128:131] cbsz:1\n\t"
      "s_and_b32 s89, s89, 0x1ffff\n\t"
      "s_xor_b32 s90, s89, s86\n\t"
      "s_mov_b32 s86, s89\n\t"
      "v_xor_b32 v7, s90, v7\n\t"
      "v_mfma_f32_16x16x128_f8f6f4 v[100:103], v[32:39], v[72:79], v[100:103] cbsz:1\n\t"
      "v_xor_b32 v8, s90, v8\n\t"
      "s_waitcnt lgkmcnt(3)\n\t"
      "v_readfirstlane_b32 s83, v22\n\t"
      "s_cmp_gt_u32 s83, s81\n\t"
      "s_cbranch_scc0 SPINL2X_%=\n\t"
      "BACKL2X_%=:\n\t"
      "s_add_u32 s81, s81, 1\n\t"
      "ds_read_b128 v[64:67], v9 offset:0\n\t"
      "v_mfma_f32_16x16x128_f8f6f4 v[116:119], v[32:39], v[80:87], v[116:119] cbsz:1\n\t"
      "ds_read_b128 v[68:71], v10 offset:0\n\t"
      "s_add_u32 s89, s85, 49152\n\t"
      "s_and_b32 s89, s89, 0x1ffff\n\t"
      "s_xor_b32 s90, s89, s85\n\t"
      "v_mfma_f32_16x16x128_f8f6f4 v[132:135], v[32:39], v[88:95], v[132:135] cbsz:1\n\t"
      "s_mov_b32 s85, s89\n\t"
      "v_xor_b32 v13, s90, v13\n\t"
      "v_xor_b32 v14, s90, v14\n\t"
      "v_mfma_f32_16x16x128_f8f6f4 v[104:107], v[40:47], v[72:79], v[104:107] cbsz:1\n\t"
      "v_xor_b32 v15, s90, v15\n\t"
      "v_xor_b32 v16, s90, v16\n\t"
      "s_add_u32 s89, s88, 0x2000\n\t"
      "v_mfma_f32_16x16x128_f8f6f4 v[120:123], v[40:47], v[80:87], v[120:123] cbsz:1\n\t"
      "s_cmp_eq_u32 s89, 0x6000\n\t"
      "s_cselect_b32 s89, 0, s89\n\t"
      "s_xor_b32 s90, s89, s88\n\t"
      "s_mov_b32 s88, s89\n\t"
      "v_xor_b32 v11, s90, v11\n\t"
      "v_mfma_f32_16x16x128_f8f6f4 v[136:139], v[40:47], v[88:95], v[136:139] cbsz:1\n\t"
      "v_xor_b32 v12, s90, v12\n\t"
      "s_add_u32 s80, s80, 1\n\t"
      "v_mfma_f32_16x16x128_f8f6f4 v[108:111], v[48:55], v[72:79], v[108:111] cbsz:1\n\t"
      "v_mfma_f32_16x16x128_f8f6f4 v[124:127], v[48:55], v[80:87], v[124:127] cbsz:1\n\t"
      "v_mfma_f32_16x16x128_f8f6f4 v[140:143], v[48:55], v[88:95], v[140:143] cbsz:1\n\t"
      "s_waitcnt lgkmcnt(3)\n\t"
      "v_mfma_f32_16x16x128_f8f6f4 v[144:147], v[24:31], v[56:63], v[144:147] cbsz:1\n\t"
      "ds_read_b128 v[72:75], v9 offset:2048\n\t"
      "ds_read_b128 v[76:79], v10 offset:2048\n\t"
      "ds_read_b128 v[80:83], v9 offset:4096\n\t"
      "s_waitcnt lgkmcnt(3)\n\t"
      "v_mfma_f32_16x16x128_f8f6f4 v[160:163], v[24:31], v[64:71], v[160:163] cbsz:1\n\t"
      "ds_read_b128 v[84:87], v10 offset:4096\n\t"
      "ds_read_b128 v[88:91], v9 offset:6144\n\t"
      "ds_read_b128 v[92:95], v10 offset:6144\n\t"
      "v_mfma_f32_16x16x128_f8f6f4 v[148:151], v[32:39], v[56:63], v[148:151] cbsz:1\n\t"
      "s_add_u32 s82, s82, 1\n\t"
      "v_mov_b32 v21, s82\n\t"
      "ds_write_b32 v18, v21\n\t"
      "v_mfma_f32_16x16x128_f8f6f4 v[164:167], v[32:39], v[64:71], v[164:167] cbsz:1\n\t"
      "ds_read_b64 v[22:23], v19\n\t"
      "s_add_u32 s89, s87, 49152\n\t"
      "s_and_b32 s89, s89, 0x1ffff\n\t"
      "s_xor_b32 s90, s89, s87\n\t"
      "v_mfma_f32_16x16x128_f8f6f4 v[152:155], v[40:47], v[56:63], v[152:155] cbsz:1\n\t"
      "s_mov_b32 s87, s89\n\t"
      "v_xor_b32 v9, s90, v9\n\t"
      "v_xor_b32 v10, s90, v10\n\t"
      "v_mfma_f32_16x16x128_f8f6f4 v[168:171], v[40:47], v[64:71], v[168:171] cbsz:1\n\t"
      "v_mfma_f32_16x16x128_f8f6f4 v[156:159], v[48:55], v[56:63], v[156:159] cbsz:1\n\t"
      "s_waitcnt lgkmcnt(0)\n\t"
      "v_mfma_f32_16x16x128_f8f6f4 v[172:175], v[48:55], v[64:71], v[172:175] cbsz:1\n\t"
      "v_readfirstlane_b32 s84, v23\n\t"
      "v_readfirstlane_b32 s83, v22\n\t"
      "s_cmp_gt_u32 s84, s80\n\t"
      "s_cbranch_scc0 Z2X_%=\n\t"
      "s_cmp_gt_u32 s83, s81\n\t"
      "s_cbranch_scc0 S2X_%=\n\t"
      "v_mfma_f32_16x16x128_f8f6f4 v[176:179], v[24:31], v[72:79], v[176:179] cbsz:1\n\t"
      "ds_read_b128 v[56:59], v11 offset:0\n\t"
      "ds_read_b128 v[60:63], v12 offset:0\n\t"
      "v_mfma_f32_16x16x128_f8f6f4 v[192:195], v[24:31], v[80:87], v[192:195] cbsz:1\n\t"
      "ds_read_b128 v[64:67], v11 offset:4096\n\t"
      "ds_read_b128 v[68:71], v12 offset:4096\n\t"
      "v_mfma_f32_16x16x128_f8f6f4 v[208:211], v[24:31], v[88:95], v[208:211] cbsz:1\n\t"
      "ds_read_b64_tr_b8 v[24:25], v13 offset:0\n\t"
      "ds_read_b64_tr_b8 v[26:27], v13 offset:1024\n\t"
      "v_mfma_f32_16x16x128_f8f6f4 v[180:183], v[32:39], v[72:79], v[180:183] cbsz:1\n\t"
      "ds_read_b64_tr_b8 v[28:29], v13 offset:8192\n\t"
      "ds_read_b64_tr_b8 v[30:31], v13 offset:9216\n\t"
      "v_mfma_f32_16x16x128_f8f6f4 v[196:199], v[32:39], v[80:87], v[196:199] cbsz:1\n\t"
      "v_mfma_f32_16x16x128_f8f6f4 v[212:215], v[32:39], v[88:95], v[212:215] cbsz:1\n\t"
      "ds_read_b64_tr_b8 v[32:33], v14 offset:0\n\t"
      "ds_read_b64_tr_b8 v[34:35], v14 offset:1024\n\t"
      "v_mfma_f32_16x16x128_f8f6f4 v[184:187], v[40:47], v[72:79], v[184:187] cbsz:1\n\t"
      "ds_read_b64_tr_b8 v[36:37], v14 offset:8192\n\t"
      "ds_read_b64_tr_b8 v[38:39], v14 offset:9216\n\t"
      "v_mfma_f32_16x16x128_f8f6f4 v[200:203], v[40:47], v[80:87], v[200:203] cbsz:1\n\t"
      "v_mfma_f32_16x16x128_f8f6f4 v[216:219], v[40:47], v[88:95], v[216:219] cbsz:1\n\t"
      "ds_read_b64_tr_b8 v[40:41], v15 offset:0\n\t"
      "ds_read_b64_tr_b8 v[42:43], v15 offset:1024\n\t"
      "v_mfma_f32_16x16x128_f8f6f4 v[188:191], v[48:55], v[72:79], v[188:191] cbsz:1\n\t"
      "ds_read_b64_tr_b8 v[44:45], v15 offset:8192\n\t"
      "ds_read_b64_tr_b8 v[46:47], v15 offset:9216\n\t"
      "s_add_u32 s82, s82, 1\n\t"
      "v_mov_b32 v21, s82\n\t"
      "v_mfma_f32_16x16x128_f8f6f4 v[204:207], v[48:55], v[80:87], v[204:207] cbsz:1\n\t"
      "v_mfma_f32_16x16x128_f8f6f4 v[220:223], v[48:55], v[88:95], v[220:223] cbsz:1\n\t"
      "ds_read_b64_tr_b8 v[48:49], v16 offset:0\n\t"
      "ds_read_b64_tr_b8 v[50:51], v16 offset:1024\n\t"
      "ds_read_b64_tr_b8 v[52:53], v16 offset:8192\n\t"
      "ds_read_b64_tr_b8 v[54:55], v16 offset:9216\n\t"
      "ds_write_b32 v18, v21\n\t"
      "JOIN2X_%=:\n\t"
      "s_add_u32 s81, s81, 1\n\t"
      "s_cmp_lt_u32 s80, 31\n\t"
      "s_cbranch_scc1 P2X_%=\n\t"
      "s_waitcnt lgkmcnt(13)\n\t"
      "v_mfma_f32_16x16x128_f8f6f4 v[224:227], v[24:31], v[56:63], v[224:227] cbsz:1\n\t"
      "ds_read_b128 v[72:75], v7 offset:0\n\t"
      "ds_read_b128 v[76:79], v8 offset:0\n\t"
      "v_mfma_f32_16x16x128_f8f6f4 v[240:243], v[24:31], v[64:71], v[240:243] cbsz:1\n\t"
      "ds_read_b128 v[80:83], v7 offset:2048\n\t"
      "ds_read_b128 v[84:87], v8 offset:2048\n\t"
      "s_waitcnt lgkmcnt(13)\n\t"
      "v_mfma_f32_16x16x128_f8f6f4 v[228:231], v[32:39], v[56:63], v[228:231] cbsz:1\n\t"
      "ds_read_b128 v[88:91], v7 offset:4096\n\t"
      "ds_read_b128 v[92:95], v8 offset:4096\n\t"
      "v_mfma_f32_16x16x128_f8f6f4 v[244:247], v[32:39], v[64:71], v[244:247] cbsz:1\n\t"
      "ds_read_b64 v[22:23], v19\n\t"
      "s_waitcnt lgkmcnt(12)\n\t"
      "v_mfma_f32_16x16x128_f8f6f4 v[232:235], v[40:47], v[56:63], v[232:235] cbsz:1\n\t"
      "v_mfma_f32_16x16x128_f8f6f4 v[248:251], v[40:47], v[64:71], v[248:251] cbsz:1\n\t"
      "s_waitcnt lgkmcnt(8)\n\t"
      "v_mfma_f32_16x16x128_f8f6f4 v[236:239], v[48:55], v[56:63], v[236:239] cbsz:1\n\t"
      "v_mfma_f32_16x16x128_f8f6f4 v[252:255], v[48:55], v[64:71], v[252:255] cbsz:1\n\t"
      "s_waitcnt lgkmcnt(5)\n\t"
      "v_mfma_f32_16x16x128_f8f6f4 v[96:99], v[24:31], v[72:79], v[96:99] cbsz:1\n\t"
      "ds_read_b128 v[56:59], v7 offset:6144\n\t"
      "ds_read_b128 v[60:63], v8 offset:6144\n\t"
      "s_waitcnt lgkmcnt(5)\n\t"
      "v_mfma_f32_16x16x128_f8f6f4 v[112:115], v[24:31], v[80:87], v[112:115] cbsz:1\n\t"
      "s_add_u32 s82, s82, 1\n\t"
      "v_mov_b32 v21, s82\n\t"
      "ds_write_b32 v18, v21\n\t"
      "s_waitcnt lgkmcnt(4)\n\t"
      "v_mfma_f32_16x16x128_f8f6f4 v[128:131], v[24:31], v[88:95], v[128:131] cbsz:1\n\t"
      "s_waitcnt lgkmcnt(3)\n\t"
      "v_readfirstlane_b32 s83, v22\n\t"
      "s_cmp_gt_u32 s83, s81\n\t"
      "s_cbranch_scc0 SPINL2LX_%=\n\t"
      "BACKL2LX_%=:\n\t"
      "s_add_u32 s81, s81, 1\n\t"
      "ds_read_b128 v[64:67], v9 offset:0\n\t"
      "v_mfma_f32_16x16x128_f8f6f4 v[100:103], v[32:39], v[72:79], v[100:103] cbsz:1\n\t"
      "ds_read_b128 v[68:71], v10 offset:0\n\t"
      "s_add_u32 s89, s85, 49152\n\t"
      "s_and_b32 s89, s89, 0x1ffff\n\t"
      "v_mfma_f32_16x16x128_f8f6f4 v[116:119], v[32:39], v[80:87], v[116:119] cbsz:1\n\t"
      "s_xor_b32 s90, s89, s85\n\t"
      "s_mov_b32 s85, s89\n\t"
      "v_xor_b32 v13, s90, v13\n\t"
      "v_mfma_f32_16x16x128_f8f6f4 v[132:135], v[32:39], v[88:95], v[132:135] cbsz:1\n\t"
      "v_xor_b32 v14, s90, v14\n\t"
      "v_xor_b32 v15, s90, v15\n\t"
      "v_mfma_f32_16x16x128_f8f6f4 v[104:107], v[40:47], v[72:79], v[104:107] cbsz:1\n\t"
      "v_xor_b32 v16, s90, v16\n\t"
      "s_add_u32 s89, s88, 0x2000\n\t"
      "s_cmp_eq_u32 s89, 0x6000\n\t"
      "v_mfma_f32_16x16x128_f8f6f4 v[120:123], v[40:47], v[80:87], v[120:123] cbsz:1\n\t"
      "s_cselect_b32 s89, 0, s89\n\t"
      "s_xor_b32 s90, s89, s88\n\t"
      "s_mov_b32 s88, s89\n\t"
      "v_xor_b32 v11, s90, v11\n\t"
      "v_mfma_f32_16x16x128_f8f6f4 v[136:139], v[40:47], v[88:95], v[136:139] cbsz:1\n\t"
      "v_xor_b32 v12, s90, v12\n\t"
      "s_add_u32 s80, s80, 1\n\t"
      "v_mfma_f32_16x16x128_f8f6f4 v[108:111], v[48:55], v[72:79], v[108:111] cbsz:1\n\t"
      "v_mfma_f32_16x16x128_f8f6f4 v[124:127], v[48:55], v[80:87], v[124:127] cbsz:1\n\t"
      "v_mfma_f32_16x16x128_f8f6f4 v[140:143], v[48:55], v[88:95], v[140:143] cbsz:1\n\t"
      "s_waitcnt lgkmcnt(3)\n\t"
      "v_mfma_f32_16x16x128_f8f6f4 v[144:147], v[24:31], v[56:63], v[144:147] cbsz:1\n\t"
      "ds_read_b128 v[72:75], v9 offset:2048\n\t"
      "ds_read_b128 v[76:79], v10 offset:2048\n\t"
      "ds_read_b128 v[80:83], v9 offset:4096\n\t"
      "s_waitcnt lgkmcnt(3)\n\t"
      "v_mfma_f32_16x16x128_f8f6f4 v[160:163], v[24:31], v[64:71], v[160:163] cbsz:1\n\t"
      "ds_read_b128 v[84:87], v10 offset:4096\n\t"
      "ds_read_b128 v[88:91], v9 offset:6144\n\t"
      "ds_read_b128 v[92:95], v10 offset:6144\n\t"
      "v_mfma_f32_16x16x128_f8f6f4 v[148:151], v[32:39], v[56:63], v[148:151] cbsz:1\n\t"
      "s_add_u32 s82, s82, 1\n\t"
      "v_mov_b32 v21, s82\n\t"
      "ds_write_b32 v18, v21\n\t"
      "v_mfma_f32_16x16x128_f8f6f4 v[164:167], v[32:39], v[64:71], v[164:167] cbsz:1\n\t"
      "ds_read_b64 v[22:23], v19\n\t"
      "s_add_u32 s89, s87, 32768\n\t"
      "s_and_b32 s89, s89, 0x1ffff\n\t"
      "s_xor_b32 s90, s89, s87\n\t"
      "v_mfma_f32_16x16x128_f8f6f4 v[152:155], v[40:47], v[56:63], v[152:155] cbsz:1\n\t"
      "s_mov_b32 s87, s89\n\t"
      "v_xor_b32 v9, s90, v9\n\t"
      "v_xor_b32 v10, s90, v10\n\t"
      "v_mfma_f32_16x16x128_f8f6f4 v[168:171], v[40:47], v[64:71], v[168:171] cbsz:1\n\t"
      "v_mfma_f32_16x16x128_f8f6f4 v[156:159], v[48:55], v[56:63], v[156:159] cbsz:1\n\t"
      "s_waitcnt lgkmcnt(0)\n\t"
      "v_mfma_f32_16x16x128_f8f6f4 v[172:175], v[48:55], v[64:71], v[172:175] cbsz:1\n\t"
      "v_readfirstlane_b32 s84, v23\n\t"
      "v_readfirstlane_b32 s83, v22\n\t"
      "s_cmp_gt_u32 s84, s80\n\t"
      "s_cbranch_scc0 Z2LX_%=\n\t"
      "s_cmp_gt_u32 s83, s81\n\t"
      "s_cbranch_scc0 S2LX_%=\n\t"
      "v_mfma_f32_16x16x128_f8f6f4 v[176:179], v[24:31], v[72:79], v[176:179] cbsz:1\n\t"
      "ds_read_b128 v[56:59], v9 offset:0\n\t"
      "ds_read_b128 v[60:63], v10 offset:0\n\t"
      "v_mfma_f32_16x16x128_f8f6f4 v[192:195], v[24:31], v[80:87], v[192:195] cbsz:1\n\t"
      "ds_read_b128 v[64:67], v9 offset:2048\n\t"
      "ds_read_b128 v[68:71], v10 offset:2048\n\t"
      "v_mfma_f32_16x16x128_f8f6f4 v[208:211], v[24:31], v[88:95], v[208:211] cbsz:1\n\t"
      "ds_read_b64_tr_b8 v[24:25], v13 offset:0\n\t"
      "ds_read_b64_tr_b8 v[26:27], v13 offset:1024\n\t"
      "v_mfma_f32_16x16x128_f8f6f4 v[180:183], v[32:39], v[72:79], v[180:183] cbsz:1\n\t"
      "ds_read_b64_tr_b8 v[28:29], v13 offset:8192\n\t"
      "ds_read_b64_tr_b8 v[30:31], v13 offset:9216\n\t"
      "v_mfma_f32_16x16x128_f8f6f4 v[196:199], v[32:39], v[80:87], v[196:199] cbsz:1\n\t"
      "v_mfma_f32_16x16x128_f8f6f4 v[212:215], v[32:39], v[88:95], v[212:215] cbsz:1\n\t"
      "ds_read_b64_tr_b8 v[32:33], v14 offset:0\n\t"
      "ds_read_b64_tr_b8 v[34:35], v14 offset:1024\n\t"
      "v_mfma_f32_16x16x128_f8f6f4 v[184:187], v[40:47], v[72:79], v[184:187] cbsz:1\n\t"
      "ds_read_b64_tr_b8 v[36:37], v14 offset:8192\n\t"
      "ds_read_b64_tr_b8 v[38:39], v14 offset:9216\n\t"
      "v_mfma_f32_16x16x128_f8f6f4 v[200:203], v[40:47], v[80:87], v[200:203] cbsz:1\n\t"
      "v_mfma_f32_16x16x128_f8f6f4 v[216:219], v[40:47], v[88:95], v[216:219] cbsz:1\n\t"
      "ds_read_b64_tr_b8 v[40:41], v15 offset:0\n\t"
      "ds_read_b64_tr_b8 v[42:43], v15 offset:1024\n\t"
      "v_mfma_f32_16x16x128_f8f6f4 v[188:191], v[48:55], v[72:79], v[188:191] cbsz:1\n\t"
      "ds_read_b64_tr_b8 v[44:45], v15 offset:8192\n\t"
      "ds_read_b64_tr_b8 v[46:47], v15 offset:9216\n\t"
      "s_add_u32 s82, s82, 1\n\t"
      "v_mov_b32 v21, s82\n\t"
      "v_mfma_f32_16x16x128_f8f6f4 v[204:207], v[48:55], v[80:87], v[204:207] cbsz:1\n\t"
      "v_mfma_f32_16x16x128_f8f6f4 v[220:223], v[48:55], v[88:95], v[220:223] cbsz:1\n\t"
      "ds_read_b64_tr_b8 v[48:49], v16 offset:0\n\t"
      "ds_read_b64_tr_b8 v[50:51], v16 offset:1024\n\t"
      "ds_read_b64_tr_b8 v[52:53], v16 offset:8192\n\t"
      "ds_read_b64_tr_b8 v[54:55], v16 offset:9216\n\t"
      "ds_write_b32 v18, v21\n\t"
      "JOIN2LX_%=:\n\t"
      "s_add_u32 s81, s81, 1\n\t"
      "EWX_%=:\n\t"
      "ds_read_b128 v[72:75], v19 offset:16\n\t"
      "ds_read_b32 v22, v19 offset:36\n\t"
      "s_waitcnt lgkmcnt(0)\n\t"
      "v_readfirstlane_b32 s89, v22\n\t"
      "s_cmp_lg_u32 s89, 0\n\t"
      "s_cselect_b32 s89, 0x7fc00000, %[sscale]\n\t"
      "v_mov_b32 v20, s89\n\t"
      "v_min_u32 v72, v72, v73\n\t"
      "v_min3_u32 v72, v72, v74, v75\n\t"
      "v_lshl_or_b32 v72, v22, 8, v72\n\t"
      "s_nop 0\n\t"
      "v_readfirstlane_b32 s89, v72\n\t"
      "s_cmp_gt_u32 s89, s95\n\t"
      "s_cbranch_scc0 EWX_%=\n\t"
      "v_pk_mul_f32 v[96:97], v[96:97], v[20:21] op_sel_hi:[1,0]\n\t"
      "v_pk_mul_f32 v[98:99], v[98:99], v[20:21] op_sel_hi:[1,0]\n\t"
      "v_pk_mul_f32 v[100:101], v[100:101], v[20:21] op_sel_hi:[1,0]\n\t"
      "v_pk_mul_f32 v[102:103], v[102:103], v[20:21] op_sel_hi:[1,0]\n\t"
      "ds_write_b128 %[vsw0], v[96:99]\n\t"
      "ds_write_b128 v17, v[100:103]\n\t"
      "ds_read_b128 v[96:99], %[vsr]\n\t"
      "ds_read_b128 v[100:103], %[vsr] offset:1024\n\t"
      "v_pk_mul_f32 v[104:105], v[104:105], v[20:21] op_sel_hi:[1,0]\n\t"
      "v_pk_mul_f32 v[106:107], v[106:107], v[20:21] op_sel_hi:[1,0]\n\t"
      "v_pk_mul_f32 v[108:109], v[108:109], v[20:21] op_sel_hi:[1,0]\n\t"
      "v_pk_mul_f32 v[110:111], v[110:111], v[20:21] op_sel_hi:[1,0]\n\t"
      "ds_write_b128 %[vsw0], v[104:107]\n\t"
      "ds_write_b128 v17, v[108:111]\n\t"
      "ds_read_b128 v[104:107], %[vsr]\n\t"
      "ds_read_b128 v[108:111], %[vsr] offset:1024\n\t"
      "v_pk_mul_f32 v[112:113], v[112:113], v[20:21] op_sel_hi:[1,0]\n\t"
      "v_pk_mul_f32 v[114:115], v[114:115], v[20:21] op_sel_hi:[1,0]\n\t"
      "v_pk_mul_f32 v[116:117], v[116:117], v[20:21] op_sel_hi:[1,0]\n\t"
      "v_pk_mul_f32 v[118:119], v[118:119], v[20:21] op_sel_hi:[1,0]\n\t"
      "ds_write_b128 %[vsw0], v[112:115]\n\t"
      "ds_write_b128 v17, v[116:119]\n\t"
      "ds_read_b128 v[112:115], %[vsr]\n\t"
      "ds_read_b128 v[116:119], %[vsr] offset:1024\n\t"
      "s_waitcnt lgkmcnt(8)\n\t"
      "s_add_u32 s93, %[sout], 0\n\t"
      "s_add_u32 s92, s93, 0x10000\n\t"
      "buffer_store_dwordx4 v[96:99], %[vout], %[ro], s93 offen offset:0 sc1\n\t"
      "buffer_store_dwordx4 v[100:103], %[vout], %[ro], s92 offen offset:0 sc1\n\t"
      "v_pk_mul_f32 v[120:121], v[120:121], v[20:21] op_sel_hi:[1,0]\n\t"
      "v_pk_mul_f32 v[122:123], v[122:123], v[20:21] op_sel_hi:[1,0]\n\t"
      "v_pk_mul_f32 v[124:125], v[124:125], v[20:21] op_sel_hi:[1,0]\n\t"
      "v_pk_mul_f32 v[126:127], v[126:127], v[20:21] op_sel_hi:[1,0]\n\t"
      "ds_write_b128 %[vsw0], v[120:123]\n\t"
      "ds_write_b128 v17, v[124:127]\n\t"
      "ds_read_b128 v[120:123], %[vsr]\n\t"
      "ds_read_b128 v[124:127], %[vsr] offset:1024\n\t"
      "s_waitcnt lgkmcnt(8)\n\t"
      "buffer_store_dwordx4 v[104:107], %[vout], %[ro], s93 offen offset:128 sc1\n\t"
      "buffer_store_dwordx4 v[108:111], %[vout], %[ro], s92 offen offset:128 sc1\n\t"
      "v_pk_mul_f32 v[128:129], v[128:129], v[20:21] op_sel_hi:[1,0]\n\t"
      "v_pk_mul_f32 v[130:131], v[130:131], v[20:21] op_sel_hi:[1,0]\n\t"
      "v_pk_mul_f32 v[132:133], v[132:133], v[20:21] op_sel_hi:[1,0]\n\t"
      "v_pk_mul_f32 v[134:135], v[134:135], v[20:21] op_sel_hi:[1,0]\n\t"
      "ds_write_b128 %[vsw0], v[128:131]\n\t"
      "ds_write_b128 v17, v[132:135]\n\t"
      "ds_read_b128 v[128:131], %[vsr]\n\t"
      "ds_read_b128 v[132:135], %[vsr] offset:1024\n\t"
      "s_waitcnt lgkmcnt(8)\n\t"
      "s_add_u32 s93, %[sout], 131072\n\t"
      "s_add_u32 s92, s93, 0x10000\n\t"
      "buffer_store_dwordx4 v[112:115], %[vout], %[ro], s93 offen offset:0 sc1\n\t"
      "buffer_store_dwordx4 v[116:119], %[vout], %[ro], s92 offen offset:0 sc1\n\t"
      "v_pk_mul_f32 v[136:137], v[136:137], v[20:21] op_sel_hi:[1,0]\n\t"
      "v_pk_mul_f32 v[138:139], v[138:139], v[20:21] op_sel_hi:[1,0]\n\t"
      "v_pk_mul_f32 v[140:141], v[140:141], v[20:21] op_sel_hi:[1,0]\n\t"
      "v_pk_mul_f32 v[142:143], v[142:143], v[20:21] op_sel_hi:[1,0]\n\t"
      "ds_write_b128 %[vsw0], v[136:139]\n\t"
      "ds_write_b128 v17, v[140:143]\n\t"
      "ds_read_b128 v[136:139], %[vsr]\n\t"
      "ds_read_b128 v[140:143], %[vsr] offset:1024\n\t"
      "s_waitcnt lgkmcnt(8)\n\t"
      "buffer_store_dwordx4 v[120:123], %[vout], %[ro], s93 offen offset:128 sc1\n\t"
      "buffer_store_dwordx4 v[124:127], %[vout], %[ro], s92 offen offset:128 sc1\n\t"
      "v_pk_mul_f32 v[144:145], v[144:145], v[20:21] op_sel_hi:[1,0]\n\t"
      "v_pk_mul_f32 v[146:147], v[146:147], v[20:21] op_sel_hi:[1,0]\n\t"
      "v_pk_mul_f32 v[148:149], v[148:149], v[20:21] op_sel_hi:[1,0]\n\t"
      "v_pk_mul_f32 v[150:151], v[150:151], v[20:21] op_sel_hi:[1,0]\n\t"
      "ds_write_b128 %[vsw0], v[144:147]\n\t"
      "ds_write_b128 v17, v[148:151]\n\t"
      "ds_read_b128 v[144:147], %[vsr]\n\t"
      "ds_read_b128 v[148:151], %[vsr] offset:1024\n\t"
      "s_waitcnt lgkmcnt(8)\n\t"
      "s_add_u32 s93, %[sout], 262144\n\t"
      "s_add_u32 s92, s93, 0x10000\n\t"
      "buffer_store_dwordx4 v[128:131], %[vout], %[ro], s93 offen offset:0 sc1\n\t"
      "buffer_store_dwordx4 v[132:135], %[vout], %[ro], s92 offen offset:0 sc1\n\t"
      "v_pk_mul_f32 v[152:153], v[152:153], v[20:21] op_sel_hi:[1,0]\n\t"
      "v_pk_mul_f32 v[154:155], v[154:155], v[20:21] op_sel_hi:[1,0]\n\t"
      "v_pk_mul_f32 v[156:157], v[156:157], v[20:21] op_sel_hi:[1,0]\n\t"
      "v_pk_mul_f32 v[158:159], v[158:159], v[20:21] op_sel_hi:[1,0]\n\t"
      "ds_write_b128 %[vsw0], v[152:155]\n\t"
      "ds_write_b128 v17, v[156:159]\n\t"
      "ds_read_b128 v[152:155], %[vsr]\n\t"
      "ds_read_b128 v[156:159], %[vsr] offset:1024\n\t"
      "s_waitcnt lgkmcnt(8)\n\t"
      "buffer_store_dwordx4 v[136:139], %[vout], %[ro], s93 offen offset:128 sc1\n\t"
      "buffer_store_dwordx4 v[140:143], %[vout], %[ro], s92 offen offset:128 sc1\n\t"
      "v_pk_mul_f32 v[224:225], v[224:225], v[20:21] op_sel_hi:[1,0]\n\t"
      "v_pk_mul_f32 v[226:227], v[226:227], v[20:21] op_sel_hi:[1,0]\n\t"
      "v_pk_mul_f32 v[228:229], v[228:229], v[20:21] op_sel_hi:[1,0]\n\t"
      "v_pk_mul_f32 v[230:231], v[230:231], v[20:21] op_sel_hi:[1,0]\n\t"
      "ds_write_b128 %[vsw0], v[224:227]\n\t"
      "ds_write_b128 v17, v[228:231]\n\t"
      "ds_read_b128 v[224:227], %[vsr]\n\t"
      "ds_read_b128 v[228:231], %[vsr] offset:1024\n\t"
      "s_waitcnt lgkmcnt(8)\n\t"
      "s_add_u32 s93, %[sout], 393216\n\t"
      "s_add_u32 s92, s93, 0x10000\n\t"
      "buffer_store_dwordx4 v[144:147], %[vout], %[ro], s93 offen offset:0 sc1\n\t"
      "buffer_store_dwordx4 v[148:151], %[vout], %[ro], s92 offen offset:0 sc1\n\t"
      "v_pk_mul_f32 v[232:233], v[232:233], v[20:21] op_sel_hi:[1,0]\n\t"
      "v_pk_mul_f32 v[234:235], v[234:235], v[20:21] op_sel_hi:[1,0]\n\t"
      "v_pk_mul_f32 v[236:237], v[236:237], v[20:21] op_sel_hi:[1,0]\n\t"
      "v_pk_mul_f32 v[238:239], v[238:239], v[20:21] op_sel_hi:[1,0]\n\t"
      "ds_write_b128 %[vsw0], v[232:235]\n\t"
      "ds_write_b128 v17, v[236:239]\n\t"
      "ds_read_b128 v[232:235], %[vsr]\n\t"
      "ds_read_b128 v[236:239], %[vsr] offset:1024\n\t"
      "s_waitcnt lgkmcnt(8)\n\t"
      "buffer_store_dwordx4 v[152:155], %[vout], %[ro], s93 offen offset:128 sc1\n\t"
      "buffer_store_dwordx4 v[156:159], %[vout], %[ro], s92 offen offset:128 sc1\n\t"
      "v_pk_mul_f32 v[240:241], v[240:241], v[20:21] op_sel_hi:[1,0]\n\t"
      "v_pk_mul_f32 v[242:243], v[242:243], v[20:21] op_sel_hi:[1,0]\n\t"
      "v_pk_mul_f32 v[244:245], v[244:245], v[20:21] op_sel_hi:[1,0]\n\t"
      "v_pk_mul_f32 v[246:247], v[246:247], v[20:21] op_sel_hi:[1,0]\n\t"
      "ds_write_b128 %[vsw0], v[240:243]\n\t"
      "ds_write_b128 v17, v[244:247]\n\t"
      "ds_read_b128 v[240:243], %[vsr]\n\t"
      "ds_read_b128 v[244:247], %[vsr] offset:1024\n\t"
      "s_waitcnt lgkmcnt(8)\n\t"
      "s_mov_b32 s93, %[soute]\n\t"
      "s_add_u32 s92, s93, 0x10000\n\t"
      "buffer_store_dwordx4 v[224:227], %[vout], %[ro], s93 offen offset:0 sc1\n\t"
      "buffer_store_dwordx4 v[228:231], %[vout], %[ro], s92 offen offset:0 sc1\n\t"
      "v_pk_mul_f32 v[248:249], v[248:249], v[20:21] op_sel_hi:[1,0]\n\t"
      "v_pk_mul_f32 v[250:251], v[250:251], v[20:21] op_sel_hi:[1,0]\n\t"
      "v_pk_mul_f32 v[252:253], v[252:253], v[20:21] op_sel_hi:[1,0]\n\t"
      "v_pk_mul_f32 v[254:255], v[254:255], v[20:21] op_sel_hi:[1,0]\n\t"
      "ds_write_b128 %[vsw0], v[248:251]\n\t"
      "ds_write_b128 v17, v[252:255]\n\t"
      "ds_read_b128 v[248:251], %[vsr]\n\t"
      "ds_read_b128 v[252:255], %[vsr] offset:1024\n\t"
      "s_waitcnt lgkmcnt(8)\n\t"
      "buffer_store_dwordx4 v[232:235], %[vout], %[ro], s93 offen offset:128 sc1\n\t"
      "buffer_store_dwordx4 v[236:239], %[vout], %[ro], s92 offen offset:128 sc1\n\t"
      "s_waitcnt lgkmcnt(4)\n\t"
      "s_mov_b32 s93, %[soute2]\n\t"
      "s_add_u32 s92, s93, 0x10000\n\t"
      "buffer_store_dwordx4 v[240:243], %[vout], %[ro], s93 offen offset:0 sc1\n\t"
      "buffer_store_dwordx4 v[244:247], %[vout], %[ro], s92 offen offset:0 sc1\n\t"
      "s_waitcnt lgkmcnt(0)\n\t"
      "buffer_store_dwordx4 v[248:251], %[vout], %[ro], s93 offen offset:128 sc1\n\t"
      "buffer_store_dwordx4 v[252:255], %[vout], %[ro], s92 offen offset:128 sc1\n\t"
      "s_branch P3_%=\n\t"
      "P3_%=:\n\t"
      "s_waitcnt lgkmcnt(13)\n\t"
      "v_mfma_f32_16x16x128_f8f6f4 v[160:163], v[24:31], v[56:63], v[160:163] cbsz:1\n\t"
      "ds_read_b128 v[72:75], v9 offset:4096\n\t"
      "ds_read_b128 v[76:79], v10 offset:4096\n\t"
      "ds_read_b128 v[80:83], v9 offset:6144\n\t"
      "v_mfma_f32_16x16x128_f8f6f4 v[176:179], v[24:31], v[64:71], v[176:179] cbsz:1\n\t"
      "ds_read_b128 v[84:87], v10 offset:6144\n\t"
      "s_add_u32 s82, s82, 1\n\t"
      "v_mov_b32 v21, s82\n\t"
      "ds_write_b32 v18, v21\n\t"
      "s_waitcnt lgkmcnt(14)\n\t"
      "v_mfma_f32_16x16x128_f8f6f4 v[164:167], v[32:39], v[56:63], v[164:167] cbsz:1\n\t"
      "s_add_u32 s89, s87, 32768\n\t"
      "s_and_b32 s89, s89, 0x1ffff\n\t"
      "s_xor_b32 s90, s89, s87\n\t"
      "s_mov_b32 s87, s89\n\t"
      "v_xor_b32 v9, s90, v9\n\t"
      "v_mfma_f32_16x16x128_f8f6f4 v[180:183], v[32:39], v[64:71], v[180:183] cbsz:1\n\t"
      "v_xor_b32 v10, s90, v10\n\t"
      "ds_read_b64 v[22:23], v19\n\t"
      "s_add_u32 s89, s85, 32768\n\t"
      "s_and_b32 s89, s89, 0x1ffff\n\t"
      "s_waitcnt lgkmcnt(11)\n\t"
      "v_mfma_f32_16x16x128_f8f6f4 v[168:171], v[40:47], v[56:63], v[168:171] cbsz:1\n\t"
      "s_xor_b32 s90, s89, s85\n\t"
      "s_mov_b32 s85, s89\n\t"
      "v_xor_b32 v13, s90, v13\n\t"
      "v_xor_b32 v14, s90, v14\n\t"
      "v_mfma_f32_16x16x128_f8f6f4 v[184:187], v[40:47], v[64:71], v[184:187] cbsz:1\n\t"
      "v_xor_b32 v15, s90, v15\n\t"
      "v_xor_b32 v16, s90, v16\n\t"
      "s_add_u32 s80, s80, 1\n\t"
      "s_waitcnt lgkmcnt(7)\n\t"
      "v_mfma_f32_16x16x128_f8f6f4 v[172:175], v[48:55], v[56:63], v[172:175] cbsz:1\n\t"
      "s_waitcnt lgkmcnt(0)\n\t"
      "v_mfma_f32_16x16x128_f8f6f4 v[188:191], v[48:55], v[64:71], v[188:191] cbsz:1\n\t"
      "v_readfirstlane_b32 s84, v23\n\t"
      "v_readfirstlane_b32 s83, v22\n\t"
      "s_cmp_gt_u32 s84, s80\n\t"
      "s_cbranch_scc0 Z3_%=\n\t"
      "s_cmp_gt_u32 s83, s81\n\t"
      "s_cbranch_scc0 S3_%=\n\t"
      "v_mfma_f32_16x16x128_f8f6f4 v[192:195], v[24:31], v[72:79], v[192:195] cbsz:1\n\t"
      "ds_read_b128 v[56:59], v9 offset:0\n\t"
      "ds_read_b128 v[60:63], v10 offset:0\n\t"
      "ds_read_b128 v[64:67], v9 offset:2048\n\t"
      "v_mfma_f32_16x16x128_f8f6f4 v[208:211], v[24:31], v[80:87], v[208:211] cbsz:1\n\t"
      "ds_read_b128 v[68:71], v10 offset:2048\n\t"
      "ds_read_b64_tr_b8 v[24:25], v13 offset:0\n\t"
      "ds_read_b64_tr_b8 v[26:27], v13 offset:1024\n\t"
      "v_mfma_f32_16x16x128_f8f6f4 v[196:199], v[32:39], v[72:79], v[196:199] cbsz:1\n\t"
      "ds_read_b64_tr_b8 v[28:29], v13 offset:8192\n\t"
      "ds_read_b64_tr_b8 v[30:31], v13 offset:9216\n\t"
      "v_mfma_f32_16x16x128_f8f6f4 v[212:215], v[32:39], v[80:87], v[212:215] cbsz:1\n\t"
      "ds_read_b64_tr_b8 v[32:33], v14 offset:0\n\t"
      "ds_read_b64_tr_b8 v[34:35], v14 offset:1024\n\t"
      "ds_read_b64_tr_b8 v[36:37], v14 offset:8192\n\t"
      "v_mfma_f32_16x16x128_f8f6f4 v[200:203], v[40:47], v[72:79], v[200:203] cbsz:1\n\t"
      "ds_read_b64_tr_b8 v[38:39], v14 offset:9216\n\t"
      "v_mfma_f32_16x16x128_f8f6f4 v[216:219], v[40:47], v[80:87], v[216:219] cbsz:1\n\t"
      "ds_read_b64_tr_b8 v[40:41], v15 offset:0\n\t"
      "ds_read_b64_tr_b8 v[42:43], v15 offset:1024\n\t"
      "ds_read_b64_tr_b8 v[44:45], v15 offset:8192\n\t"
      "v_mfma_f32_16x16x128_f8f6f4 v[204:207], v[48:55], v[72:79], v[204:207] cbsz:1\n\t"
      "ds_read_b64_tr_b8 v[46:47], v15 offset:9216\n\t"
      "s_add_u32 s82, s82, 1\n\t"
      "v_mov_b32 v21, s82\n\t"
      "v_mfma_f32_16x16x128_f8f6f4 v[220:223], v[48:55], v[80:87], v[220:223] cbsz:1\n\t"
      "ds_read_b64_tr_b8 v[48:49], v16 offset:0\n\t"
      "ds_read_b64_tr_b8 v[50:51], v16 offset:1024\n\t"
      "ds_read_b64_tr_b8 v[52:53], v16 offset:8192\n\t"
      "ds_read_b64_tr_b8 v[54:55], v16 offset:9216\n\t"
      "ds_write_b32 v18, v21\n\t"
      "JOIN3_%=:\n\t"
      "s_add_u32 s81, s81, 1\n\t"
      "s_branch P3_%=\n\t"
      "EPI_%=:\n\t"
      "ds_read_b32 v22, v19 offset:36\n\t"
      "s_waitcnt lgkmcnt(0)\n\t"
      "v_readfirstlane_b32 s89, v22\n\t"
      "s_cmp_lg_u32 s89, 0\n\t"
      "s_cselect_b32 s89, 0x7fc00000, %[sscale]\n\t"
      "v_mov_b32 v20, s89\n\t"
      "s_nop 7\n\t"
      "v_pk_mul_f32 v[160:161], v[160:161], v[20:21] op_sel_hi:[1,0]\n\t"
      "v_pk_mul_f32 v[162:163], v[162:163], v[20:21] op_sel_hi:[1,0]\n\t"
      "v_pk_mul_f32 v[164:165], v[164:165], v[20:21] op_sel_hi:[1,0]\n\t"
      "v_pk_mul_f32 v[166:167], v[166:167], v[20:21] op_sel_hi:[1,0]\n\t"
      "ds_write_b128 %[vsw0], v[160:163]\n\t"
      "ds_write_b128 v17, v[164:167]\n\t"
      "ds_read_b128 v[160:163], %[vsr]\n\t"
      "ds_read_b128 v[164:167], %[vsr] offset:1024\n\t"
      "v_pk_mul_f32 v[168:169], v[168:169], v[20:21] op_sel_hi:[1,0]\n\t"
      "v_pk_mul_f32 v[170:171], v[170:171], v[20:21] op_sel_hi:[1,0]\n\t"
      "v_pk_mul_f32 v[172:173], v[172:173], v[20:21] op_sel_hi:[1,0]\n\t"
      "v_pk_mul_f32 v[174:175], v[174:175], v[20:21] op_sel_hi:[1,0]\n\t"
      "ds_write_b128 %[vsw0], v[168:171]\n\t"
      "ds_write_b128 v17, v[172:175]\n\t"
      "ds_read_b128 v[168:171], %[vsr]\n\t"
      "ds_read_b128 v[172:175], %[vsr] offset:1024\n\t"
      "v_pk_mul_f32 v[176:177], v[176:177], v[20:21] op_sel_hi:[1,0]\n\t"
      "v_pk_mul_f32 v[178:179], v[178:179], v[20:21] op_sel_hi:[1,0]\n\t"
      "v_pk_mul_f32 v[180:181], v[180:181], v[20:21] op_sel_hi:[1,0]\n\t"
      "v_pk_mul_f32 v[182:183], v[182:183], v[20:21] op_sel_hi:[1,0]\n\t"
      "ds_write_b128 %[vsw0], v[176:179]\n\t"
      "ds_write_b128 v17, v[180:183]\n\t"
      "ds_read_b128 v[176:179], %[vsr]\n\t"
      "ds_read_b128 v[180:183], %[vsr] offset:1024\n\t"
      "s_waitcnt lgkmcnt(8)\n\t"
      "s_add_u32 s93, %[sout], 1048576\n\t"
      "s_add_u32 s92, s93, 0x10000\n\t"
      "buffer_store_dwordx4 v[160:163], %[vout], %[ro], s93 offen offset:0 sc1\n\t"
      "buffer_store_dwordx4 v[164:167], %[vout], %[ro], s92 offen offset:0 sc1\n\t"
      "v_pk_mul_f32 v[184:185], v[184:185], v[20:21] op_sel_hi:[1,0]\n\t"
      "v_pk_mul_f32 v[186:187], v[186:187], v[20:21] op_sel_hi:[1,0]\n\t"
      "v_pk_mul_f32 v[188:189], v[188:189], v[20:21] op_sel_hi:[1,0]\n\t"
      "v_pk_mul_f32 v[190:191], v[190:191], v[20:21] op_sel_hi:[1,0]\n\t"
      "ds_write_b128 %[vsw0], v[184:187]\n\t"
      "ds_write_b128 v17, v[188:191]\n\t"
      "ds_read_b128 v[184:187], %[vsr]\n\t"
      "ds_read_b128 v[188:191], %[vsr] offset:1024\n\t"
      "s_waitcnt lgkmcnt(8)\n\t"
      "buffer_store_dwordx4 v[168:171], %[vout], %[ro], s93 offen offset:128 sc1\n\t"
      "buffer_store_dwordx4 v[172:175], %[vout], %[ro], s92 offen offset:128 sc1\n\t"
      "v_pk_mul_f32 v[192:193], v[192:193], v[20:21] op_sel_hi:[1,0]\n\t"
      "v_pk_mul_f32 v[194:195], v[194:195], v[20:21] op_sel_hi:[1,0]\n\t"
      "v_pk_mul_f32 v[196:197], v[196:197], v[20:21] op_sel_hi:[1,0]\n\t"
      "v_pk_mul_f32 v[198:199], v[198:199], v[20:21] op_sel_hi:[1,0]\n\t"
      "ds_write_b128 %[vsw0], v[192:195]\n\t"
      "ds_write_b128 v17, v[196:199]\n\t"
      "ds_read_b128 v[192:195], %[vsr]\n\t"
      "ds_read_b128 v[196:199], %[vsr] offset:1024\n\t"
      "s_waitcnt lgkmcnt(8)\n\t"
      "s_add_u32 s93, %[sout], 1179648\n\t"
      "s_add_u32 s92, s93, 0x10000\n\t"
      "buffer_store_dwordx4 v[176:179], %[vout], %[ro], s93 offen offset:0 sc1\n\t"
      "buffer_store_dwordx4 v[180:183], %[vout], %[ro], s92 offen offset:0 sc1\n\t"
      "v_pk_mul_f32 v[200:201], v[200:201], v[20:21] op_sel_hi:[1,0]\n\t"
      "v_pk_mul_f32 v[202:203], v[202:203], v[20:21] op_sel_hi:[1,0]\n\t"
      "v_pk_mul_f32 v[204:205], v[204:205], v[20:21] op_sel_hi:[1,0]\n\t"
      "v_pk_mul_f32 v[206:207], v[206:207], v[20:21] op_sel_hi:[1,0]\n\t"
      "ds_write_b128 %[vsw0], v[200:203]\n\t"
      "ds_write_b128 v17, v[204:207]\n\t"
      "ds_read_b128 v[200:203], %[vsr]\n\t"
      "ds_read_b128 v[204:207], %[vsr] offset:1024\n\t"
      "s_waitcnt lgkmcnt(8)\n\t"
      "buffer_store_dwordx4 v[184:187], %[vout], %[ro], s93 offen offset:128 sc1\n\t"
      "buffer_store_dwordx4 v[188:191], %[vout], %[ro], s92 offen offset:128 sc1\n\t"
      "v_pk_mul_f32 v[208:209], v[208:209], v[20:21] op_sel_hi:[1,0]\n\t"
      "v_pk_mul_f32 v[210:211], v[210:211], v[20:21] op_sel_hi:[1,0]\n\t"
      "v_pk_mul_f32 v[212:213], v[212:213], v[20:21] op_sel_hi:[1,0]\n\t"
      "v_pk_mul_f32 v[214:215], v[214:215], v[20:21] op_sel_hi:[1,0]\n\t"
      "ds_write_b128 %[vsw0], v[208:211]\n\t"
      "ds_write_b128 v17, v[212:215]\n\t"
      "ds_read_b128 v[208:211], %[vsr]\n\t"
      "ds_read_b128 v[212:215], %[vsr] offset:1024\n\t"
      "s_waitcnt lgkmcnt(8)\n\t"
      "s_add_u32 s93, %[sout], 1310720\n\t"
      "s_add_u32 s92, s93, 0x10000\n\t"
      "buffer_store_dwordx4 v[192:195], %[vout], %[ro], s93 offen offset:0 sc1\n\t"
      "buffer_store_dwordx4 v[196:199], %[vout], %[ro], s92 offen offset:0 sc1\n\t"
      "v_pk_mul_f32 v[216:217], v[216:217], v[20:21] op_sel_hi:[1,0]\n\t"
      "v_pk_mul_f32 v[218:219], v[218:219], v[20:21] op_sel_hi:[1,0]\n\t"
      "v_pk_mul_f32 v[220:221], v[220:221], v[20:21] op_sel_hi:[1,0]\n\t"
      "v_pk_mul_f32 v[222:223], v[222:223], v[20:21] op_sel_hi:[1,0]\n\t"
      "ds_write_b128 %[vsw0], v[216:219]\n\t"
      "ds_write_b128 v17, v[220:223]\n\t"
      "ds_read_b128 v[216:219], %[vsr]\n\t"
      "ds_read_b128 v[220:223], %[vsr] offset:1024\n\t"
      "s_waitcnt lgkmcnt(8)\n\t"
      "buffer_store_dwordx4 v[200:203], %[vout], %[ro], s93 offen offset:128 sc1\n\t"
      "buffer_store_dwordx4 v[204:207], %[vout], %[ro], s92 offen offset:128 sc1\n\t"
      "s_waitcnt lgkmcnt(4)\n\t"
      "s_add_u32 s93, %[sout], 1441792\n\t"
      "s_add_u32 s92, s93, 0x10000\n\t"
      "buffer_store_dwordx4 v[208:211], %[vout], %[ro], s93 offen offset:0 sc1\n\t"
      "buffer_store_dwordx4 v[212:215], %[vout], %[ro], s92 offen offset:0 sc1\n\t"
      "s_waitcnt lgkmcnt(0)\n\t"
      "buffer_store_dwordx4 v[216:219], %[vout], %[ro], s93 offen offset:128 sc1\n\t"
      "buffer_store_dwordx4 v[220:223], %[vout], %[ro], s92 offen offset:128 sc1\n\t"
      "s_branch END_%=\n\t"
      "S1N_%=:\n\t"
      "v_mfma_f32_16x16x128_f8f6f4 v[112:115], v[24:31], v[72:79], v[112:115] cbsz:1\n\t"
      "ds_read_b128 v[56:59], v11 offset:0\n\t"
      "ds_read_b128 v[60:63], v12 offset:0\n\t"
      "v_mfma_f32_16x16x128_f8f6f4 v[128:131], v[24:31], v[80:87], v[128:131] cbsz:1\n\t"
      "v_mfma_f32_16x16x128_f8f6f4 v[144:147], v[24:31], v[88:95], v[144:147] cbsz:1\n\t"
      "ds_read_b64_tr_b8 v[24:25], v13 offset:0\n\t"
      "ds_read_b64_tr_b8 v[26:27], v13 offset:1024\n\t"
      "v_mfma_f32_16x16x128_f8f6f4 v[116:119], v[32:39], v[72:79], v[116:119] cbsz:1\n\t"
      "ds_read_b64_tr_b8 v[28:29], v13 offset:8192\n\t"
      "ds_read_b64_tr_b8 v[30:31], v13 offset:9216\n\t"
      "v_mfma_f32_16x16x128_f8f6f4 v[132:135], v[32:39], v[80:87], v[132:135] cbsz:1\n\t"
      "v_mfma_f32_16x16x128_f8f6f4 v[148:151], v[32:39], v[88:95], v[148:151] cbsz:1\n\t"
      "ds_read_b64_tr_b8 v[32:33], v14 offset:0\n\t"
      "ds_read_b64_tr_b8 v[34:35], v14 offset:1024\n\t"
      "v_mfma_f32_16x16x128_f8f6f4 v[120:123], v[40:47], v[72:79], v[120:123] cbsz:1\n\t"
      "ds_read_b64_tr_b8 v[36:37], v14 offset:8192\n\t"
      "ds_read_b64_tr_b8 v[38:39], v14 offset:9216\n\t"
      "v_mfma_f32_16x16x128_f8f6f4 v[136:139], v[40:47], v[80:87], v[136:139] cbsz:1\n\t"
      "v_mfma_f32_16x16x128_f8f6f4 v[152:155], v[40:47], v[88:95], v[152:155] cbsz:1\n\t"
      "ds_read_b64_tr_b8 v[40:41], v15 offset:0\n\t"
      "ds_read_b64_tr_b8 v[42:43], v15 offset:1024\n\t"
      "v_mfma_f32_16x16x128_f8f6f4 v[124:127], v[48:55], v[72:79], v[124:127] cbsz:1\n\t"
      "ds_read_b64_tr_b8 v[44:45], v15 offset:8192\n\t"
      "ds_read_b64_tr_b8 v[46:47], v15 offset:9216\n\t"
      "s_add_u32 s82, s82, 1\n\t"
      "v_mov_b32 v21, s82\n\t"
      "v_mfma_f32_16x16x128_f8f6f4 v[140:143], v[48:55], v[80:87], v[140:143] cbsz:1\n\t"
      "v_mfma_f32_16x16x128_f8f6f4 v[156:159], v[48:55], v[88:95], v[156:159] cbsz:1\n\t"
      "ds_read_b64_tr_b8 v[48:49], v16 offset:0\n\t"
      "ds_read_b64_tr_b8 v[50:51], v16 offset:1024\n\t"
      "ds_read_b64_tr_b8 v[52:53], v16 offset:8192\n\t"
      "ds_read_b64_tr_b8 v[54:55], v16 offset:9216\n\t"
      "ds_write_b32 v18, v21\n\t"
      "SPINS1N_%=:\n\t"
      "ds_read_b64 v[22:23], v19\n\t"
      "s_waitcnt lgkmcnt(0)\n\t"
      "v_readfirstlane_b32 s83, v22\n\t"
      "s_cmp_gt_u32 s83, s81\n\t"
      "s_cbranch_scc1 GOS1N_%=\n\t"
      "s_sleep 1\n\t"
      "s_add_u32 s91, s91, 1\n\t"
      "s_and_b32 s96, s91, 1023\n\t"
      "s_cbranch_scc1 SPINS1N_%=\n\t"
      "s_memrealtime s[96:97]\n\t"
      "s_waitcnt lgkmcnt(0)\n\t"
      "s_sub_u32 s96, %[tlo], s96\n\t"
      "s_subb_u32 s97, %[thi], s97\n\t"
      "s_cbranch_scc0 SPINS1N_%=\n\t"
      "v_mov_b32 v22, 1\n\t"
      "ds_write_b32 v19, v22 offset:36\n\t"
      "GOS1N_%=:\n\t"
      "ds_read_b128 v[64:67], v7 offset:0\n\t"
      "ds_read_b128 v[68:71], v8 offset:0\n\t"
      "s_waitcnt lgkmcnt(0)\n\t"
      "s_branch JOIN1N_%=\n\t"
      "Z1N_%=:\n\t"
      "v_mfma_f32_16x16x128_f8f6f4 v[112:115], v[24:31], v[72:79], v[112:115] cbsz:1\n\t"
      "v_mfma_f32_16x16x128_f8f6f4 v[128:131], v[24:31], v[80:87], v[128:131] cbsz:1\n\t"
      "v_mfma_f32_16x16x128_f8f6f4 v[144:147], v[24:31], v[88:95], v[144:147] cbsz:1\n\t"
      "v_mfma_f32_16x16x128_f8f6f4 v[116:119], v[32:39], v[72:79], v[116:119] cbsz:1\n\t"
      "v_mfma_f32_16x16x128_f8f6f4 v[132:135], v[32:39], v[80:87], v[132:135] cbsz:1\n\t"
      "v_mfma_f32_16x16x128_f8f6f4 v[148:151], v[32:39], v[88:95], v[148:151] cbsz:1\n\t"
      "v_mfma_f32_16x16x128_f8f6f4 v[120:123], v[40:47], v[72:79], v[120:123] cbsz:1\n\t"
      "v_mfma_f32_16x16x128_f8f6f4 v[136:139], v[40:47], v[80:87], v[136:139] cbsz:1\n\t"
      "v_mfma_f32_16x16x128_f8f6f4 v[152:155], v[40:47], v[88:95], v[152:155] cbsz:1\n\t"
      "v_mfma_f32_16x16x128_f8f6f4 v[124:127], v[48:55], v[72:79], v[124:127] cbsz:1\n\t"
      "v_mfma_f32_16x16x128_f8f6f4 v[140:143], v[48:55], v[80:87], v[140:143] cbsz:1\n\t"
      "v_mfma_f32_16x16x128_f8f6f4 v[156:159], v[48:55], v[88:95], v[156:159] cbsz:1\n\t"
      "SPINZ1N_%=:\n\t"
      "ds_read_b64 v[22:23], v19\n\t"
      "s_waitcnt lgkmcnt(0)\n\t"
      "v_readfirstlane_b32 s83, v22\n\t"
      "v_readfirstlane_b32 s84, v23\n\t"
      "s_cmp_gt_u32 s84, s80\n\t"
      "s_cselect_b32 s89, s83, 0\n\t"
      "s_cmp_gt_u32 s89, s81\n\t"
      "s_cbranch_scc1 GOZ1N_%=\n\t"
      "s_sleep 1\n\t"
      "s_add_u32 s91, s91, 1\n\t"
      "s_and_b32 s96, s91, 1023\n\t"
      "s_cbranch_scc1 SPINZ1N_%=\n\t"
      "s_memrealtime s[96:97]\n\t"
      "s_waitcnt lgkmcnt(0)\n\t"
      "s_sub_u32 s96, %[tlo], s96\n\t"
      "s_subb_u32 s97, %[thi], s97\n\t"
      "s_cbranch_scc0 SPINZ1N_%=\n\t"
      "v_mov_b32 v22, 1\n\t"
      "ds_write_b32 v19, v22 offset:36\n\t"
      "GOZ1N_%=:\n\t"
      "ds_read_b128 v[56:59], v11 offset:0\n\t"
      "ds_read_b128 v[60:63], v12 offset:0\n\t"
      "ds_read_b128 v[64:67], v7 offset:0\n\t"
      "ds_read_b128 v[68:71], v8 offset:0\n\t"
      "ds_read_b64_tr_b8 v[24:25], v13 offset:0\n\t"
      "ds_read_b64_tr_b8 v[26:27], v13 offset:1024\n\t"
      "ds_read_b64_tr_b8 v[28:29], v13 offset:8192\n\t"
      "ds_read_b64_tr_b8 v[30:31], v13 offset:9216\n\t"
      "ds_read_b64_tr_b8 v[32:33], v14 offset:0\n\t"
      "ds_read_b64_tr_b8 v[34:35], v14 offset:1024\n\t"
      "ds_read_b64_tr_b8 v[36:37], v14 offset:8192\n\t"
      "ds_read_b64_tr_b8 v[38:39], v14 offset:9216\n\t"
      "ds_read_b64_tr_b8 v[40:41], v15 offset:0\n\t"
      "ds_read_b64_tr_b8 v[42:43], v15 offset:1024\n\t"
      "ds_read_b64_tr_b8 v[44:45], v15 offset:8192\n\t"
      "ds_read_b64_tr_b8 v[46:47], v15 offset:9216\n\t"
      "ds_read_b64_tr_b8 v[48:49], v16 offset:0\n\t"
      "ds_read_b64_tr_b8 v[50:51], v16 offset:1024\n\t"
      "ds_read_b64_tr_b8 v[52:53], v16 offset:8192\n\t"
      "ds_read_b64_tr_b8 v[54:55], v16 offset:9216\n\t"
      "s_add_u32 s82, s82, 1\n\t"
      "v_mov_b32 v21, s82\n\t"
      "ds_write_b32 v18, v21\n\t"
      "s_branch JOIN1N_%=\n\t"
      "SPINL2N_%=:\n\t"
      "ds_read_b64 v[22:23], v19\n\t"
      "s_waitcnt lgkmcnt(0)\n\t"
      "v_readfirstlane_b32 s83, v22\n\t"
      "s_cmp_gt_u32 s83, s81\n\t"
      "s_cbranch_scc1 GOL2N_%=\n\t"
      "s_sleep 1\n\t"
      "s_add_u32 s91, s91, 1\n\t"
      "s_and_b32 s96, s91, 1023\n\t"
      "s_cbranch_scc1 SPINL2N_%=\n\t"
      "s_memrealtime s[96:97]\n\t"
      "s_waitcnt lgkmcnt(0)\n\t"
      "s_sub_u32 s96, %[tlo], s96\n\t"
      "s_subb_u32 s97, %[thi], s97\n\t"
      "s_cbranch_scc0 SPINL2N_%=\n\t"
      "v_mov_b32 v22, 1\n\t"
      "ds_write_b32 v19, v22 offset:36\n\t"
      "GOL2N_%=:\n\t"
      "s_branch BACKL2N_%=\n\t"
      "S2N_%=:\n\t"
      "v_mfma_f32_16x16x128_f8f6f4 v[192:195], v[24:31], v[72:79], v[192:195] cbsz:1\n\t"
      "ds_read_b128 v[56:59], v11 offset:0\n\t"
      "ds_read_b128 v[60:63], v12 offset:0\n\t"
      "v_mfma_f32_16x16x128_f8f6f4 v[208:211], v[24:31], v[80:87], v[208:211] cbsz:1\n\t"
      "ds_read_b64_tr_b8 v[24:25], v13 offset:0\n\t"
      "ds_read_b64_tr_b8 v[26:27], v13 offset:1024\n\t"
      "ds_read_b64_tr_b8 v[28:29], v13 offset:8192\n\t"
      "v_mfma_f32_16x16x128_f8f6f4 v[196:199], v[32:39], v[72:79], v[196:199] cbsz:1\n\t"
      "ds_read_b64_tr_b8 v[30:31], v13 offset:9216\n\t"
      "v_mfma_f32_16x16x128_f8f6f4 v[212:215], v[32:39], v[80:87], v[212:215] cbsz:1\n\t"
      "ds_read_b64_tr_b8 v[32:33], v14 offset:0\n\t"
      "ds_read_b64_tr_b8 v[34:35], v14 offset:1024\n\t"
      "ds_read_b64_tr_b8 v[36:37], v14 offset:8192\n\t"
      "v_mfma_f32_16x16x128_f8f6f4 v[200:203], v[40:47], v[72:79], v[200:203] cbsz:1\n\t"
      "ds_read_b64_tr_b8 v[38:39], v14 offset:9216\n\t"
      "v_mfma_f32_16x16x128_f8f6f4 v[216:219], v[40:47], v[80:87], v[216:219] cbsz:1\n\t"
      "ds_read_b64_tr_b8 v[40:41], v15 offset:0\n\t"
      "ds_read_b64_tr_b8 v[42:43], v15 offset:1024\n\t"
      "ds_read_b64_tr_b8 v[44:45], v15 offset:8192\n\t"
      "v_mfma_f32_16x16x128_f8f6f4 v[204:207], v[48:55], v[72:79], v[204:207] cbsz:1\n\t"
      "ds_read_b64_tr_b8 v[46:47], v15 offset:9216\n\t"
      "s_add_u32 s82, s82, 1\n\t"
      "v_mov_b32 v21, s82\n\t"
      "v_mfma_f32_16x16x128_f8f6f4 v[220:223], v[48:55], v[80:87], v[220:223] cbsz:1\n\t"
      "ds_read_b64_tr_b8 v[48:49], v16 offset:0\n\t"
      "ds_read_b64_tr_b8 v[50:51], v16 offset:1024\n\t"
      "ds_read_b64_tr_b8 v[52:53], v16 offset:8192\n\t"
      "ds_read_b64_tr_b8 v[54:55], v16 offset:9216\n\t"
      "ds_write_b32 v18, v21\n\t"
      "SPINS2N_%=:\n\t"
      "ds_read_b64 v[22:23], v19\n\t"
      "s_waitcnt lgkmcnt(0)\n\t"
      "v_readfirstlane_b32 s83, v22\n\t"
      "s_cmp_gt_u32 s83, s81\n\t"
      "s_cbranch_scc1 GOS2N_%=\n\t"
      "s_sleep 1\n\t"
      "s_add_u32 s91, s91, 1\n\t"
      "s_and_b32 s96, s91, 1023\n\t"
      "s_cbranch_scc1 SPINS2N_%=\n\t"
      "s_memrealtime s[96:97]\n\t"
      "s_waitcnt lgkmcnt(0)\n\t"
      "s_sub_u32 s96, %[tlo], s96\n\t"
      "s_subb_u32 s97, %[thi], s97\n\t"
      "s_cbranch_scc0 SPINS2N_%=\n\t"
      "v_mov_b32 v22, 1\n\t"
      "ds_write_b32 v19, v22 offset:36\n\t"
      "GOS2N_%=:\n\t"
      "ds_read_b128 v[64:67], v7 offset:0\n\t"
      "ds_read_b128 v[68:71], v8 offset:0\n\t"
      "s_waitcnt lgkmcnt(0)\n\t"
      "s_branch JOIN2N_%=\n\t"
      "Z2N_%=:\n\t"
      "v_mfma_f32_16x16x128_f8f6f4 v[192:195], v[24:31], v[72:79], v[192:195] cbsz:1\n\t"
      "v_mfma_f32_16x16x128_f8f6f4 v[208:211], v[24:31], v[80:87], v[208:211] cbsz:1\n\t"
      "v_mfma_f32_16x16x128_f8f6f4 v[196:199], v[32:39], v[72:79], v[196:199] cbsz:1\n\t"
      "v_mfma_f32_16x16x128_f8f6f4 v[212:215], v[32:39], v[80:87], v[212:215] cbsz:1\n\t"
      "v_mfma_f32_16x16x128_f8f6f4 v[200:203], v[40:47], v[72:79], v[200:203] cbsz:1\n\t"
      "v_mfma_f32_16x16x128_f8f6f4 v[216:219], v[40:47], v[80:87], v[216:219] cbsz:1\n\t"
      "v_mfma_f32_16x16x128_f8f6f4 v[204:207], v[48:55], v[72:79], v[204:207] cbsz:1\n\t"
      "v_mfma_f32_16x16x128_f8f6f4 v[220:223], v[48:55], v[80:87], v[220:223] cbsz:1\n\t"
      "SPINZ2N_%=:\n\t"
      "ds_read_b64 v[22:23], v19\n\t"
      "s_waitcnt lgkmcnt(0)\n\t"
      "v_readfirstlane_b32 s83, v22\n\t"
      "v_readfirstlane_b32 s84, v23\n\t"
      "s_cmp_gt_u32 s84, s80\n\t"
      "s_cselect_b32 s89, s83, 0\n\t"
      "s_cmp_gt_u32 s89, s81\n\t"
      "s_cbranch_scc1 GOZ2N_%=\n\t"
      "s_sleep 1\n\t"
      "s_add_u32 s91, s91, 1\n\t"
      "s_and_b32 s96, s91, 1023\n\t"
      "s_cbranch_scc1 SPINZ2N_%=\n\t"
      "s_memrealtime s[96:97]\n\t"
      "s_waitcnt lgkmcnt(0)\n\t"
      "s_sub_u32 s96, %[tlo], s96\n\t"
      "s_subb_u32 s97, %[thi], s97\n\t"
      "s_cbranch_scc0 SPINZ2N_%=\n\t"
      "v_mov_b32 v22, 1\n\t"
      "ds_write_b32 v19, v22 offset:36\n\t"
      "GOZ2N_%=:\n\t"
      "ds_read_b128 v[56:59], v11 offset:0\n\t"
      "ds_read_b128 v[60:63], v12 offset:0\n\t"
      "ds_read_b128 v[64:67], v7 offset:0\n\t"
      "ds_read_b128 v[68:71], v8 offset:0\n\t"
      "ds_read_b64_tr_b8 v[24:25], v13 offset:0\n\t"
      "ds_read_b64_tr_b8 v[26:27], v13 offset:1024\n\t"
      "ds_read_b64_tr_b8 v[28:29], v13 offset:8192\n\t"
      "ds_read_b64_tr_b8 v[30:31], v13 offset:9216\n\t"
      "ds_read_b64_tr_b8 v[32:33], v14 offset:0\n\t"
      "ds_read_b64_tr_b8 v[34:35], v14 offset:1024\n\t"
      "ds_read_b64_tr_b8 v[36:37], v14 offset:8192\n\t"
      "ds_read_b64_tr_b8 v[38:39], v14 offset:9216\n\t"
      "ds_read_b64_tr_b8 v[40:41], v15 offset:0\n\t"
      "ds_read_b64_tr_b8 v[42:43], v15 offset:1024\n\t"
      "ds_read_b64_tr_b8 v[44:45], v15 offset:8192\n\t"
      "ds_read_b64_tr_b8 v[46:47], v15 offset:9216\n\t"
      "ds_read_b64_tr_b8 v[48:49], v16 offset:0\n\t"
      "ds_read_b64_tr_b8 v[50:51], v16 offset:1024\n\t"
      "ds_read_b64_tr_b8 v[52:53], v16 offset:8192\n\t"
      "ds_read_b64_tr_b8 v[54:55], v16 offset:9216\n\t"
      "s_add_u32 s82, s82, 1\n\t"
      "v_mov_b32 v21, s82\n\t"
      "ds_write_b32 v18, v21\n\t"
      "s_branch JOIN2N_%=\n\t"
      "SPINL2LN_%=:\n\t"
      "ds_read_b64 v[22:23], v19\n\t"
      "s_waitcnt lgkmcnt(0)\n\t"
      "v_readfirstlane_b32 s83, v22\n\t"
      "s_cmp_gt_u32 s83, s81\n\t"
      "s_cbranch_scc1 GOL2LN_%=\n\t"
      "s_sleep 1\n\t"
      "s_add_u32 s91, s91, 1\n\t"
      "s_and_b32 s96, s91, 1023\n\t"
      "s_cbranch_scc1 SPINL2LN_%=\n\t"
      "s_memrealtime s[96:97]\n\t"
      "s_waitcnt lgkmcnt(0)\n\t"
      "s_sub_u32 s96, %[tlo], s96\n\t"
      "s_subb_u32 s97, %[thi], s97\n\t"
      "s_cbranch_scc0 SPINL2LN_%=\n\t"
      "v_mov_b32 v22, 1\n\t"
      "ds_write_b32 v19, v22 offset:36\n\t"
      "GOL2LN_%=:\n\t"
      "s_branch BACKL2LN_%=\n\t"
      "S2LN_%=:\n\t"
      "v_mfma_f32_16x16x128_f8f6f4 v[192:195], v[24:31], v[72:79], v[192:195] cbsz:1\n\t"
      "v_mfma_f32_16x16x128_f8f6f4 v[208:211], v[24:31], v[80:87], v[208:211] cbsz:1\n\t"
      "ds_read_b64_tr_b8 v[24:25], v13 offset:0\n\t"
      "ds_read_b64_tr_b8 v[26:27], v13 offset:1024\n\t"
      "ds_read_b64_tr_b8 v[28:29], v13 offset:8192\n\t"
      "v_mfma_f32_16x16x128_f8f6f4 v[196:199], v[32:39], v[72:79], v[196:199] cbsz:1\n\t"
      "ds_read_b64_tr_b8 v[30:31], v13 offset:9216\n\t"
      "v_mfma_f32_16x16x128_f8f6f4 v[212:215], v[32:39], v[80:87], v[212:215] cbsz:1\n\t"
      "ds_read_b64_tr_b8 v[32:33], v14 offset:0\n\t"
      "ds_read_b64_tr_b8 v[34:35], v14 offset:1024\n\t"
      "ds_read_b64_tr_b8 v[36:37], v14 offset:8192\n\t"
      "v_mfma_f32_16x16x128_f8f6f4 v[200:203], v[40:47], v[72:79], v[200:203] cbsz:1\n\t"
      "ds_read_b64_tr_b8 v[38:39], v14 offset:9216\n\t"
      "v_mfma_f32_16x16x128_f8f6f4 v[216:219], v[40:47], v[80:87], v[216:219] cbsz:1\n\t"
      "ds_read_b64_tr_b8 v[40:41], v15 offset:0\n\t"
      "ds_read_b64_tr_b8 v[42:43], v15 offset:1024\n\t"
      "ds_read_b64_tr_b8 v[44:45], v15 offset:8192\n\t"
      "v_mfma_f32_16x16x128_f8f6f4 v[204:207], v[48:55], v[72:79], v[204:207] cbsz:1\n\t"
      "ds_read_b64_tr_b8 v[46:47], v15 offset:9216\n\t"
      "s_add_u32 s82, s82, 1\n\t"
      "v_mov_b32 v21, s82\n\t"
      "v_mfma_f32_16x16x128_f8f6f4 v[220:223], v[48:55], v[80:87], v[220:223] cbsz:1\n\t"
      "ds_read_b64_tr_b8 v[48:49], v16 offset:0\n\t"
      "ds_read_b64_tr_b8 v[50:51], v16 offset:1024\n\t"
      "ds_read_b64_tr_b8 v[52:53], v16 offset:8192\n\t"
      "ds_read_b64_tr_b8 v[54:55], v16 offset:9216\n\t"
      "ds_write_b32 v18, v21\n\t"
      "SPINS2LN_%=:\n\t"
      "ds_read_b64 v[22:23], v19\n\t"
      "s_waitcnt lgkmcnt(0)\n\t"
      "v_readfirstlane_b32 s83, v22\n\t"
      "s_cmp_gt_u32 s83, s81\n\t"
      "s_cbranch_scc1 GOS2LN_%=\n\t"
      "s_sleep 1\n\t"
      "s_add_u32 s91, s91, 1\n\t"
      "s_and_b32 s96, s91, 1023\n\t"
      "s_cbranch_scc1 SPINS2LN_%=\n\t"
      "s_memrealtime s[96:97]\n\t"
      "s_waitcnt lgkmcnt(0)\n\t"
      "s_sub_u32 s96, %[tlo], s96\n\t"
      "s_subb_u32 s97, %[thi], s97\n\t"
      "s_cbranch_scc0 SPINS2LN_%=\n\t"
      "v_mov_b32 v22, 1\n\t"
      "ds_write_b32 v19, v22 offset:36\n\t"
      "GOS2LN_%=:\n\t"
      "ds_read_b128 v[56:59], v9 offset:0\n\t"
      "ds_read_b128 v[60:63], v10 offset:0\n\t"
      "ds_read_b128 v[64:67], v9 offset:2048\n\t"
      "ds_read_b128 v[68:71], v10 offset:2048\n\t"
      "s_waitcnt lgkmcnt(0)\n\t"
      "s_branch JOIN2LN_%=\n\t"
      "Z2LN_%=:\n\t"
      "v_mfma_f32_16x16x128_f8f6f4 v[192:195], v[24:31], v[72:79], v[192:195] cbsz:1\n\t"
      "v_mfma_f32_16x16x128_f8f6f4 v[208:211], v[24:31], v[80:87], v[208:211] cbsz:1\n\t"
      "v_mfma_f32_16x16x128_f8f6f4 v[196:199], v[32:39], v[72:79], v[196:199] cbsz:1\n\t"
      "v_mfma_f32_16x16x128_f8f6f4 v[212:215], v[32:39], v[80:87], v[212:215] cbsz:1\n\t"
      "v_mfma_f32_16x16x128_f8f6f4 v[200:203], v[40:47], v[72:79], v[200:203] cbsz:1\n\t"
      "v_mfma_f32_16x16x128_f8f6f4 v[216:219], v[40:47], v[80:87], v[216:219] cbsz:1\n\t"
      "v_mfma_f32_16x16x128_f8f6f4 v[204:207], v[48:55], v[72:79], v[204:207] cbsz:1\n\t"
      "v_mfma_f32_16x16x128_f8f6f4 v[220:223], v[48:55], v[80:87], v[220:223] cbsz:1\n\t"
      "SPINZ2LN_%=:\n\t"
      "ds_read_b64 v[22:23], v19\n\t"
      "s_waitcnt lgkmcnt(0)\n\t"
      "v_readfirstlane_b32 s83, v22\n\t"
      "v_readfirstlane_b32 s84, v23\n\t"
      "s_cmp_gt_u32 s84, s80\n\t"
      "s_cselect_b32 s89, s83, 0\n\t"
      "s_cmp_gt_u32 s89, s81\n\t"
      "s_cbranch_scc1 GOZ2LN_%=\n\t"
      "s_sleep 1\n\t"
      "s_add_u32 s91, s91, 1\n\t"
      "s_and_b32 s96, s91, 1023\n\t"
      "s_cbranch_scc1 SPINZ2LN_%=\n\t"
      "s_memrealtime s[96:97]\n\t"
      "s_waitcnt lgkmcnt(0)\n\t"
      "s_sub_u32 s96, %[tlo], s96\n\t"
      "s_subb_u32 s97, %[thi], s97\n\t"
      "s_cbranch_scc0 SPINZ2LN_%=\n\t"
      "v_mov_b32 v22, 1\n\t"
      "ds_write_b32 v19, v22 offset:36\n\t"
      "GOZ2LN_%=:\n\t"
      "ds_read_b128 v[56:59], v9 offset:0\n\t"
      "ds_read_b128 v[60:63], v10 offset:0\n\t"
      "ds_read_b128 v[64:67], v9 offset:2048\n\t"
      "ds_read_b128 v[68:71], v10 offset:2048\n\t"
      "ds_read_b64_tr_b8 v[24:25], v13 offset:0\n\t"
      "ds_read_b64_tr_b8 v[26:27], v13 offset:1024\n\t"
      "ds_read_b64_tr_b8 v[28:29], v13 offset:8192\n\t"
      "ds_read_b64_tr_b8 v[30:31], v13 offset:9216\n\t"
      "ds_read_b64_tr_b8 v[32:33], v14 offset:0\n\t"
      "ds_read_b64_tr_b8 v[34:35], v14 offset:1024\n\t"
      "ds_read_b64_tr_b8 v[36:37], v14 offset:8192\n\t"
      "ds_read_b64_tr_b8 v[38:39], v14 offset:9216\n\t"
      "ds_read_b64_tr_b8 v[40:41], v15 offset:0\n\t"
      "ds_read_b64_tr_b8 v[42:43], v15 offset:1024\n\t"
      "ds_read_b64_tr_b8 v[44:45], v15 offset:8192\n\t"
      "ds_read_b64_tr_b8 v[46:47], v15 offset:9216\n\t"
      "ds_read_b64_tr_b8 v[48:49], v16 offset:0\n\t"
      "ds_read_b64_tr_b8 v[50:51], v16 offset:1024\n\t"
      "ds_read_b64_tr_b8 v[52:53], v16 offset:8192\n\t"
      "ds_read_b64_tr_b8 v[54:55], v16 offset:9216\n\t"
      "s_add_u32 s82, s82, 1\n\t"
      "v_mov_b32 v21, s82\n\t"
      "ds_write_b32 v18, v21\n\t"
      "s_branch JOIN2LN_%=\n\t"
      "S1X_%=:\n\t"
      "v_mfma_f32_16x16x128_f8f6f4 v[96:99], v[24:31], v[72:79], v[96:99] cbsz:1\n\t"
      "ds_read_b128 v[56:59], v11 offset:0\n\t"
      "ds_read_b128 v[60:63], v12 offset:0\n\t"
      "v_mfma_f32_16x16x128_f8f6f4 v[112:115], v[24:31], v[80:87], v[112:115] cbsz:1\n\t"
      "ds_read_b128 v[64:67], v11 offset:4096\n\t"
      "ds_read_b128 v[68:71], v12 offset:4096\n\t"
      "v_mfma_f32_16x16x128_f8f6f4 v[128:131], v[24:31], v[88:95], v[128:131] cbsz:1\n\t"
      "v_mfma_f32_16x16x128_f8f6f4 v[144:147], v[24:31], v[160:167], v[144:147] cbsz:1\n\t"
      "ds_read_b64_tr_b8 v[24:25], v13 offset:0\n\t"
      "ds_read_b64_tr_b8 v[26:27], v13 offset:1024\n\t"
      "v_mfma_f32_16x16x128_f8f6f4 v[100:103], v[32:39], v[72:79], v[100:103] cbsz:1\n\t"
      "ds_read_b64_tr_b8 v[28:29], v13 offset:8192\n\t"
      "ds_read_b64_tr_b8 v[30:31], v13 offset:9216\n\t"
      "v_mfma_f32_16x16x128_f8f6f4 v[116:119], v[32:39], v[80:87], v[116:119] cbsz:1\n\t"
      "v_mfma_f32_16x16x128_f8f6f4 v[132:135], v[32:39], v[88:95], v[132:135] cbsz:1\n\t"
      "v_mfma_f32_16x16x128_f8f6f4 v[148:151], v[32:39], v[160:167], v[148:151] cbsz:1\n\t"
      "ds_read_b64_tr_b8 v[32:33], v14 offset:0\n\t"
      "ds_read_b64_tr_b8 v[34:35], v14 offset:1024\n\t"
      "v_mfma_f32_16x16x128_f8f6f4 v[104:107], v[40:47], v[72:79], v[104:107] cbsz:1\n\t"
      "ds_read_b64_tr_b8 v[36:37], v14 offset:8192\n\t"
      "ds_read_b64_tr_b8 v[38:39], v14 offset:9216\n\t"
      "v_mfma_f32_16x16x128_f8f6f4 v[120:123], v[40:47], v[80:87], v[120:123] cbsz:1\n\t"
      "v_mfma_f32_16x16x128_f8f6f4 v[136:139], v[40:47], v[88:95], v[136:139] cbsz:1\n\t"
      "v_mfma_f32_16x16x128_f8f6f4 v[152:155], v[40:47], v[160:167], v[152:155] cbsz:1\n\t"
      "ds_read_b64_tr_b8 v[40:41], v15 offset:0\n\t"
      "ds_read_b64_tr_b8 v[42:43], v15 offset:1024\n\t"
      "v_mfma_f32_16x16x128_f8f6f4 v[108:111], v[48:55], v[72:79], v[108:111] cbsz:1\n\t"
      "ds_read_b64_tr_b8 v[44:45], v15 offset:8192\n\t"
      "ds_read_b64_tr_b8 v[46:47], v15 offset:9216\n\t"
      "s_add_u32 s82, s82, 1\n\t"
      "v_mov_b32 v21, s82\n\t"
      "v_mfma_f32_16x16x128_f8f6f4 v[124:127], v[48:55], v[80:87], v[124:127] cbsz:1\n\t"
      "v_mfma_f32_16x16x128_f8f6f4 v[140:143], v[48:55], v[88:95], v[140:143] cbsz:1\n\t"
      "v_mfma_f32_16x16x128_f8f6f4 v[156:159], v[48:55], v[160:167], v[156:159] cbsz:1\n\t"
      "ds_read_b64_tr_b8 v[48:49], v16 offset:0\n\t"
      "ds_read_b64_tr_b8 v[50:51], v16 offset:1024\n\t"
      "ds_read_b64_tr_b8 v[52:53], v16 offset:8192\n\t"
      "ds_read_b64_tr_b8 v[54:55], v16 offset:9216\n\t"
      "ds_write_b32 v18, v21\n\t"
      "SPINS1X_%=:\n\t"
      "ds_read_b64 v[22:23], v19\n\t"
      "s_waitcnt lgkmcnt(0)\n\t"
      "v_readfirstlane_b32 s83, v22\n\t"
      "s_cmp_gt_u32 s83, s81\n\t"
      "s_cbranch_scc1 GOS1X_%=\n\t"
      "s_sleep 1\n\t"
      "s_add_u32 s91, s91, 1\n\t"
      "s_and_b32 s96, s91, 1023\n\t"
      "s_cbranch_scc1 SPINS1X_%=\n\t"
      "s_memrealtime s[96:97]\n\t"
      "s_waitcnt lgkmcnt(0)\n\t"
      "s_sub_u32 s96, %[tlo], s96\n\t"
      "s_subb_u32 s97, %[thi], s97\n\t"
      "s_cbranch_scc0 SPINS1X_%=\n\t"
      "v_mov_b32 v22, 1\n\t"
      "ds_write_b32 v19, v22 offset:36\n\t"
      "GOS1X_%=:\n\t"
      "s_waitcnt lgkmcnt(0)\n\t"
      "s_branch JOIN1X_%=\n\t"
      "Z1X_%=:\n\t"
      "v_mfma_f32_16x16x128_f8f6f4 v[96:99], v[24:31], v[72:79], v[96:99] cbsz:1\n\t"
      "v_mfma_f32_16x16x128_f8f6f4 v[112:115], v[24:31], v[80:87], v[112:115] cbsz:1\n\t"
      "v_mfma_f32_16x16x128_f8f6f4 v[128:131], v[24:31], v[88:95], v[128:131] cbsz:1\n\t"
      "v_mfma_f32_16x16x128_f8f6f4 v[144:147], v[24:31], v[160:167], v[144:147] cbsz:1\n\t"
      "v_mfma_f32_16x16x128_f8f6f4 v[100:103], v[32:39], v[72:79], v[100:103] cbsz:1\n\t"
      "v_mfma_f32_16x16x128_f8f6f4 v[116:119], v[32:39], v[80:87], v[116:119] cbsz:1\n\t"
      "v_mfma_f32_16x16x128_f8f6f4 v[132:135], v[32:39], v[88:95], v[132:135] cbsz:1\n\t"
      "v_mfma_f32_16x16x128_f8f6f4 v[148:151], v[32:39], v[160:167], v[148:151] cbsz:1\n\t"
      "v_mfma_f32_16x16x128_f8f6f4 v[104:107], v[40:47], v[72:79], v[104:107] cbsz:1\n\t"
      "v_mfma_f32_16x16x128_f8f6f4 v[120:123], v[40:47], v[80:87], v[120:123] cbsz:1\n\t"
      "v_mfma_f32_16x16x128_f8f6f4 v[136:139], v[40:47], v[88:95], v[136:139] cbsz:1\n\t"
      "v_mfma_f32_16x16x128_f8f6f4 v[152:155], v[40:47], v[160:167], v[152:155] cbsz:1\n\t"
      "v_mfma_f32_16x16x128_f8f6f4 v[108:111], v[48:55], v[72:79], v[108:111] cbsz:1\n\t"
      "v_mfma_f32_16x16x128_f8f6f4 v[124:127], v[48:55], v[80:87], v[124:127] cbsz:1\n\t"
      "v_mfma_f32_16x16x128_f8f6f4 v[140:143], v[48:55], v[88:95], v[140:143] cbsz:1\n\t"
      "v_mfma_f32_16x16x128_f8f6f4 v[156:159], v[48:55], v[160:167], v[156:159] cbsz:1\n\t"
      "SPINZ1X_%=:\n\t"
      "ds_read_b64 v[22:23], v19\n\t"
      "s_waitcnt lgkmcnt(0)\n\t"
      "v_readfirstlane_b32 s83, v22\n\t"
      "v_readfirstlane_b32 s84, v23\n\t"
      "s_cmp_gt_u32 s84, s80\n\t"
      "s_cselect_b32 s89, s83, 0\n\t"
      "s_cmp_gt_u32 s89, s81\n\t"
      "s_cbranch_scc1 GOZ1X_%=\n\t"
      "s_sleep 1\n\t"
      "s_add_u32 s91, s91, 1\n\t"
      "s_and_b32 s96, s91, 1023\n\t"
      "s_cbranch_scc1 SPINZ1X_%=\n\t"
      "s_memrealtime s[96:97]\n\t"
      "s_waitcnt lgkmcnt(0)\n\t"
      "s_sub_u32 s96, %[tlo], s96\n\t"
      "s_subb_u32 s97, %[thi], s97\n\t"
      "s_cbranch_scc0 SPINZ1X_%=\n\t"
      "v_mov_b32 v22, 1\n\t"
      "ds_write_b32 v19, v22 offset:36\n\t"
      "GOZ1X_%=:\n\t"
      "ds_read_b128 v[56:59], v11 offset:0\n\t"
      "ds_read_b128 v[60:63], v12 offset:0\n\t"
      "ds_read_b128 v[64:67], v11 offset:4096\n\t"
      "ds_read_b128 v[68:71], v12 offset:4096\n\t"
      "ds_read_b64_tr_b8 v[24:25], v13 offset:0\n\t"
      "ds_read_b64_tr_b8 v[26:27], v13 offset:1024\n\t"
      "ds_read_b64_tr_b8 v[28:29], v13 offset:8192\n\t"
      "ds_read_b64_tr_b8 v[30:31], v13 offset:9216\n\t"
      "ds_read_b64_tr_b8 v[32:33], v14 offset:0\n\t"
      "ds_read_b64_tr_b8 v[34:35], v14 offset:1024\n\t"
      "ds_read_b64_tr_b8 v[36:37], v14 offset:8192\n\t"
      "ds_read_b64_tr_b8 v[38:39], v14 offset:9216\n\t"
      "ds_read_b64_tr_b8 v[40:41], v15 offset:0\n\t"
      "ds_read_b64_tr_b8 v[42:43], v15 offset:1024\n\t"
      "ds_read_b64_tr_b8 v[44:45], v15 offset:8192\n\t"
      "ds_read_b64_tr_b8 v[46:47], v15 offset:9216\n\t"
      "ds_read_b64_tr_b8 v[48:49], v16 offset:0\n\t"
      "ds_read_b64_tr_b8 v[50:51], v16 offset:1024\n\t"
      "ds_read_b64_tr_b8 v[52:53], v16 offset:8192\n\t"
      "ds_read_b64_tr_b8 v[54:55], v16 offset:9216\n\t"
      "s_add_u32 s82, s82, 1\n\t"
      "v_mov_b32 v21, s82\n\t"
      "ds_write_b32 v18, v21\n\t"
      "s_branch JOIN1X_%=\n\t"
      "SPINL2X_%=:\n\t"
      "ds_read_b64 v[22:23], v19\n\t"
      "s_waitcnt lgkmcnt(0)\n\t"
      "v_readfirstlane_b32 s83, v22\n\t"
      "s_cmp_gt_u32 s83, s81\n\t"
      "s_cbranch_scc1 GOL2X_%=\n\t"
      "s_sleep 1\n\t"
      "s_add_u32 s91, s91, 1\n\t"
      "s_and_b32 s96, s91, 1023\n\t"
      "s_cbranch_scc1 SPINL2X_%=\n\t"
      "s_memrealtime s[96:97]\n\t"
      "s_waitcnt lgkmcnt(0)\n\t"
      "s_sub_u32 s96, %[tlo], s96\n\t"
      "s_subb_u32 s97, %[thi], s97\n\t"
      "s_cbranch_scc0 SPINL2X_%=\n\t"
      "v_mov_b32 v22, 1\n\t"
      "ds_write_b32 v19, v22 offset:36\n\t"
      "GOL2X_%=:\n\t"
      "s_branch BACKL2X_%=\n\t"
      "S2X_%=:\n\t"
      "v_mfma_f32_16x16x128_f8f6f4 v[176:179], v[24:31], v[72:79], v[176:179] cbsz:1\n\t"
      "ds_read_b128 v[56:59], v11 offset:0\n\t"
      "ds_read_b128 v[60:63], v12 offset:0\n\t"
      "v_mfma_f32_16x16x128_f8f6f4 v[192:195], v[24:31], v[80:87], v[192:195] cbsz:1\n\t"
      "ds_read_b128 v[64:67], v11 offset:4096\n\t"
      "ds_read_b128 v[68:71], v12 offset:4096\n\t"
      "v_mfma_f32_16x16x128_f8f6f4 v[208:211], v[24:31], v[88:95], v[208:211] cbsz:1\n\t"
      "ds_read_b64_tr_b8 v[24:25], v13 offset:0\n\t"
      "ds_read_b64_tr_b8 v[26:27], v13 offset:1024\n\t"
      "v_mfma_f32_16x16x128_f8f6f4 v[180:183], v[32:39], v[72:79], v[180:183] cbsz:1\n\t"
      "ds_read_b64_tr_b8 v[28:29], v13 offset:8192\n\t"
      "ds_read_b64_tr_b8 v[30:31], v13 offset:9216\n\t"
      "v_mfma_f32_16x16x128_f8f6f4 v[196:199], v[32:39], v[80:87], v[196:199] cbsz:1\n\t"
      "v_mfma_f32_16x16x128_f8f6f4 v[212:215], v[32:39], v[88:95], v[212:215] cbsz:1\n\t"
      "ds_read_b64_tr_b8 v[32:33], v14 offset:0\n\t"
      "ds_read_b64_tr_b8 v[34:35], v14 offset:1024\n\t"
      "v_mfma_f32_16x16x128_f8f6f4 v[184:187], v[40:47], v[72:79], v[184:187] cbsz:1\n\t"
      "ds_read_b64_tr_b8 v[36:37], v14 offset:8192\n\t"
      "ds_read_b64_tr_b8 v[38:39], v14 offset:9216\n\t"
      "v_mfma_f32_16x16x128_f8f6f4 v[200:203], v[40:47], v[80:87], v[200:203] cbsz:1\n\t"
      "v_mfma_f32_16x16x128_f8f6f4 v[216:219], v[40:47], v[88:95], v[216:219] cbsz:1\n\t"
      "ds_read_b64_tr_b8 v[40:41], v15 offset:0\n\t"
      "ds_read_b64_tr_b8 v[42:43], v15 offset:1024\n\t"
      "v_mfma_f32_16x16x128_f8f6f4 v[188:191], v[48:55], v[72:79], v[188:191] cbsz:1\n\t"
      "ds_read_b64_tr_b8 v[44:45], v15 offset:8192\n\t"
      "ds_read_b64_tr_b8 v[46:47], v15 offset:9216\n\t"
      "s_add_u32 s82, s82, 1\n\t"
      "v_mov_b32 v21, s82\n\t"
      "v_mfma_f32_16x16x128_f8f6f4 v[204:207], v[48:55], v[80:87], v[204:207] cbsz:1\n\t"
      "v_mfma_f32_16x16x128_f8f6f4 v[220:223], v[48:55], v[88:95], v[220:223] cbsz:1\n\t"
      "ds_read_b64_tr_b8 v[48:49], v16 offset:0\n\t"
      "ds_read_b64_tr_b8 v[50:51], v16 offset:1024\n\t"
      "ds_read_b64_tr_b8 v[52:53], v16 offset:8192\n\t"
      "ds_read_b64_tr_b8 v[54:55], v16 offset:9216\n\t"
      "ds_write_b32 v18, v21\n\t"
      "SPINS2X_%=:\n\t"
      "ds_read_b64 v[22:23], v19\n\t"
      "s_waitcnt lgkmcnt(0)\n\t"
      "v_readfirstlane_b32 s83, v22\n\t"
      "s_cmp_gt_u32 s83, s81\n\t"
      "s_cbranch_scc1 GOS2X_%=\n\t"
      "s_sleep 1\n\t"
      "s_add_u32 s91, s91, 1\n\t"
      "s_and_b32 s96, s91, 1023\n\t"
      "s_cbranch_scc1 SPINS2X_%=\n\t"
      "s_memrealtime s[96:97]\n\t"
      "s_waitcnt lgkmcnt(0)\n\t"
      "s_sub_u32 s96, %[tlo], s96\n\t"
      "s_subb_u32 s97, %[thi], s97\n\t"
      "s_cbranch_scc0 SPINS2X_%=\n\t"
      "v_mov_b32 v22, 1\n\t"
      "ds_write_b32 v19, v22 offset:36\n\t"
      "GOS2X_%=:\n\t"
      "s_waitcnt lgkmcnt(0)\n\t"
      "s_branch JOIN2X_%=\n\t"
      "Z2X_%=:\n\t"
      "v_mfma_f32_16x16x128_f8f6f4 v[176:179], v[24:31], v[72:79], v[176:179] cbsz:1\n\t"
      "v_mfma_f32_16x16x128_f8f6f4 v[192:195], v[24:31], v[80:87], v[192:195] cbsz:1\n\t"
      "v_mfma_f32_16x16x128_f8f6f4 v[208:211], v[24:31], v[88:95], v[208:211] cbsz:1\n\t"
      "v_mfma_f32_16x16x128_f8f6f4 v[180:183], v[32:39], v[72:79], v[180:183] cbsz:1\n\t"
      "v_mfma_f32_16x16x128_f8f6f4 v[196:199], v[32:39], v[80:87], v[196:199] cbsz:1\n\t"
      "v_mfma_f32_16x16x128_f8f6f4 v[212:215], v[32:39], v[88:95], v[212:215] cbsz:1\n\t"
      "v_mfma_f32_16x16x128_f8f6f4 v[184:187], v[40:47], v[72:79], v[184:187] cbsz:1\n\t"
      "v_mfma_f32_16x16x128_f8f6f4 v[200:203], v[40:47], v[80:87], v[200:203] cbsz:1\n\t"
      "v_mfma_f32_16x16x128_f8f6f4 v[216:219], v[40:47], v[88:95], v[216:219] cbsz:1\n\t"
      "v_mfma_f32_16x16x128_f8f6f4 v[188:191], v[48:55], v[72:79], v[188:191] cbsz:1\n\t"
      "v_mfma_f32_16x16x128_f8f6f4 v[204:207], v[48:55], v[80:87], v[204:207] cbsz:1\n\t"
      "v_mfma_f32_16x16x128_f8f6f4 v[220:223], v[48:55], v[88:95], v[220:223] cbsz:1\n\t"
      "SPINZ2X_%=:\n\t"
      "ds_read_b64 v[22:23], v19\n\t"
      "s_waitcnt lgkmcnt(0)\n\t"
      "v_readfirstlane_b32 s83, v22\n\t"
      "v_readfirstlane_b32 s84, v23\n\t"
      "s_cmp_gt_u32 s84, s80\n\t"
      "s_cselect_b32 s89, s83, 0\n\t"
      "s_cmp_gt_u32 s89, s81\n\t"
      "s_cbranch_scc1 GOZ2X_%=\n\t"
      "s_sleep 1\n\t"
      "s_add_u32 s91, s91, 1\n\t"
      "s_and_b32 s96, s91, 1023\n\t"
      "s_cbranch_scc1 SPINZ2X_%=\n\t"
      "s_memrealtime s[96:97]\n\t"
      "s_waitcnt lgkmcnt(0)\n\t"
      "s_sub_u32 s96, %[tlo], s96\n\t"
      "s_subb_u32 s97, %[thi], s97\n\t"
      "s_cbranch_scc0 SPINZ2X_%=\n\t"
      "v_mov_b32 v22, 1\n\t"
      "ds_write_b32 v19, v22 offset:36\n\t"
      "GOZ2X_%=:\n\t"
      "ds_read_b128 v[56:59], v11 offset:0\n\t"
      "ds_read_b128 v[60:63], v12 offset:0\n\t"
      "ds_read_b128 v[64:67], v11 offset:4096\n\t"
      "ds_read_b128 v[68:71], v12 offset:4096\n\t"
      "ds_read_b64_tr_b8 v[24:25], v13 offset:0\n\t"
      "ds_read_b64_tr_b8 v[26:27], v13 offset:1024\n\t"
      "ds_read_b64_tr_b8 v[28:29], v13 offset:8192\n\t"
      "ds_read_b64_tr_b8 v[30:31], v13 offset:9216\n\t"
      "ds_read_b64_tr_b8 v[32:33], v14 offset:0\n\t"
      "ds_read_b64_tr_b8 v[34:35], v14 offset:1024\n\t"
      "ds_read_b64_tr_b8 v[36:37], v14 offset:8192\n\t"
      "ds_read_b64_tr_b8 v[38:39], v14 offset:9216\n\t"
      "ds_read_b64_tr_b8 v[40:41], v15 offset:0\n\t"
      "ds_read_b64_tr_b8 v[42:43], v15 offset:1024\n\t"
      "ds_read_b64_tr_b8 v[44:45], v15 offset:8192\n\t"
      "ds_read_b64_tr_b8 v[46:47], v15 offset:9216\n\t"
      "ds_read_b64_tr_b8 v[48:49], v16 offset:0\n\t"
      "ds_read_b64_tr_b8 v[50:51], v16 offset:1024\n\t"
      "ds_read_b64_tr_b8 v[52:53], v16 offset:8192\n\t"
      "ds_read_b64_tr_b8 v[54:55], v16 offset:9216\n\t"
      "s_add_u32 s82, s82, 1\n\t"
      "v_mov_b32 v21, s82\n\t"
      "ds_write_b32 v18, v21\n\t"
      "s_branch JOIN2X_%=\n\t"
      "SPINL2LX_%=:\n\t"
      "ds_read_b64 v[22:23], v19\n\t"
      "s_waitcnt lgkmcnt(0)\n\t"
      "v_readfirstlane_b32 s83, v22\n\t"
      "s_cmp_gt_u32 s83, s81\n\t"
      "s_cbranch_scc1 GOL2LX_%=\n\t"
      "s_sleep 1\n\t"
      "s_add_u32 s91, s91, 1\n\t"
      "s_and_b32 s96, s91, 1023\n\t"
      "s_cbranch_scc1 SPINL2LX_%=\n\t"
      "s_memrealtime s[96:97]\n\t"
      "s_waitcnt lgkmcnt(0)\n\t"
      "s_sub_u32 s96, %[tlo], s96\n\t"
      "s_subb_u32 s97, %[thi], s97\n\t"
      "s_cbranch_scc0 SPINL2LX_%=\n\t"
      "v_mov_b32 v22, 1\n\t"
      "ds_write_b32 v19, v22 offset:36\n\t"
      "GOL2LX_%=:\n\t"
      "s_branch BACKL2LX_%=\n\t"
      "S2LX_%=:\n\t"
      "v_mfma_f32_16x16x128_f8f6f4 v[176:179], v[24:31], v[72:79], v[176:179] cbsz:1\n\t"
      "v_mfma_f32_16x16x128_f8f6f4 v[192:195], v[24:31], v[80:87], v[192:195] cbsz:1\n\t"
      "v_mfma_f32_16x16x128_f8f6f4 v[208:211], v[24:31], v[88:95], v[208:211] cbsz:1\n\t"
      "ds_read_b64_tr_b8 v[24:25], v13 offset:0\n\t"
      "ds_read_b64_tr_b8 v[26:27], v13 offset:1024\n\t"
      "v_mfma_f32_16x16x128_f8f6f4 v[180:183], v[32:39], v[72:79], v[180:183] cbsz:1\n\t"
      "ds_read_b64_tr_b8 v[28:29], v13 offset:8192\n\t"
      "ds_read_b64_tr_b8 v[30:31], v13 offset:9216\n\t"
      "v_mfma_f32_16x16x128_f8f6f4 v[196:199], v[32:39], v[80:87], v[196:199] cbsz:1\n\t"
      "v_mfma_f32_16x16x128_f8f6f4 v[212:215], v[32:39], v[88:95], v[212:215] cbsz:1\n\t"
      "ds_read_b64_tr_b8 v[32:33], v14 offset:0\n\t"
      "ds_read_b64_tr_b8 v[34:35], v14 offset:1024\n\t"
      "v_mfma_f32_16x16x128_f8f6f4 v[184:187], v[40:47], v[72:79], v[184:187] cbsz:1\n\t"
      "ds_read_b64_tr_b8 v[36:37], v14 offset:8192\n\t"
      "ds_read_b64_tr_b8 v[38:39], v14 offset:9216\n\t"
      "v_mfma_f32_16x16x128_f8f6f4 v[200:203], v[40:47], v[80:87], v[200:203] cbsz:1\n\t"
      "v_mfma_f32_16x16x128_f8f6f4 v[216:219], v[40:47], v[88:95], v[216:219] cbsz:1\n\t"
      "ds_read_b64_tr_b8 v[40:41], v15 offset:0\n\t"
      "ds_read_b64_tr_b8 v[42:43], v15 offset:1024\n\t"
      "v_mfma_f32_16x16x128_f8f6f4 v[188:191], v[48:55], v[72:79], v[188:191] cbsz:1\n\t"
      "ds_read_b64_tr_b8 v[44:45], v15 offset:8192\n\t"
      "ds_read_b64_tr_b8 v[46:47], v15 offset:9216\n\t"
      "s_add_u32 s82, s82, 1\n\t"
      "v_mov_b32 v21, s82\n\t"
      "v_mfma_f32_16x16x128_f8f6f4 v[204:207], v[48:55], v[80:87], v[204:207] cbsz:1\n\t"
      "v_mfma_f32_16x16x128_f8f6f4 v[220:223], v[48:55], v[88:95], v[220:223] cbsz:1\n\t"
      "ds_read_b64_tr_b8 v[48:49], v16 offset:0\n\t"
      "ds_read_b64_tr_b8 v[50:51], v16 offset:1024\n\t"
      "ds_read_b64_tr_b8 v[52:53], v16 offset:8192\n\t"
      "ds_read_b64_tr_b8 v[54:55], v16 offset:9216\n\t"
      "ds_write_b32 v18, v21\n\t"
      "SPINS2LX_%=:\n\t"
      "ds_read_b64 v[22:23], v19\n\t"
      "s_waitcnt lgkmcnt(0)\n\t"
      "v_readfirstlane_b32 s83, v22\n\t"
      "s_cmp_gt_u32 s83, s81\n\t"
      "s_cbranch_scc1 GOS2LX_%=\n\t"
      "s_sleep 1\n\t"
      "s_add_u32 s91, s91, 1\n\t"
      "s_and_b32 s96, s91, 1023\n\t"
      "s_cbranch_scc1 SPINS2LX_%=\n\t"
      "s_memrealtime s[96:97]\n\t"
      "s_waitcnt lgkmcnt(0)\n\t"
      "s_sub_u32 s96, %[tlo], s96\n\t"
      "s_subb_u32 s97, %[thi], s97\n\t"
      "s_cbranch_scc0 SPINS2LX_%=\n\t"
      "v_mov_b32 v22, 1\n\t"
      "ds_write_b32 v19, v22 offset:36\n\t"
      "GOS2LX_%=:\n\t"
      "ds_read_b128 v[56:59], v9 offset:0\n\t"
      "ds_read_b128 v[60:63], v10 offset:0\n\t"
      "ds_read_b128 v[64:67], v9 offset:2048\n\t"
      "ds_read_b128 v[68:71], v10 offset:2048\n\t"
      "s_waitcnt lgkmcnt(0)\n\t"
      "s_branch JOIN2LX_%=\n\t"
      "Z2LX_%=:\n\t"
      "v_mfma_f32_16x16x128_f8f6f4 v[176:179], v[24:31], v[72:79], v[176:179] cbsz:1\n\t"
      "v_mfma_f32_16x16x128_f8f6f4 v[192:195], v[24:31], v[80:87], v[192:195] cbsz:1\n\t"
      "v_mfma_f32_16x16x128_f8f6f4 v[208:211], v[24:31], v[88:95], v[208:211] cbsz:1\n\t"
      "v_mfma_f32_16x16x128_f8f6f4 v[180:183], v[32:39], v[72:79], v[180:183] cbsz:1\n\t"
      "v_mfma_f32_16x16x128_f8f6f4 v[196:199], v[32:39], v[80:87], v[196:199] cbsz:1\n\t"
      "v_mfma_f32_16x16x128_f8f6f4 v[212:215], v[32:39], v[88:95], v[212:215] cbsz:1\n\t"
      "v_mfma_f32_16x16x128_f8f6f4 v[184:187], v[40:47], v[72:79], v[184:187] cbsz:1\n\t"
      "v_mfma_f32_16x16x128_f8f6f4 v[200:203], v[40:47], v[80:87], v[200:203] cbsz:1\n\t"
      "v_mfma_f32_16x16x128_f8f6f4 v[216:219], v[40:47], v[88:95], v[216:219] cbsz:1\n\t"
      "v_mfma_f32_16x16x128_f8f6f4 v[188:191], v[48:55], v[72:79], v[188:191] cbsz:1\n\t"
      "v_mfma_f32_16x16x128_f8f6f4 v[204:207], v[48:55], v[80:87], v[204:207] cbsz:1\n\t"
      "v_mfma_f32_16x16x128_f8f6f4 v[220:223], v[48:55], v[88:95], v[220:223] cbsz:1\n\t"
      "SPINZ2LX_%=:\n\t"
      "ds_read_b64 v[22:23], v19\n\t"
      "s_waitcnt lgkmcnt(0)\n\t"
      "v_readfirstlane_b32 s83, v22\n\t"
      "v_readfirstlane_b32 s84, v23\n\t"
      "s_cmp_gt_u32 s84, s80\n\t"
      "s_cselect_b32 s89, s83, 0\n\t"
      "s_cmp_gt_u32 s89, s81\n\t"
      "s_cbranch_scc1 GOZ2LX_%=\n\t"
      "s_sleep 1\n\t"
      "s_add_u32 s91, s91, 1\n\t"
      "s_and_b32 s96, s91, 1023\n\t"
      "s_cbranch_scc1 SPINZ2LX_%=\n\t"
      "s_memrealtime s[96:97]\n\t"
      "s_waitcnt lgkmcnt(0)\n\t"
      "s_sub_u32 s96, %[tlo], s96\n\t"
      "s_subb_u32 s97, %[thi], s97\n\t"
      "s_cbranch_scc0 SPINZ2LX_%=\n\t"
      "v_mov_b32 v22, 1\n\t"
      "ds_write_b32 v19, v22 offset:36\n\t"
      "GOZ2LX_%=:\n\t"
      "ds_read_b128 v[56:59], v9 offset:0\n\t"
      "ds_read_b128 v[60:63], v10 offset:0\n\t"
      "ds_read_b128 v[64:67], v9 offset:2048\n\t"
      "ds_read_b128 v[68:71], v10 offset:2048\n\t"
      "ds_read_b64_tr_b8 v[24:25], v13 offset:0\n\t"
      "ds_read_b64_tr_b8 v[26:27], v13 offset:1024\n\t"
      "ds_read_b64_tr_b8 v[28:29], v13 offset:8192\n\t"
      "ds_read_b64_tr_b8 v[30:31], v13 offset:9216\n\t"
      "ds_read_b64_tr_b8 v[32:33], v14 offset:0\n\t"
      "ds_read_b64_tr_b8 v[34:35], v14 offset:1024\n\t"
      "ds_read_b64_tr_b8 v[36:37], v14 offset:8192\n\t"
      "ds_read_b64_tr_b8 v[38:39], v14 offset:9216\n\t"
      "ds_read_b64_tr_b8 v[40:41], v15 offset:0\n\t"
      "ds_read_b64_tr_b8 v[42:43], v15 offset:1024\n\t"
      "ds_read_b64_tr_b8 v[44:45], v15 offset:8192\n\t"
      "ds_read_b64_tr_b8 v[46:47], v15 offset:9216\n\t"
      "ds_read_b64_tr_b8 v[48:49], v16 offset:0\n\t"
      "ds_read_b64_tr_b8 v[50:51], v16 offset:1024\n\t"
      "ds_read_b64_tr_b8 v[52:53], v16 offset:8192\n\t"
      "ds_read_b64_tr_b8 v[54:55], v16 offset:9216\n\t"
      "s_add_u32 s82, s82, 1\n\t"
      "v_mov_b32 v21, s82\n\t"
      "ds_write_b32 v18, v21\n\t"
      "s_branch JOIN2LX_%=\n\t"
      "S3_%=:\n\t"
      "v_mfma_f32_16x16x128_f8f6f4 v[192:195], v[24:31], v[72:79], v[192:195] cbsz:1\n\t"
      "v_mfma_f32_16x16x128_f8f6f4 v[208:211], v[24:31], v[80:87], v[208:211] cbsz:1\n\t"
      "ds_read_b64_tr_b8 v[24:25], v13 offset:0\n\t"
      "ds_read_b64_tr_b8 v[26:27], v13 offset:1024\n\t"
      "ds_read_b64_tr_b8 v[28:29], v13 offset:8192\n\t"
      "v_mfma_f32_16x16x128_f8f6f4 v[196:199], v[32:39], v[72:79], v[196:199] cbsz:1\n\t"
      "ds_read_b64_tr_b8 v[30:31], v13 offset:9216\n\t"
      "v_mfma_f32_16x16x128_f8f6f4 v[212:215], v[32:39], v[80:87], v[212:215] cbsz:1\n\t"
      "ds_read_b64_tr_b8 v[32:33], v14 offset:0\n\t"
      "ds_read_b64_tr_b8 v[34:35], v14 offset:1024\n\t"
      "ds_read_b64_tr_b8 v[36:37], v14 offset:8192\n\t"
      "v_mfma_f32_16x16x128_f8f6f4 v[200:203], v[40:47], v[72:79], v[200:203] cbsz:1\n\t"
      "ds_read_b64_tr_b8 v[38:39], v14 offset:9216\n\t"
      "v_mfma_f32_16x16x128_f8f6f4 v[216:219], v[40:47], v[80:87], v[216:219] cbsz:1\n\t"
      "ds_read_b64_tr_b8 v[40:41], v15 offset:0\n\t"
      "ds_read_b64_tr_b8 v[42:43], v15 offset:1024\n\t"
      "ds_read_b64_tr_b8 v[44:45], v15 offset:8192\n\t"
      "v_mfma_f32_16x16x128_f8f6f4 v[204:207], v[48:55], v[72:79], v[204:207] cbsz:1\n\t"
      "ds_read_b64_tr_b8 v[46:47], v15 offset:9216\n\t"
      "s_add_u32 s82, s82, 1\n\t"
      "v_mov_b32 v21, s82\n\t"
      "v_mfma_f32_16x16x128_f8f6f4 v[220:223], v[48:55], v[80:87], v[220:223] cbsz:1\n\t"
      "ds_read_b64_tr_b8 v[48:49], v16 offset:0\n\t"
      "ds_read_b64_tr_b8 v[50:51], v16 offset:1024\n\t"
      "ds_read_b64_tr_b8 v[52:53], v16 offset:8192\n\t"
      "ds_read_b64_tr_b8 v[54:55], v16 offset:9216\n\t"
      "ds_write_b32 v18, v21\n\t"
      "SPINS3_%=:\n\t"
      "ds_read_b64 v[22:23], v19\n\t"
      "s_waitcnt lgkmcnt(0)\n\t"
      "v_readfirstlane_b32 s83, v22\n\t"
      "s_cmp_gt_u32 s83, s81\n\t"
      "s_cbranch_scc1 GOS3_%=\n\t"
      "s_sleep 1\n\t"
      "s_add_u32 s91, s91, 1\n\t"
      "s_and_b32 s96, s91, 1023\n\t"
      "s_cbranch_scc1 SPINS3_%=\n\t"
      "s_memrealtime s[96:97]\n\t"
      "s_waitcnt lgkmcnt(0)\n\t"
      "s_sub_u32 s96, %[tlo], s96\n\t"
      "s_subb_u32 s97, %[thi], s97\n\t"
      "s_cbranch_scc0 SPINS3_%=\n\t"
      "v_mov_b32 v22, 1\n\t"
      "ds_write_b32 v19, v22 offset:36\n\t"
      "GOS3_%=:\n\t"
      "ds_read_b128 v[56:59], v9 offset:0\n\t"
      "ds_read_b128 v[60:63], v10 offset:0\n\t"
      "ds_read_b128 v[64:67], v9 offset:2048\n\t"
      "ds_read_b128 v[68:71], v10 offset:2048\n\t"
      "s_waitcnt lgkmcnt(0)\n\t"
      "s_branch JOIN3_%=\n\t"
      "Z3_%=:\n\t"
      "v_mfma_f32_16x16x128_f8f6f4 v[192:195], v[24:31], v[72:79], v[192:195] cbsz:1\n\t"
      "v_mfma_f32_16x16x128_f8f6f4 v[208:211], v[24:31], v[80:87], v[208:211] cbsz:1\n\t"
      "v_mfma_f32_16x16x128_f8f6f4 v[196:199], v[32:39], v[72:79], v[196:199] cbsz:1\n\t"
      "v_mfma_f32_16x16x128_f8f6f4 v[212:215], v[32:39], v[80:87], v[212:215] cbsz:1\n\t"
      "v_mfma_f32_16x16x128_f8f6f4 v[200:203], v[40:47], v[72:79], v[200:203] cbsz:1\n\t"
      "v_mfma_f32_16x16x128_f8f6f4 v[216:219], v[40:47], v[80:87], v[216:219] cbsz:1\n\t"
      "v_mfma_f32_16x16x128_f8f6f4 v[204:207], v[48:55], v[72:79], v[204:207] cbsz:1\n\t"
      "v_mfma_f32_16x16x128_f8f6f4 v[220:223], v[48:55], v[80:87], v[220:223] cbsz:1\n\t"
      "s_cmp_ge_u32 s80, s94\n\t"
      "s_cbranch_scc1 EPI_%=\n\t"
      "SPINZ3_%=:\n\t"
      "ds_read_b64 v[22:23], v19\n\t"
      "s_waitcnt lgkmcnt(0)\n\t"
      "v_readfirstlane_b32 s83, v22\n\t"
      "v_readfirstlane_b32 s84, v23\n\t"
      "s_cmp_gt_u32 s84, s80\n\t"
      "s_cselect_b32 s89, s83, 0\n\t"
      "s_cmp_gt_u32 s89, s81\n\t"
      "s_cbranch_scc1 GOZ3_%=\n\t"
      "s_sleep 1\n\t"
      "s_add_u32 s91, s91, 1\n\t"
      "s_and_b32 s96, s91, 1023\n\t"
      "s_cbranch_scc1 SPINZ3_%=\n\t"
      "s_memrealtime s[96:97]\n\t"
      "s_waitcnt lgkmcnt(0)\n\t"
      "s_sub_u32 s96, %[tlo], s96\n\t"
      "s_subb_u32 s97, %[thi], s97\n\t"
      "s_cbranch_scc0 SPINZ3_%=\n\t"
      "v_mov_b32 v22, 1\n\t"
      "ds_write_b32 v19, v22 offset:36\n\t"
      "GOZ3_%=:\n\t"
      "ds_read_b128 v[56:59], v9 offset:0\n\t"
      "ds_read_b128 v[60:63], v10 offset:0\n\t"
      "ds_read_b128 v[64:67], v9 offset:2048\n\t"
      "ds_read_b128 v[68:71], v10 offset:2048\n\t"
      "ds_read_b64_tr_b8 v[24:25], v13 offset:0\n\t"
      "ds_read_b64_tr_b8 v[26:27], v13 offset:1024\n\t"
      "ds_read_b64_tr_b8 v[28:29], v13 offset:8192\n\t"
      "ds_read_b64_tr_b8 v[30:31], v13 offset:9216\n\t"
      "ds_read_b64_tr_b8 v[32:33], v14 offset:0\n\t"
      "ds_read_b64_tr_b8 v[34:35], v14 offset:1024\n\t"
      "ds_read_b64_tr_b8 v[36:37], v14 offset:8192\n\t"
      "ds_read_b64_tr_b8 v[38:39], v14 offset:9216\n\t"
      "ds_read_b64_tr_b8 v[40:41], v15 offset:0\n\t"
      "ds_read_b64_tr_b8 v[42:43], v15 offset:1024\n\t"
      "ds_read_b64_tr_b8 v[44:45], v15 offset:8192\n\t"
      "ds_read_b64_tr_b8 v[46:47], v15 offset:9216\n\t"
      "ds_read_b64_tr_b8 v[48:49], v16 offset:0\n\t"
      "ds_read_b64_tr_b8 v[50:51], v16 offset:1024\n\t"
      "ds_read_b64_tr_b8 v[52:53], v16 offset:8192\n\t"
      "ds_read_b64_tr_b8 v[54:55], v16 offset:9216\n\t"
      "s_add_u32 s82, s82, 1\n\t"
      "v_mov_b32 v21, s82\n\t"
      "ds_write_b32 v18, v21\n\t"
      "s_branch JOIN3_%=\n\t"
      "END_%=:\n\t"
      "AG_C1:\n\t"
      "s_setprio 0\n\t"
      :
      : [sout] "s"(sout), [soute] "s"(soute), [soute2] "s"(soute2), [sext] "s"(sext), [dt] "s"(dt),
        [ixa] "v"(ixa), [iwa] "v"(iwa), [vout] "v"(vout), [vsw0] "v"(vsw0), [vsr] "v"(vsr), [sfree] "s"(L_FREE + wid * 4), [sdxe] "s"(L_E - wm * 6144),
        [ro] "s"(ro), [sscale] "s"(sscale), [tlo] "s"((unsigned)tlim), [thi] "s"((unsigned)(tlim >> 32))
      : "memory", "scc", "vcc",
        "s80", "s81", "s82", "s83", "s84", "s85", "s86", "s87", "s88", "s89", "s90", "s91", "s92", "s93", "s94", "s95",
        "s96", "s97", "v7", "v8", "v9", "v10", "v11", "v12", "v13", "v14", "v15", "v16", "v17", "v18", "v19", "v20",
        "v21", "v22", "v23", "v24", "v25", "v26", "v27", "v28", "v29", "v30", "v31", "v32", "v33", "v34", "v35", "v36",
        "v37", "v38", "v39", "v40", "v41", "v42", "v43", "v44", "v45", "v46", "v47", "v48", "v49", "v50", "v51", "v52",
        "v53", "v54", "v55", "v56", "v57", "v58", "v59", "v60", "v61", "v62", "v63", "v64", "v65", "v66", "v67", "v68",
        "v69", "v70", "v71", "v72", "v73", "v74", "v75", "v76", "v77", "v78", "v79", "v80", "v81", "v82", "v83", "v84",
        "v85", "v86", "v87", "v88", "v89", "v90", "v91", "v92", "v93", "v94", "v95", "v96", "v97", "v98", "v99", "v100",
        "v101", "v102", "v103", "v104", "v105", "v106", "v107", "v108", "v109", "v110", "v111", "v112", "v113", "v114", "v115", "v116",
        "v117", "v118", "v119", "v120", "v121", "v122", "v123", "v124", "v125", "v126", "v127", "v128", "v129", "v130", "v131", "v132",
        "v133", "v134", "v135", "v136", "v137", "v138", "v139", "v140", "v141", "v142", "v143", "v144", "v145", "v146", "v147", "v148",
        "v149", "v150", "v151", "v152", "v153", "v154", "v155", "v156", "v157", "v158", "v159", "v160", "v161", "v162", "v163", "v164",
        "v165", "v166", "v167", "v168", "v169", "v170", "v171", "v172", "v173", "v174", "v175", "v176", "v177", "v178", "v179", "v180",
        "v181", "v182", "v183", "v184", "v185", "v186", "v187", "v188", "v189", "v190", "v191", "v192", "v193", "v194", "v195", "v196",
        "v197", "v198", "v199", "v200", "v201", "v202", "v203", "v204", "v205", "v206", "v207", "v208", "v209", "v210", "v211", "v212",
        "v213", "v214", "v215", "v216", "v217", "v218", "v219", "v220", "v221", "v222", "v223", "v224", "v225", "v226", "v227", "v228",
        "v229", "v230", "v231", "v232", "v233", "v234", "v235", "v236", "v237", "v238", "v239", "v240", "v241", "v242", "v243", "v244",
        "v245", "v246", "v247", "v248", "v249", "v250", "v251", "v252", "v253", "v254", "v255"
  );
}

extern "C" void kernel_launch(int rank, int nranks, ncclComm_t comm,
                              hipStream_t stream,
                              void* const* d_in, const int* in_sizes, int n_in,
                              void* d_out, int out_size,
                              const int* in_dtypes, int out_dtype,
                              void* const* d_out_all,
                              void* const* d_ws_all, size_t ws_bytes) {
  if (nranks != NR || n_in < 4 || ws_bytes < WS_NEED || (unsigned)rank >= (unsigned)NR ||
      in_sizes[0] != MS * KD || in_sizes[1] != KD * NL || out_size != NR * MS * NL) {
    fprintf(stderr, "ag_gemm: unexpected problem shape\n");
    return;
  }
  HIP_CHECK(hipFuncSetAttribute((const void*)ag_gemm, hipFuncAttributeMaxDynamicSharedMemorySize, LDS_BYTES));
  Params p;
  for (int i = 0; i < NR; ++i) p.ws[i] = (char*)d_ws_all[(rank + i) % NR];
  p.x = (const char*)d_in[0];
  p.w = (const char*)d_in[1];
  p.sx = (const float*)d_in[2];
  p.sw = (const float*)d_in[3];
  p.out = (float*)d_out;
  p.dbg = nullptr;
  p.rank = rank;
  p.pad = 0;
  ag_gemm<<<256, 512, LDS_BYTES, stream>>>(p);
  HIP_CHECK(hipGetLastError());
}
```
